```python
import numpy as np

kernel_name = "gpu_gemm_a2a_m4096_k4096_n4096_bf16_qe4m3"


ABSMAX_RTOL = 0.09

COMPUTE_DTYPE = "bfloat16"
EPILOGUE = "quant_e4m3"
IN_QUANT = None
OUT_QUANT = "e4m3"
IN_DTYPES = {"x": "bfloat16", "w_mat": "bfloat16"}
WORLD_SIZE = 8

def _to_bf16(x):
    u = np.ascontiguousarray(x, dtype=np.float32).view(np.uint32)
    return ((u + 0x7FFF + ((u >> 16) & 1)) & 0xFFFF0000).view(np.float32)

def setup_inputs():

    x = _to_bf16(np.random.randn(4096, 4096).astype(np.float32))
    w_mat = _to_bf16((np.random.randn(4096, 4096) / np.sqrt(4096)).astype(np.float32))
    return {"x": x, "w_mat": w_mat}

def _snap_fp8(x, mant_bits, max_abs, min_normal):
    a = np.abs(x.astype(np.float64))
    _m, _e = np.frexp(a)
    q = np.where(a >= min_normal, np.ldexp(1.0, _e - 1 - mant_bits),
                 min_normal * 2.0 ** -mant_bits)
    snapped = np.minimum(np.round(a / q) * q, max_abs)
    return (np.sign(x) * snapped).astype(np.float32)
def _quant_dequant(y, amax):
    scale = amax / 448.0
    q = _snap_fp8(y / scale, 3, 448.0, 2.0 ** -6)
    return (q * scale).astype(np.float32)
def _epilogue(y):
    return _quant_dequant(y, np.abs(y).max())

def reference(x, w_mat):
    y = (x @ w_mat).astype(np.float32)
    return _epilogue(y)
```

```cpp
#include <hip/hip_runtime.h>
#include <rccl/rccl.h>
#include <cstdio>

#define HIP_CHECK(expr) do { hipError_t _e = (expr); if (_e != hipSuccess) { \
    fprintf(stderr, "HIP error %s at %s:%d\n", hipGetErrorString(_e), __FILE__, __LINE__); \
    return; } } while (0)

namespace {

typedef int i32x4 __attribute__((ext_vector_type(4)));
typedef unsigned u32x4 __attribute__((ext_vector_type(4)));
typedef short v4s __attribute__((ext_vector_type(4)));
typedef short v8s __attribute__((ext_vector_type(8)));
typedef __bf16 v8bf __attribute__((ext_vector_type(8)));
typedef float f4 __attribute__((ext_vector_type(4)));
#define LAS __attribute__((address_space(3)))

constexpr int NR = 8;
constexpr int M = 512, N = 4096, K = 4096;
constexpr int NWG = 256;
constexpr int OUT_LD = N / NR;

#define BAND_HALF 128
#define NSTAGE 3
constexpr int NS = NSTAGE;
constexpr int STAGE_B = 32768;
constexpr int S1 = (K / 2 - BAND_HALF) / 64;
constexpr int S2 = 2 * BAND_HALF / 64;
constexpr int S = S1 + S2;
static_assert(S1 % 2 == 0 && S2 % 2 == 0 && S2 >= 4 && (NS == 3 || NS == 4), "stage counts");
constexpr int LDS_MISC = NS * STAGE_B;
constexpr int LDS_BYTES = LDS_MISC + 256;

constexpr size_t WS_CNT = 0;
constexpr size_t WS_TMO = 16384;
constexpr size_t WS_AMAX = 32768;
constexpr size_t WS_DONE = 65536;
constexpr size_t WS_XFLAG = 131072;
constexpr size_t WS_XBUF = 1u << 20;
constexpr size_t WS_STAMP = WS_XBUF + (8u << 20);
constexpr size_t WS_NEED = WS_STAMP + (1u << 20);

constexpr unsigned long long DEADLINE = 50000000ull;

struct Params {
  const unsigned short* x;
  const unsigned short* w;
  unsigned char* ws_self;
  int rank, pad;
  unsigned char* out[NR];
  unsigned char* ws[NR];
};
#define GAS __attribute__((address_space(1)))

__device__ __forceinline__ void dma16(unsigned voff, i32x4 srd, unsigned soff, unsigned m0v) {
  asm volatile("s_mov_b32 m0, %3\n\ts_nop 0\n\tbuffer_load_dwordx4 %0, %1, %2 offen lds" ::"v"(voff), "s"(srd), "s"(soff), "s"(m0v) : "memory");
}
template <int N_> __device__ __forceinline__ void waitvm() { asm volatile("s_waitcnt vmcnt(%0)" ::"i"(N_) : "memory"); }
__device__ __forceinline__ void wg_barrier() {
  asm volatile("" ::: "memory");
  __builtin_amdgcn_s_barrier();
  asm volatile("" ::: "memory");
}
__device__ __forceinline__ __amdgpu_buffer_rsrc_t make_rsrc(const void* p, unsigned bytes) {
  return __builtin_amdgcn_make_buffer_rsrc(const_cast<void*>(p), 0, bytes, 0x00020000);
}
__device__ __forceinline__ unsigned wave_max_u32(unsigned v) {
  v = max(v, (unsigned)__builtin_amdgcn_update_dpp(0, (int)v, 0xB1, 0xf, 0xf, true));
  v = max(v, (unsigned)__builtin_amdgcn_update_dpp(0, (int)v, 0x4E, 0xf, 0xf, true));
  v = max(v, (unsigned)__builtin_amdgcn_update_dpp(0, (int)v, 0x141, 0xf, 0xf, true));
  v = max(v, (unsigned)__builtin_amdgcn_update_dpp(0, (int)v, 0x140, 0xf, 0xf, true));
  const unsigned a = (unsigned)__builtin_amdgcn_readlane((int)v, 0), b = (unsigned)__builtin_amdgcn_readlane((int)v, 16);
  const unsigned c = (unsigned)__builtin_amdgcn_readlane((int)v, 32), d = (unsigned)__builtin_amdgcn_readlane((int)v, 48);
  return max(max(a, b), max(c, d));
}
constexpr int AUX_AGENT = 16;
constexpr int AUX_SYS = 17;

#define STAMP(i) do { } while (0)

__global__ __launch_bounds__(512) void gemm_a2a_fp8(Params P) {
  extern __shared__ __attribute__((aligned(16))) char smem[];
  const int b = blockIdx.x, xcd = b & 7, idx = b >> 3;
  const int kh = xcd & 1, nb = xcd >> 1;
  const int m0 = (idx & 3) * 128, n0 = nb * 1024 + (idx >> 2) * 128;
  const int wv = __builtin_amdgcn_readfirstlane(threadIdx.x >> 6), lane = threadIdx.x & 63;
  const int rank = P.rank;
  unsigned char* const ws = P.ws_self;
  LAS char* const L = (LAS char*)smem;
  LAS unsigned* const Lm = (LAS unsigned*)(L + LDS_MISC);
  unsigned* const tmo = (unsigned*)(ws + WS_TMO);
  STAMP(0);
  unsigned* const cnt = (unsigned*)(ws + WS_CNT + (size_t)b * 64);
  const int dst = n0 / OUT_LD;
  unsigned seq;
  unsigned char* peer_ws;
  unsigned char* out_dst;
  unsigned char* ws_dst;

  const int fr = lane & 15, fq = lane >> 4;
  const int wm = (wv & 1) * 64, wn = ((wv >> 1) & 1) * 64;
  f4 acc[4][4];

  if (wv >= 4) {
    const int lw = wv - 4;
    i32x4 sx, sw;
    { unsigned long long p = (unsigned long long)P.x; sx[0] = (int)p; sx[1] = (int)(p >> 32); sx[2] = M * K * 2; sx[3] = 0x00020000; }
    { unsigned long long p = (unsigned long long)P.w; sw[0] = (int)p; sw[1] = (int)(p >> 32); sw[2] = K * N * 2; sw[3] = 0x00020000; }
    const unsigned vx = (lane >> 3) * (K * 2) + (((lane & 7) ^ ((4 * lw + (lane >> 4)) & 7)) * 16);
    const unsigned vw = (lane >> 4) * (N * 2) + (((lane & 15) ^ (((lane >> 4) << 2) | lw)) * 16);
    const unsigned lds0 = (unsigned)(size_t)smem + lw * 1024;
    const unsigned sxo = (unsigned)((m0 + lw * 8) * (K * 2));
    const unsigned swo = (unsigned)((lw * 4) * (N * 2) + n0 * 2);
    const int kfull = kh ? (K / 2 + BAND_HALF) : 0;
    auto issue = [&](int st, unsigned buf) {
      if (st < S1) {
        const unsigned kk = kfull + st * 64;
        const unsigned a = sxo + kk * 2, bb = swo + kk * (N * 2);
#pragma unroll
        for (int i = 0; i < 4; ++i) {
          dma16(vx, sx, a + i * (32u * K * 2), lds0 + buf + i * 4096);
          dma16(vw, sw, bb + i * (16u * N * 2), lds0 + buf + 16384 + i * 4096);
        }
      } else if (st < S) {
        const unsigned kk = (K / 2 - BAND_HALF) + (st - S1) * 64;
        const unsigned a = sxo + kk * 2 + kh * (32u * K * 2), bb = swo + kk * (N * 2);
#pragma unroll
        for (int i = 0; i < 2; ++i) {
          dma16(vx, sx, a + i * (64u * K * 2), lds0 + buf + kh * 4096 + i * 8192);
          dma16(vw, sw, bb + (2 * i) * (16u * N * 2), lds0 + buf + 16384 + (2 * i) * 4096);
          dma16(vw, sw, bb + (2 * i + 1) * (16u * N * 2), lds0 + buf + 16384 + (2 * i + 1) * 4096);
        }
      }
    };
#pragma unroll
    for (int st = 0; st < NS - 1; ++st) issue(st, st * STAGE_B);
    unsigned seq_raw = __hip_atomic_load(cnt, __ATOMIC_RELAXED, __HIP_MEMORY_SCOPE_AGENT);
    peer_ws = P.ws[lane & (NR - 1)];
    out_dst = nullptr; ws_dst = nullptr;
    unsigned wbuf = (NS - 1) * STAGE_B;
    for (int s = 0; s < S; ++s) {
      if (s + NS - 2 < S1) waitvm<8 * (NS - 2)>();
      else {
        int c = 0;
#pragma unroll
        for (int j = 1; j <= NS - 2; ++j) c += (s + j < S1) ? 8 : ((s + j < S) ? 6 : 0);
        if (c >= 14) waitvm<14>();
        else if (c >= 12) waitvm<12>();
        else if (c >= 6) waitvm<6>();
        else waitvm<0>();
      }
      wg_barrier();
      issue(s + NS - 1, wbuf);
      wbuf = (wbuf + STAGE_B == NS * STAGE_B) ? 0 : wbuf + STAGE_B;
    }
    asm volatile("" : "+v"(seq_raw), "+v"(peer_ws));
    seq = (unsigned)__builtin_amdgcn_readfirstlane(seq_raw) + 1u;
    STAMP(1);
  } else {
    unsigned seq_raw = __hip_atomic_load(cnt, __ATOMIC_RELAXED, __HIP_MEMORY_SCOPE_AGENT);
    peer_ws = nullptr;
    out_dst = P.out[dst];
    ws_dst = P.ws[dst];
    asm volatile("" : "+s"(out_dst), "+s"(ws_dst));
    unsigned xk[2], xs[2];
#pragma unroll
    for (int ks = 0; ks < 2; ++ks) {
      const unsigned o = (wm + fr) * 128 + (((4 * ks + fq) ^ (fr >> 1)) * 16);
      xk[ks] = o + kh * 4096;
      xs[ks] = o + (kh ^ 1) * 4096;
    }
    unsigned woff[4][2];
    {
      const int q = (lane & 15) >> 2, p = lane & 3;
#pragma unroll
      for (int cc = 0; cc < 4; ++cc)
#pragma unroll
        for (int t = 0; t < 2; ++t) {
          const int row = 8 * fq + 4 * t + q;
          const int ch = (wn / 8) + 2 * cc + (p >> 1);
          woff[cc][t] = 16384 + 256 * row + 16 * (ch ^ (((row & 3) << 2) | ((row >> 2) & 3))) + 8 * (p & 1);
        }
    }
#pragma unroll
    for (int i = 0; i < 4; ++i)
#pragma unroll
      for (int j = 0; j < 4; ++j) acc[i][j] = f4{0, 0, 0, 0};
    v8s xf[2][2][4], wf[2][2][4];
#pragma unroll
    for (int ks = 0; ks < 2; ++ks)
#pragma unroll
      for (int j = 0; j < 4; ++j) { xf[1][ks][j] = v8s{0, 0, 0, 0, 0, 0, 0, 0}; wf[1][ks][j] = v8s{0, 0, 0, 0, 0, 0, 0, 0}; }

    unsigned rbuf = 0;
    auto rd = [&](int set, int mt) {
#pragma unroll
      for (int ks = 0; ks < 2; ++ks) {
#pragma unroll
        for (int j = 0; j < 4; ++j) {
          v4s a = __builtin_amdgcn_ds_read_tr16_b64_v4i16((LAS v4s*)(L + rbuf + woff[j][0] + ks * 8192));
          v4s c = __builtin_amdgcn_ds_read_tr16_b64_v4i16((LAS v4s*)(L + rbuf + woff[j][1] + ks * 8192));
          wf[set][ks][j] = __builtin_shufflevector(a, c, 0, 1, 2, 3, 4, 5, 6, 7);
        }
#pragma unroll
        for (int i = 0; i < 4; ++i)
          if (i < mt) xf[set][ks][i] = *(LAS v8s*)(L + rbuf + (i < 2 ? xk[ks] : xs[ks]) + (i & 1) * 2048);
      }
    };
    auto mm = [&](int set, int i0, int i1) {
#pragma unroll
      for (int ks = 0; ks < 2; ++ks)
#pragma unroll
        for (int i = 0; i < 4; ++i)
          if (i >= i0 && i < i1) {
#pragma unroll
            for (int j = 0; j < 4; ++j)
              acc[i][j] = __builtin_amdgcn_mfma_f32_16x16x32_bf16(__builtin_bit_cast(v8bf, wf[set][ks][j]), __builtin_bit_cast(v8bf, xf[set][ks][i]), acc[i][j], 0, 0, 0);
          }
    };
    auto top = [&]() {
      __builtin_amdgcn_s_waitcnt(0xC07F);
      asm volatile("" ::: "memory");
      __builtin_amdgcn_sched_barrier(0);
      __builtin_amdgcn_s_barrier();
      asm volatile("" ::: "memory");
      __builtin_amdgcn_sched_barrier(0);
    };
    auto bottom = [&]() {
      __builtin_amdgcn_sched_barrier(0);
      rbuf = (rbuf + STAGE_B == NS * STAGE_B) ? 0 : rbuf + STAGE_B;
    };
    auto interleave = [&](int n) {
      for (int i = 0; i < n; ++i) {
        __builtin_amdgcn_sched_group_barrier(0x100, 1, 0);
        __builtin_amdgcn_sched_group_barrier(0x8, 1, 0);
      }
    };
    for (int s = 0; s < S1; s += 2) {
      top(); rd(0, 4); mm(1, 0, 4); interleave(24); bottom();
      top(); rd(1, 4); mm(0, 0, 4); interleave(24); bottom();
    }
    STAMP(1);
    const __amdgpu_buffer_rsrc_t rx = make_rsrc(ws + WS_XBUF, 8u << 20);
    top(); rd(0, 2); mm(1, 2, 4); interleave(16);
    __builtin_amdgcn_sched_barrier(0);
    {
      const unsigned so = (unsigned)(b * 32768 + wv * 8192);
#pragma unroll
      for (int i = 0; i < 2; ++i)
#pragma unroll
        for (int j = 0; j < 4; ++j)
          __builtin_amdgcn_raw_buffer_store_b128(__builtin_bit_cast(u32x4, acc[2 + i][j]), rx, lane * 16 + (i * 4 + j) * 1024, so, AUX_AGENT);
    }
    mm(1, 0, 2); bottom();
    top(); rd(1, 2); mm(0, 0, 2); interleave(16); bottom();
    waitvm<0>();
    asm volatile("" : "+v"(seq_raw));
    seq = (unsigned)__builtin_amdgcn_readfirstlane(seq_raw) + 1u;
    if (lane == 0) __hip_atomic_store((unsigned*)(ws + WS_XFLAG + (size_t)(b * 4 + wv) * 64), seq, __ATOMIC_RELAXED, __HIP_MEMORY_SCOPE_AGENT);
    STAMP(2);
    for (int s = S1 + 2; s < S - 2; s += 2) {
      top(); rd(0, 2); mm(1, 0, 2); interleave(16); bottom();
      top(); rd(1, 2); mm(0, 0, 2); interleave(16); bottom();
    }
    top(); rd(0, 2); mm(1, 0, 2); interleave(16); bottom();
    top(); rd(1, 2); mm(0, 0, 2); interleave(16); bottom();
    STAMP(3);
    unsigned mx = 0;
    {
      const unsigned* flag = (const unsigned*)(ws + WS_XFLAG + (size_t)((b ^ 1) * 4 + wv) * 64);
      const unsigned long long t0 = __builtin_amdgcn_s_memrealtime();
      while ((unsigned)__builtin_amdgcn_readfirstlane(__hip_atomic_load(flag, __ATOMIC_RELAXED, __HIP_MEMORY_SCOPE_AGENT)) < seq) {
        if (__hip_atomic_load(tmo, __ATOMIC_RELAXED, __HIP_MEMORY_SCOPE_AGENT) == seq || __builtin_amdgcn_s_memrealtime() - t0 > DEADLINE) {
          __hip_atomic_store(tmo, seq, __ATOMIC_RELAXED, __HIP_MEMORY_SCOPE_AGENT);
          mx = 0x7fc00000u;
          break;
        }
        __builtin_amdgcn_s_sleep(1);
      }
      asm volatile("" ::: "memory");
      const unsigned so = (unsigned)((b ^ 1) * 32768 + wv * 8192);
#pragma unroll
      for (int i = 0; i < 2; ++i)
#pragma unroll
        for (int j = 0; j < 4; ++j)
          acc[2 + i][j] = __builtin_bit_cast(f4, __builtin_amdgcn_raw_buffer_load_b128(rx, lane * 16 + (i * 4 + j) * 1024, so, AUX_AGENT));
    }
    __builtin_amdgcn_sched_barrier(0);
    mm(1, 0, 2);
    __builtin_amdgcn_sched_barrier(0);
#pragma unroll
    for (int i = 0; i < 2; ++i)
#pragma unroll
      for (int j = 0; j < 4; ++j) acc[i][j] += acc[2 + i][j];
#pragma unroll
    for (int i = 0; i < 2; ++i)
#pragma unroll
      for (int j = 0; j < 4; ++j)
#pragma unroll
        for (int e = 0; e < 4; ++e) { const float t = acc[i][j][e]; mx = max(mx, __float_as_uint(t) & 0x7fffffffu); }
    mx = wave_max_u32(mx);
    if (lane == 0) Lm[wv] = mx;
    __builtin_amdgcn_s_waitcnt(0xC07F);
    STAMP(4);
  }
  wg_barrier();
  const size_t par = (size_t)(seq & 1u) * (NR * NWG * 8);
  if (wv == 4) {
    const unsigned mx = max(max(Lm[0], Lm[1]), max(Lm[2], Lm[3]));
    if (lane < NR)
      __hip_atomic_store((GAS unsigned long long*)(peer_ws + WS_AMAX + par + (size_t)(rank * NWG + b) * 8),
                         ((unsigned long long)seq << 32) | mx, __ATOMIC_RELAXED, __HIP_MEMORY_SCOPE_SYSTEM);
  }
  {
    const __amdgpu_buffer_rsrc_t ra = make_rsrc(ws + WS_AMAX, 2 * NR * NWG * 8);
    const unsigned so = (unsigned)par + wv * 2048;
    unsigned mx;
    const unsigned long long t0 = __builtin_amdgcn_s_memrealtime();
    for (;;) {
      const u32x4 g0 = __builtin_amdgcn_raw_buffer_load_b128(ra, lane * 16, so, AUX_SYS);
      const u32x4 g1 = __builtin_amdgcn_raw_buffer_load_b128(ra, lane * 16 + 1024, so, AUX_SYS);
      const bool ok = g0[1] == seq && g0[3] == seq && g1[1] == seq && g1[3] == seq;
      mx = max(max(g0[0], g0[2]), max(g1[0], g1[2]));
      if (__all(ok)) break;
      if (__hip_atomic_load(tmo, __ATOMIC_RELAXED, __HIP_MEMORY_SCOPE_AGENT) == seq || __builtin_amdgcn_s_memrealtime() - t0 > DEADLINE) {
        __hip_atomic_store(tmo, seq, __ATOMIC_RELAXED, __HIP_MEMORY_SCOPE_AGENT);
        mx = 0x7fc00000u;
        break;
      }
      __builtin_amdgcn_s_sleep(1);
    }
    mx = wave_max_u32(mx);
    if (lane == 0) Lm[8 + wv] = mx;
    __builtin_amdgcn_s_waitcnt(0xC07F);
  }
  STAMP(5);
  wg_barrier();
  if (wv < 4) {
    unsigned gm = 0;
#pragma unroll
    for (int i = 0; i < 8; ++i) gm = max(gm, Lm[8 + i]);
    const float amax = __builtin_bit_cast(float, gm);
    const float scale = amax / 448.0f;
    const float inv = (gm == 0u) ? 0.0f : 1.0f / scale;
    const __amdgpu_buffer_rsrc_t ro = make_rsrc(out_dst, (unsigned)(NR * M * OUT_LD));
#pragma unroll
    for (int i = 0; i < 2; ++i) {
      unsigned q[4];
#pragma unroll
      for (int j = 0; j < 4; ++j) {
        float v[4];
#pragma unroll
        for (int e = 0; e < 4; ++e) v[e] = __builtin_amdgcn_fmed3f(acc[i][j][e] * inv, 448.0f, -448.0f);
        int pk = __builtin_amdgcn_cvt_pk_fp8_f32(v[0], v[1], 0, false);
        pk = __builtin_amdgcn_cvt_pk_fp8_f32(v[2], v[3], pk, true);
        q[j] = (gm > 0x7f800000u) ? 0x7f7f7f7fu : (unsigned)pk;
      }
      {
        auto r0 = __builtin_amdgcn_permlane32_swap(q[0], q[2], false, false); q[0] = r0[0]; q[2] = r0[1];
        auto r1 = __builtin_amdgcn_permlane32_swap(q[1], q[3], false, false); q[1] = r1[0]; q[3] = r1[1];
        auto r2 = __builtin_amdgcn_permlane16_swap(q[0], q[1], false, false); q[0] = r2[0]; q[1] = r2[1];
        auto r3 = __builtin_amdgcn_permlane16_swap(q[2], q[3], false, false); q[2] = r3[0]; q[3] = r3[1];
      }
      const int row = rank * M + m0 + wm + kh * 32 + i * 16 + fr;
      const unsigned vo = (unsigned)(row * OUT_LD + (n0 % OUT_LD) + wn + fq * 16);
      __builtin_amdgcn_raw_buffer_store_b128(u32x4{q[0], q[1], q[2], q[3]}, ro, vo, 0, AUX_SYS);
    }
    STAMP(6);
    waitvm<0>();
    STAMP(7);
    if (dst != rank && lane == 0) {
      const int slot = ((idx & 3) * 4 + ((idx >> 2) & 3)) * 2 + kh;
      __hip_atomic_store((GAS unsigned*)(ws_dst + WS_DONE) + (rank * 32 + slot) * 4 + wv, seq, __ATOMIC_RELAXED, __HIP_MEMORY_SCOPE_SYSTEM);
    }
  } else {
    if (wv == 4 && lane == 0)
      __hip_atomic_store(cnt, seq, __ATOMIC_RELAXED, __HIP_MEMORY_SCOPE_AGENT);
    if (b == 0) {
      const __amdgpu_buffer_rsrc_t rdn = make_rsrc(ws + WS_DONE, NR * 32 * 4 * 4);
      const unsigned vo = (unsigned)(((wv - 4) * 64 + lane) * 16);
      const bool mine = ((wv - 4) * 64 + lane) / 32 == rank;
      const unsigned long long t0 = __builtin_amdgcn_s_memrealtime();
      for (;;) {
        const u32x4 g = __builtin_amdgcn_raw_buffer_load_b128(rdn, vo, 0, AUX_SYS);
        const bool ok = mine || (g[0] >= seq && g[1] >= seq && g[2] >= seq && g[3] >= seq);
        if (__all(ok)) break;
        if (__hip_atomic_load(tmo, __ATOMIC_RELAXED, __HIP_MEMORY_SCOPE_AGENT) == seq || __builtin_amdgcn_s_memrealtime() - t0 > DEADLINE) {
          __hip_atomic_store(tmo, seq, __ATOMIC_RELAXED, __HIP_MEMORY_SCOPE_AGENT);
          if (lane == 0) P.out[rank][(size_t)(rank * M) * OUT_LD] = 0x7f;
          break;
        }
        __builtin_amdgcn_s_sleep(1);
      }
      STAMP(6);
    }
  }
}

}

extern "C" void kernel_launch(int rank, int nranks, ncclComm_t comm,
                              hipStream_t stream,
                              void* const* d_in, const int* in_sizes, int n_in,
                              void* d_out, int out_size,
                              const int* in_dtypes, int out_dtype,
                              void* const* d_out_all,
                              void* const* d_ws_all, size_t ws_bytes) {
  (void)comm; (void)in_dtypes; (void)out_dtype; (void)d_out;
  if (nranks != NR || n_in < 2 || in_sizes[0] != M * K || in_sizes[1] != K * N || out_size != NR * M * OUT_LD || ws_bytes < WS_NEED ||
      (unsigned)rank >= (unsigned)NR) {
    fprintf(stderr, "kernel_launch: unexpected problem shape\n");
    return;
  }
  HIP_CHECK(hipFuncSetAttribute((const void*)gemm_a2a_fp8, hipFuncAttributeMaxDynamicSharedMemorySize, LDS_BYTES));
  Params P;
  P.x = (const unsigned short*)d_in[0];
  P.w = (const unsigned short*)d_in[1];
  for (int p = 0; p < NR; ++p) {
    P.out[p] = (unsigned char*)d_out_all[p];
    P.ws[p] = (unsigned char*)d_ws_all[p];
  }
  P.ws_self = (unsigned char*)d_ws_all[rank];
  P.rank = rank;
  P.pad = 0;
  gemm_a2a_fp8<<<NWG, 512, LDS_BYTES, stream>>>(P);
  HIP_CHECK(hipGetLastError());
}
```

```cpp
#include <hip/hip_runtime.h>
#include <rccl/rccl.h>
#include <cstdio>

#define HIP_CHECK(expr) do { hipError_t _e = (expr); if (_e != hipSuccess) { \
    fprintf(stderr, "HIP error %s at %s:%d\n", hipGetErrorString(_e), __FILE__, __LINE__); \
    return; } } while (0)

#define LOOP_PX 1088
#define LOOP_PW 1040
#define LOOP_T1 11
#define LOOP_NSLOT 9
#define ACLOB "a0", "a1", "a2", "a3", "a4", "a5", "a6", "a7", "a8", "a9", "a10", "a11", "a12", "a13", "a14", "a15", "a16", "a17", "a18", "a19", "a20", "a21", "a22", "a23", "a24", "a25", "a26", "a27", "a28", "a29", "a30", "a31", "a32", "a33", "a34", "a35", "a36", "a37", "a38", "a39", "a40", "a41", "a42", "a43", "a44", "a45", "a46", "a47", "a48", "a49", "a50", "a51", "a52", "a53", "a54", "a55", "a56", "a57", "a58", "a59", "a60", "a61", "a62", "a63"

namespace {

typedef int i32x4 __attribute__((ext_vector_type(4)));
typedef int i32x8 __attribute__((ext_vector_type(8)));
typedef unsigned u32x4 __attribute__((ext_vector_type(4)));
typedef unsigned u32x2 __attribute__((ext_vector_type(2)));
typedef float f4 __attribute__((ext_vector_type(4)));
typedef float f16v __attribute__((ext_vector_type(16)));
typedef __bf16 bf8 __attribute__((ext_vector_type(8)));
typedef float f2 __attribute__((ext_vector_type(2)));
typedef __bf16 bf2 __attribute__((ext_vector_type(2)));
#define INL __attribute__((always_inline))
#define LAS __attribute__((address_space(3)))
#define GAS __attribute__((address_space(1)))

constexpr int NR = 8;
constexpr int M = 512, N = 4096, K = 4096;
constexpr int NWG = 256;
constexpr int OUT_LD = N / NR;

constexpr int NTHR = 512;
constexpr int PX = LOOP_PX, PW = LOOP_PW;
constexpr int SLOT = 16 * PX;
constexpr int NT = 16;
constexpr int LDS_MISC = LOOP_NSLOT * SLOT;
constexpr int LDS_BYTES = LDS_MISC + 256;

__host__ __device__ constexpr int kquarter(int kh, int mt, int nt) { return kh * 2 + ((mt ^ (nt >> 1)) & 1); }
__host__ __device__ constexpr int class_of(int mt, int nt) { return ((mt >> 1) + nt) & 1; }
#define PERM0 0x456789acdef012b3ull
#define PERM1 0x56789acdef0124b3ull
#define PERM2 0x6789acdef01245b3ull
#define PERM3 0x789acdef012456b3ull
#define T0 0
constexpr unsigned long long PERMS[4] = {PERM0, PERM1, PERM2, PERM3};
constexpr int stage_of(int kq, int c, int t) {
  const int s = t - T0;
  int u = s < 0 ? s : c ? s ^ 1 : s == 0 ? 0 : ((s - 1) ^ 1) + 1;
  if (u >= NT - T0) u = s;
  return (int)(PERMS[kq] >> (60 - 4 * (u + T0))) & 15;
}
constexpr unsigned firsts() { unsigned f = 0; for (int i = 0; i < 8; ++i) f |= (unsigned)stage_of(i >> 1, i & 1, 0) << (4 * i); return f; }
inline void make_order(unsigned char (&ord)[8][16]) {
  for (int i = 0; i < 8; ++i)
    for (int t = 0; t < NT; ++t) ord[i][t] = (unsigned char)(stage_of(i >> 1, i & 1, t) + (i >> 1) * NT);
}

constexpr size_t WS_TMO = 0;
constexpr size_t WS_AGG = 16384;
constexpr size_t WS_AMAX = 32768;
constexpr size_t WS_DONE = 49152;
constexpr size_t WS_CNT = 262144;
constexpr size_t WS_XBUF = 1u << 20;
#define XSH 13
constexpr unsigned XPART = 1024u << XSH;
constexpr size_t WS_XB1 = WS_XBUF + XPART;
constexpr size_t WS_STG = WS_XB1 + XPART;
constexpr size_t WS_STAMP = WS_STG + (8u << 20);
constexpr size_t WS_NEED = WS_STAMP + (1u << 20);
__device__ __forceinline__ unsigned skip3(unsigned byte) { return byte + (byte >= 384u ? 128u : 0u); }
__device__ __forceinline__ unsigned stg(unsigned to, unsigned par, unsigned slot) { return (unsigned)WS_STG + (to << 20) + par * 524288u + slot * 16384u; }

constexpr unsigned long long DEADLINE = 200000000ull;

struct Params {
  i32x4 sx, sw;
  unsigned char* ws_self;
  int rank, pad;
  unsigned char* out[NR + 1];
  unsigned char* ws[NR];
  alignas(64) unsigned char ord[8][16];
};

constexpr unsigned KA_SELF = __builtin_offsetof(Params, ws_self), KA_OUT = __builtin_offsetof(Params, out), KA_WS = __builtin_offsetof(Params, ws);

template <int N_> __device__ __forceinline__ void waitvm() { asm volatile("s_waitcnt vmcnt(%0)" ::"i"(N_) : "memory"); }
__device__ __forceinline__ void wg_barrier() {
  asm volatile("" ::: "memory");
  __builtin_amdgcn_s_barrier();
  asm volatile("" ::: "memory");
}
__device__ __forceinline__ __amdgpu_buffer_rsrc_t make_rsrc(const void* p, unsigned bytes) {
  return __builtin_amdgcn_make_buffer_rsrc(const_cast<void*>(p), 0, bytes, 0x00020000);
}
__device__ __forceinline__ unsigned uni(unsigned v) { return (unsigned)__builtin_amdgcn_readfirstlane((int)v); }
__device__ __forceinline__ unsigned char* uptr(unsigned lo, unsigned hi) { return (unsigned char*)(((unsigned long long)uni(hi) << 32) | uni(lo)); }
__device__ __forceinline__ unsigned wave_max_u32(unsigned v) {
  v = max(v, (unsigned)__builtin_amdgcn_update_dpp(0, (int)v, 0xB1, 0xf, 0xf, true));
  v = max(v, (unsigned)__builtin_amdgcn_update_dpp(0, (int)v, 0x4E, 0xf, 0xf, true));
  v = max(v, (unsigned)__builtin_amdgcn_update_dpp(0, (int)v, 0x141, 0xf, 0xf, true));
  v = max(v, (unsigned)__builtin_amdgcn_update_dpp(0, (int)v, 0x140, 0xf, 0xf, true));
  const unsigned a = (unsigned)__builtin_amdgcn_readlane((int)v, 0), b = (unsigned)__builtin_amdgcn_readlane((int)v, 16);
  const unsigned c = (unsigned)__builtin_amdgcn_readlane((int)v, 32), d = (unsigned)__builtin_amdgcn_readlane((int)v, 48);
  return max(max(a, b), max(c, d));
}
__device__ __forceinline__ unsigned inv_of(unsigned mx) {
  const float am = __builtin_bit_cast(float, mx), m = __builtin_amdgcn_frexp_mantf(am);
  float r = __builtin_amdgcn_rcpf(m);
  r = __builtin_fmaf(__builtin_fmaf(-m, r, 1.0f), r, r);
  return mx == 0u ? 0u : mx >= 0x7f800000u ? 0x7fc00000u : __builtin_bit_cast(unsigned, __builtin_amdgcn_ldexpf(448.0f * r, -__builtin_amdgcn_frexp_expf(am)));
}
__device__ __forceinline__ bool expired(unsigned* tmo, unsigned seq, unsigned long long t0) {
  if ((unsigned)__builtin_amdgcn_readfirstlane(__hip_atomic_load(tmo, __ATOMIC_RELAXED, __HIP_MEMORY_SCOPE_AGENT)) != seq &&
      __builtin_amdgcn_s_memrealtime() - t0 <= DEADLINE) return false;
  __hip_atomic_store(tmo, seq, __ATOMIC_RELAXED, __HIP_MEMORY_SCOPE_AGENT);
  return true;
}
constexpr int AUX_AGENT = 16;
constexpr int AUX_SYS = 17;
constexpr unsigned NAN_BITS = 0x7fc00000u;

#define WBL 30
#define NPUSH 8
#define MARGIN 130
#define STAMP(i) do { } while (0)
#define STAMPS_OUT(ws) do { } while (0)

template <class T, class L, class OK>
__device__ __forceinline__ bool poll(T& v, L ld, OK ok, unsigned* tmo, unsigned seq, unsigned* mine = nullptr) {
  const unsigned long long t0 = __builtin_amdgcn_s_memrealtime();
  for (unsigned it = 1;; ++it) {
    v = ld();
    if (ok(v)) return true;
    if ((it & 63) == 0) {
      asm volatile("buffer_wbl2 sc1" ::: "memory");
      if (mine) asm volatile("global_atomic_add %0, %1, off sc1" ::"v"(mine), "v"(0) : "memory");
      if (expired(tmo, seq, t0)) return false;
    }
  }
}

__global__ __launch_bounds__(NTHR) void gemm_a2a_fp8(Params P) {
  extern __shared__ __attribute__((aligned(16))) char smem[];
  i32x8 sxw;
  int bx = blockIdx.x, tx = threadIdx.x;
  asm volatile("s_load_dwordx8 %0, %3, 0x0" : "=&s"(sxw), "+s"(bx), "+v"(tx) : "s"(__builtin_amdgcn_kernarg_segment_ptr()));
  int t0 = __builtin_amdgcn_readfirstlane(tx);
  asm volatile("s_cmpk_lt_u32 %0, 0x100\n\ts_cbranch_scc0 1f\n\ts_getpc_b64 s[100:101]\n\ts_lshl_b32 s99, %0, 1\n\t"
               "s_load_dword s98, s[100:101], s99 offset:0x80\n\ts_load_dword s98, s[100:101], s99 offset:0x280\n\t"
               "s_load_dword s98, s[100:101], s99 offset:0x480\n\ts_load_dword s98, s[100:101], s99 offset:0x680\n\t"
               "s_load_dword s98, s[100:101], s99 offset:0x880\n\ts_load_dword s98, s[100:101], s99 offset:0xa80\n\t"
               "\n1:"
               : "+s"(t0), "+s"(bx) :: "s97", "s98", "s99", "s100", "s101", "scc");
  const int b = bx, xcd = b & 7, idx = b >> 3;
  const int kh = xcd >> 2, nb = xcd & 3;
  const int mt = idx & 3, nt = idx >> 2;
  constexpr int X1 = 8, X2 = 4;
  const int grp = xcd, gpos = idx;
  const int tile = (nb + nt) & 3, dst = nt;
  const int m0 = mt * 128, n0 = dst * OUT_LD + tile * 128;
  const int slot = b & 31;
  const bool send = slot < NPUSH;
  const int wv = (unsigned)t0 >> 6, lane = tx & 63;
  const int fr = lane & 15, fq = lane >> 4;
  const unsigned l0 = (unsigned)(size_t)smem;
  LAS unsigned* const Lm = (LAS unsigned*)((LAS char*)smem + LDS_MISC);

  if (__builtin_expect(wv >= 4, 1)) {
    const int lw = wv - 4;
    unsigned vx[4], vg[4], vw[4];
    vg[0] = ((m0 ^ 128) + lw * 8 + kh * 32 + (lane >> 3)) * (K * 2) + (((lane & 7) ^ ((lane >> 4) & 3)) * 16);
    vg[1] = vg[0] + 64u * K * 2; vg[2] = vg[0] ^ (32u * K * 2); vg[3] = vg[2] + 64u * K * 2;
#pragma unroll
    for (int i = 0; i < 4; ++i) {
      vx[i] = vg[i] ^ (128u * K * 2);
      vw[i] = (lw * 4 + i * 16 + (lane >> 4)) * (N * 2) + n0 * 2 + (((lane & 15) ^ ((lane >> 4) << 2)) * 16);
    }
    const unsigned ldx = l0 + lw * PX, ldw = l0 + lw * PW;
    const unsigned oi = kquarter(kh, mt, nt) * 2 + class_of(mt, nt);
    unsigned t0_ = (kquarter(kh, mt, nt) * NT + ((firsts() >> (4 * oi)) & 15)) * 128, t1_ = t0_ << 12, t2_ = 0, t3_ = 0, oo = oi * 16;
    i32x4 ow;
    asm volatile("s_waitcnt lgkmcnt(0)\n\ts_load_dwordx4 %1, %4, %5 offset:%6" : "+s"(sxw), "=&s"(ow), "+s"(t0_), "+s"(t1_)
                 : "s"(__builtin_amdgcn_kernarg_segment_ptr()), "s"(oo), "i"(__builtin_offsetof(Params, ord)),
                   "v"(vg[0]), "v"(vg[1]), "v"(vg[2]), "v"(vg[3]), "v"(vw[0]), "v"(vw[1]), "v"(vw[2]), "v"(vw[3]),
                   "v"(vx[0]), "v"(vx[1]), "v"(vx[2]), "v"(vx[3]), "s"(ldx), "s"(ldw));
    const i32x4 sx = sxw.lo, sw = sxw.hi;
    asm volatile(
      "s_add_u32 m0, %[ldx], 17408\n\t"
      "s_nop 0\n\t"
      "buffer_load_dwordx4 %[vg0], %[sx], %[t0] offen lds\n\t"
      "s_add_u32 m0, %[ldx], 26112\n\t"
      "s_nop 0\n\t"
      "buffer_load_dwordx4 %[vg1], %[sx], %[t0] offen lds\n\t"
      "s_add_u32 m0, %[ldx], 21760\n\t"
      "s_nop 0\n\t"
      "buffer_load_dwordx4 %[vg2], %[sx], %[t0] offen lds\n\t"
      "s_add_u32 m0, %[ldx], 30464\n\t"
      "s_nop 0\n\t"
      "buffer_load_dwordx4 %[vg3], %[sx], %[t0] offen lds\n\t"
      "s_add_u32 m0, %[ldw], 0\n\t"
      "s_nop 0\n\t"
      "buffer_load_dwordx4 %[vw0], %[sw], %[t1] offen lds\n\t"
      "s_add_u32 m0, %[ldw], 4160\n\t"
      "s_nop 0\n\t"
      "buffer_load_dwordx4 %[vw1], %[sw], %[t1] offen lds\n\t"
      "s_add_u32 m0, %[ldw], 8320\n\t"
      "s_nop 0\n\t"
      "buffer_load_dwordx4 %[vw2], %[sw], %[t1] offen lds\n\t"
      "s_add_u32 m0, %[ldw], 12480\n\t"
      "s_nop 0\n\t"
      "buffer_load_dwordx4 %[vw3], %[sw], %[t1] offen lds\n\t"
      "s_add_u32 m0, %[ldx], 34816\n\t"
      "s_nop 0\n\t"
      "buffer_load_dwordx4 %[vx0], %[sx], %[t0] offen lds\n\t"
      "s_add_u32 m0, %[ldx], 43520\n\t"
      "s_nop 0\n\t"
      "buffer_load_dwordx4 %[vx1], %[sx], %[t0] offen lds\n\t"
      "s_add_u32 m0, %[ldx], 39168\n\t"
      "s_nop 0\n\t"
      "buffer_load_dwordx4 %[vx2], %[sx], %[t0] offen lds\n\t"
      "s_add_u32 m0, %[ldx], 47872\n\t"
      "s_nop 0\n\t"
      "buffer_load_dwordx4 %[vx3], %[sx], %[t0] offen lds"
      : [t0] "+s"(t0_), [t1] "+s"(t1_), [t2] "+s"(t2_), [t3] "+s"(t3_)
      : [vx0] "v"(vx[0]), [vx1] "v"(vx[1]), [vx2] "v"(vx[2]), [vx3] "v"(vx[3]), [vg0] "v"(vg[0]), [vg1] "v"(vg[1]), [vg2] "v"(vg[2]), [vg3] "v"(vg[3]), [vw0] "v"(vw[0]), [vw1] "v"(vw[1]), [vw2] "v"(vw[2]), [vw3] "v"(vw[3]), [sx] "s"(sx), [sw] "s"(sw), [ldx] "s"(ldx), [ldw] "s"(ldw)
      : "memory", "scc");
    STAMP(9);
    int lw_ = lw;
    asm volatile("s_waitcnt lgkmcnt(0)" : "+s"(ow), "+s"(lw_));
    i32x4 wr;
    unsigned long long ba;
    asm volatile("s_load_dwordx4 %0, %2, %4\n\ts_load_dwordx2 %1, %2, %3" : "=&s"(wr), "=&s"(ba)
                 : "s"(__builtin_amdgcn_kernarg_segment_ptr()), "s"(lw_ == 0 ? KA_SELF : lw_ & 1 ? KA_WS + dst * 8 : KA_OUT + (send ? dst : NR) * 8), "i"(KA_SELF));
    asm volatile(
      "s_bfe_u32 %[t2], %[o0], 0x80008\n\t"
      "s_lshl_b32 %[t2], %[t2], 7\n\t"
      "s_add_u32 m0, %[ldx], 69632\n\t"
      "s_nop 0\n\t"
      "buffer_load_dwordx4 %[vg0], %[sx], %[t2] offen lds\n\t"
      "s_add_u32 m0, %[ldx], 78336\n\t"
      "s_nop 0\n\t"
      "buffer_load_dwordx4 %[vg1], %[sx], %[t2] offen lds\n\t"
      "s_add_u32 m0, %[ldx], 73984\n\t"
      "s_bfe_u32 %[t3], %[o0], 0x80008\n\t"
      "buffer_load_dwordx4 %[vg2], %[sx], %[t2] offen lds\n\t"
      "s_add_u32 m0, %[ldx], 82688\n\t"
      "s_lshl_b32 %[t3], %[t3], 19\n\t"
      "buffer_load_dwordx4 %[vg3], %[sx], %[t2] offen lds\n\t"
      "s_add_u32 m0, %[ldw], 52224\n\t"
      "s_nop 0\n\t"
      "buffer_load_dwordx4 %[vw0], %[sw], %[t3] offen lds\n\t"
      "s_add_u32 m0, %[ldw], 56384\n\t"
      "s_nop 0\n\t"
      "buffer_load_dwordx4 %[vw1], %[sw], %[t3] offen lds\n\t"
      "s_add_u32 m0, %[ldw], 60544\n\t"
      "s_nop 0\n\t"
      "buffer_load_dwordx4 %[vw2], %[sw], %[t3] offen lds\n\t"
      "s_add_u32 m0, %[ldw], 64704\n\t"
      "s_nop 0\n\t"
      "buffer_load_dwordx4 %[vw3], %[sw], %[t3] offen lds\n\t"
      "s_add_u32 m0, %[ldx], 87040\n\t"
      "s_nop 0\n\t"
      "s_waitcnt vmcnt(12)\n\t"
      "s_barrier\n\t"
      "buffer_load_dwordx4 %[vx0], %[sx], %[t2] offen lds\n\t"
      "s_add_u32 m0, %[ldx], 95744\n\t"
      "s_nop 0\n\t"
      "buffer_load_dwordx4 %[vx1], %[sx], %[t2] offen lds\n\t"
      "s_add_u32 m0, %[ldx], 91392\n\t"
      "s_bfe_u32 %[t0], %[o0], 0x80010\n\t"
      "buffer_load_dwordx4 %[vx2], %[sx], %[t2] offen lds\n\t"
      "s_add_u32 m0, %[ldx], 100096\n\t"
      "s_lshl_b32 %[t0], %[t0], 19\n\t"
      "buffer_load_dwordx4 %[vx3], %[sx], %[t2] offen lds\n\t"
      "s_add_u32 m0, %[ldw], 104448\n\t"
      "s_nop 0\n\t"
      "buffer_load_dwordx4 %[vw0], %[sw], %[t0] offen lds\n\t"
      "s_add_u32 m0, %[ldw], 108608\n\t"
      "s_nop 0\n\t"
      "buffer_load_dwordx4 %[vw1], %[sw], %[t0] offen lds\n\t"
      "s_add_u32 m0, %[ldw], 112768\n\t"
      "s_bfe_u32 %[t1], %[o0], 0x80010\n\t"
      "buffer_load_dwordx4 %[vw2], %[sw], %[t0] offen lds\n\t"
      "s_add_u32 m0, %[ldw], 116928\n\t"
      "s_lshl_b32 %[t1], %[t1], 7\n\t"
      "buffer_load_dwordx4 %[vw3], %[sw], %[t0] offen lds\n\t"
      "s_add_u32 m0, %[ldx], 121856\n\t"
      "s_nop 0\n\t"
      "s_waitcnt vmcnt(16)\n\t"
      "s_barrier\n\t"
      "buffer_load_dwordx4 %[vg0], %[sx], %[t1] offen lds\n\t"
      "s_add_u32 m0, %[ldx], 130560\n\t"
      "s_nop 0\n\t"
      "buffer_load_dwordx4 %[vg1], %[sx], %[t1] offen lds\n\t"
      "s_add_u32 m0, %[ldx], 126208\n\t"
      "s_nop 0\n\t"
      "buffer_load_dwordx4 %[vg2], %[sx], %[t1] offen lds\n\t"
      "s_add_u32 m0, %[ldx], 134912\n\t"
      "s_nop 0\n\t"
      "buffer_load_dwordx4 %[vg3], %[sx], %[t1] offen lds\n\t"
      "s_add_u32 m0, %[ldx], 139264\n\t"
      "s_nop 0\n\t"
      "buffer_load_dwordx4 %[vx0], %[sx], %[t1] offen lds\n\t"
      "s_add_u32 m0, %[ldx], 147968\n\t"
      "s_nop 0\n\t"
      "buffer_load_dwordx4 %[vx1], %[sx], %[t1] offen lds\n\t"
      "s_add_u32 m0, %[ldx], 143616\n\t"
      "s_bfe_u32 %[t2], %[o0], 0x80018\n\t"
      "buffer_load_dwordx4 %[vx2], %[sx], %[t1] offen lds\n\t"
      "s_add_u32 m0, %[ldx], 152320\n\t"
      "s_lshl_b32 %[t2], %[t2], 7\n\t"
      "buffer_load_dwordx4 %[vx3], %[sx], %[t1] offen lds\n\t"
      "s_add_u32 m0, %[ldx], 17408\n\t"
      "s_nop 0\n\t"
      "s_waitcnt vmcnt(16)\n\t"
      "s_barrier\n\t"
      "buffer_load_dwordx4 %[vg0], %[sx], %[t2] offen lds\n\t"
      "s_add_u32 m0, %[ldx], 26112\n\t"
      "s_nop 0\n\t"
      "buffer_load_dwordx4 %[vg1], %[sx], %[t2] offen lds\n\t"
      "s_add_u32 m0, %[ldx], 21760\n\t"
      "s_bfe_u32 %[t3], %[o0], 0x80018\n\t"
      "buffer_load_dwordx4 %[vg2], %[sx], %[t2] offen lds\n\t"
      "s_add_u32 m0, %[ldx], 30464\n\t"
      "s_lshl_b32 %[t3], %[t3], 19\n\t"
      "buffer_load_dwordx4 %[vg3], %[sx], %[t2] offen lds\n\t"
      "s_add_u32 m0, %[ldw], 0\n\t"
      "s_nop 0\n\t"
      "buffer_load_dwordx4 %[vw0], %[sw], %[t3] offen lds\n\t"
      "s_add_u32 m0, %[ldw], 4160\n\t"
      "s_nop 0\n\t"
      "buffer_load_dwordx4 %[vw1], %[sw], %[t3] offen lds\n\t"
      "s_add_u32 m0, %[ldw], 8320\n\t"
      "s_nop 0\n\t"
      "buffer_load_dwordx4 %[vw2], %[sw], %[t3] offen lds\n\t"
      "s_add_u32 m0, %[ldw], 12480\n\t"
      "s_nop 0\n\t"
      "buffer_load_dwordx4 %[vw3], %[sw], %[t3] offen lds\n\t"
      "s_add_u32 m0, %[ldx], 34816\n\t"
      "s_nop 0\n\t"
      "buffer_load_dwordx4 %[vx0], %[sx], %[t2] offen lds\n\t"
      "s_add_u32 m0, %[ldx], 43520\n\t"
      "s_nop 0\n\t"
      "buffer_load_dwordx4 %[vx1], %[sx], %[t2] offen lds\n\t"
      "s_add_u32 m0, %[ldx], 39168\n\t"
      "s_bfe_u32 %[t0], %[o1], 0x80000\n\t"
      "buffer_load_dwordx4 %[vx2], %[sx], %[t2] offen lds\n\t"
      "s_add_u32 m0, %[ldx], 47872\n\t"
      "s_lshl_b32 %[t0], %[t0], 19\n\t"
      "buffer_load_dwordx4 %[vx3], %[sx], %[t2] offen lds\n\t"
      "s_add_u32 m0, %[ldw], 52224\n\t"
      "s_nop 0\n\t"
      "s_waitcnt vmcnt(24)\n\t"
      "s_barrier\n\t"
      "buffer_load_dwordx4 %[vw0], %[sw], %[t0] offen lds\n\t"
      "s_add_u32 m0, %[ldw], 56384\n\t"
      "s_nop 0\n\t"
      "buffer_load_dwordx4 %[vw1], %[sw], %[t0] offen lds\n\t"
      "s_add_u32 m0, %[ldw], 60544\n\t"
      "s_bfe_u32 %[t1], %[o1], 0x80000\n\t"
      "buffer_load_dwordx4 %[vw2], %[sw], %[t0] offen lds\n\t"
      "s_add_u32 m0, %[ldw], 64704\n\t"
      "s_lshl_b32 %[t1], %[t1], 7\n\t"
      "buffer_load_dwordx4 %[vw3], %[sw], %[t0] offen lds\n\t"
      "s_add_u32 m0, %[ldx], 69632\n\t"
      "s_nop 0\n\t"
      "buffer_load_dwordx4 %[vg0], %[sx], %[t1] offen lds\n\t"
      "s_add_u32 m0, %[ldx], 78336\n\t"
      "s_nop 0\n\t"
      "buffer_load_dwordx4 %[vg1], %[sx], %[t1] offen lds\n\t"
      "s_add_u32 m0, %[ldx], 73984\n\t"
      "s_nop 0\n\t"
      "buffer_load_dwordx4 %[vg2], %[sx], %[t1] offen lds\n\t"
      "s_add_u32 m0, %[ldx], 82688\n\t"
      "s_nop 0\n\t"
      "buffer_load_dwordx4 %[vg3], %[sx], %[t1] offen lds\n\t"
      "s_add_u32 m0, %[ldx], 87040\n\t"
      "s_nop 0\n\t"
      "s_waitcnt vmcnt(24)\n\t"
      "s_barrier\n\t"
      "buffer_load_dwordx4 %[vx0], %[sx], %[t1] offen lds\n\t"
      "s_add_u32 m0, %[ldx], 95744\n\t"
      "s_nop 0\n\t"
      "buffer_load_dwordx4 %[vx1], %[sx], %[t1] offen lds\n\t"
      "s_add_u32 m0, %[ldx], 91392\n\t"
      "s_bfe_u32 %[t2], %[o1], 0x80008\n\t"
      "buffer_load_dwordx4 %[vx2], %[sx], %[t1] offen lds\n\t"
      "s_add_u32 m0, %[ldx], 100096\n\t"
      "s_lshl_b32 %[t2], %[t2], 7\n\t"
      "buffer_load_dwordx4 %[vx3], %[sx], %[t1] offen lds\n\t"
      "s_add_u32 m0, %[ldx], 121856\n\t"
      "s_nop 0\n\t"
      "s_waitcnt vmcnt(24)\n\t"
      "s_barrier\n\t"
      "buffer_load_dwordx4 %[vg0], %[sx], %[t2] offen lds\n\t"
      "s_add_u32 m0, %[ldx], 130560\n\t"
      "s_nop 0\n\t"
      "buffer_load_dwordx4 %[vg1], %[sx], %[t2] offen lds\n\t"
      "s_add_u32 m0, %[ldx], 126208\n\t"
      "s_bfe_u32 %[t3], %[o1], 0x80008\n\t"
      "buffer_load_dwordx4 %[vg2], %[sx], %[t2] offen lds\n\t"
      "s_add_u32 m0, %[ldx], 134912\n\t"
      "s_lshl_b32 %[t3], %[t3], 19\n\t"
      "buffer_load_dwordx4 %[vg3], %[sx], %[t2] offen lds\n\t"
      "s_add_u32 m0, %[ldw], 104448\n\t"
      "s_nop 0\n\t"
      "buffer_load_dwordx4 %[vw0], %[sw], %[t3] offen lds\n\t"
      "s_add_u32 m0, %[ldw], 108608\n\t"
      "s_nop 0\n\t"
      "buffer_load_dwordx4 %[vw1], %[sw], %[t3] offen lds\n\t"
      "s_add_u32 m0, %[ldw], 112768\n\t"
      "s_nop 0\n\t"
      "buffer_load_dwordx4 %[vw2], %[sw], %[t3] offen lds\n\t"
      "s_add_u32 m0, %[ldw], 116928\n\t"
      "s_nop 0\n\t"
      "buffer_load_dwordx4 %[vw3], %[sw], %[t3] offen lds"
      : [t0] "+s"(t0_), [t1] "+s"(t1_), [t2] "+s"(t2_), [t3] "+s"(t3_)
      : [vx0] "v"(vx[0]), [vx1] "v"(vx[1]), [vx2] "v"(vx[2]), [vx3] "v"(vx[3]), [vg0] "v"(vg[0]), [vg1] "v"(vg[1]), [vg2] "v"(vg[2]), [vg3] "v"(vg[3]), [vw0] "v"(vw[0]), [vw1] "v"(vw[1]), [vw2] "v"(vw[2]), [vw3] "v"(vw[3]), [sx] "s"(sx), [sw] "s"(sw), [ldx] "s"(ldx), [ldw] "s"(ldw), [o0] "s"(ow[0]), [o1] "s"(ow[1]), [o2] "s"(ow[2]), [o3] "s"(ow[3])
      : "memory", "scc");
    asm volatile("s_waitcnt lgkmcnt(0)" : "+s"(wr), "+s"(ba), "+s"(lw_));
    unsigned char* const ws = (unsigned char*)(((unsigned long long)(unsigned)wr[1] << 32) | (unsigned)wr[0]);
    const int rank = wr[2];
    unsigned* const cnt = (unsigned*)(ws + WS_CNT + (size_t)b * 1024);
    unsigned cv;
    {
      const unsigned oc = (unsigned)WS_CNT + b * 1024;
      const unsigned oa = lw_ == 2 ? (unsigned)((send ? rank : dst) * M + m0) * OUT_LD : lw_ && send ? oc : stg(lw_ ? rank : dst, lw_ < 2, slot) + 1536;
      const unsigned ob = lw_ ? (unsigned)WS_XBUF + (lw_ & 1) * XPART + (b << (XSH + 2)) : stg(dst, 0, slot) + 1536;
      asm volatile("s_load_dword %0, %1, %2\n\ts_load_dword s100, %3, %4\n\ts_load_dword s101, %1, %5"
                   : "=&s"(cv) : "s"(ws), "s"(oc), "s"(ba), "s"(oa), "s"(ob) : "s100", "s101", "memory");
    }
    asm volatile(
      "s_add_u32 m0, %[ldx], 139264\n\t"
      "s_nop 0\n\t"
      "s_waitcnt vmcnt(24)\n\t"
      "s_barrier\n\t"
      "buffer_load_dwordx4 %[vx0], %[sx], %[t2] offen lds\n\t"
      "s_add_u32 m0, %[ldx], 147968\n\t"
      "s_nop 0\n\t"
      "buffer_load_dwordx4 %[vx1], %[sx], %[t2] offen lds\n\t"
      "s_add_u32 m0, %[ldx], 143616\n\t"
      "s_bfe_u32 %[t0], %[o1], 0x80010\n\t"
      "buffer_load_dwordx4 %[vx2], %[sx], %[t2] offen lds\n\t"
      "s_add_u32 m0, %[ldx], 152320\n\t"
      "s_lshl_b32 %[t0], %[t0], 19\n\t"
      "buffer_load_dwordx4 %[vx3], %[sx], %[t2] offen lds\n\t"
      "s_add_u32 m0, %[ldw], 0\n\t"
      "s_nop 0\n\t"
      "s_waitcnt vmcnt(24)\n\t"
      "s_barrier\n\t"
      "buffer_load_dwordx4 %[vw0], %[sw], %[t0] offen lds\n\t"
      "s_add_u32 m0, %[ldw], 4160\n\t"
      "s_nop 0\n\t"
      "buffer_load_dwordx4 %[vw1], %[sw], %[t0] offen lds\n\t"
      "s_add_u32 m0, %[ldw], 8320\n\t"
      "s_bfe_u32 %[t1], %[o1], 0x80010\n\t"
      "buffer_load_dwordx4 %[vw2], %[sw], %[t0] offen lds\n\t"
      "s_add_u32 m0, %[ldw], 12480\n\t"
      "s_lshl_b32 %[t1], %[t1], 7\n\t"
      "buffer_load_dwordx4 %[vw3], %[sw], %[t0] offen lds\n\t"
      "s_add_u32 m0, %[ldx], 17408\n\t"
      "s_nop 0\n\t"
      "buffer_load_dwordx4 %[vg0], %[sx], %[t1] offen lds\n\t"
      "s_add_u32 m0, %[ldx], 26112\n\t"
      "s_nop 0\n\t"
      "buffer_load_dwordx4 %[vg1], %[sx], %[t1] offen lds\n\t"
      "s_add_u32 m0, %[ldx], 21760\n\t"
      "s_nop 0\n\t"
      "buffer_load_dwordx4 %[vg2], %[sx], %[t1] offen lds\n\t"
      "s_add_u32 m0, %[ldx], 30464\n\t"
      "s_nop 0\n\t"
      "buffer_load_dwordx4 %[vg3], %[sx], %[t1] offen lds\n\t"
      "s_add_u32 m0, %[ldx], 34816\n\t"
      "s_nop 0\n\t"
      "s_waitcnt vmcnt(24)\n\t"
      "s_barrier\n\t"
      "buffer_load_dwordx4 %[vx0], %[sx], %[t1] offen lds\n\t"
      "s_add_u32 m0, %[ldx], 43520\n\t"
      "s_nop 0\n\t"
      "buffer_load_dwordx4 %[vx1], %[sx], %[t1] offen lds\n\t"
      "s_add_u32 m0, %[ldx], 39168\n\t"
      "s_bfe_u32 %[t2], %[o1], 0x80018\n\t"
      "buffer_load_dwordx4 %[vx2], %[sx], %[t1] offen lds\n\t"
      "s_add_u32 m0, %[ldx], 47872\n\t"
      "s_lshl_b32 %[t2], %[t2], 7\n\t"
      "buffer_load_dwordx4 %[vx3], %[sx], %[t1] offen lds\n\t"
      "s_add_u32 m0, %[ldx], 69632\n\t"
      "s_nop 0\n\t"
      "s_waitcnt vmcnt(24)\n\t"
      "s_barrier\n\t"
      "buffer_load_dwordx4 %[vg0], %[sx], %[t2] offen lds\n\t"
      "s_add_u32 m0, %[ldx], 78336\n\t"
      "s_nop 0\n\t"
      "buffer_load_dwordx4 %[vg1], %[sx], %[t2] offen lds\n\t"
      "s_add_u32 m0, %[ldx], 73984\n\t"
      "s_bfe_u32 %[t3], %[o1], 0x80018\n\t"
      "buffer_load_dwordx4 %[vg2], %[sx], %[t2] offen lds\n\t"
      "s_add_u32 m0, %[ldx], 82688\n\t"
      "s_lshl_b32 %[t3], %[t3], 19\n\t"
      "buffer_load_dwordx4 %[vg3], %[sx], %[t2] offen lds\n\t"
      "s_add_u32 m0, %[ldw], 52224\n\t"
      "s_nop 0\n\t"
      "buffer_load_dwordx4 %[vw0], %[sw], %[t3] offen lds\n\t"
      "s_add_u32 m0, %[ldw], 56384\n\t"
      "s_nop 0\n\t"
      "buffer_load_dwordx4 %[vw1], %[sw], %[t3] offen lds\n\t"
      "s_add_u32 m0, %[ldw], 60544\n\t"
      "s_nop 0\n\t"
      "buffer_load_dwordx4 %[vw2], %[sw], %[t3] offen lds\n\t"
      "s_add_u32 m0, %[ldw], 64704\n\t"
      "s_nop 0\n\t"
      "buffer_load_dwordx4 %[vw3], %[sw], %[t3] offen lds\n\t"
      "s_add_u32 m0, %[ldx], 87040\n\t"
      "s_nop 0\n\t"
      "s_waitcnt vmcnt(24)\n\t"
      "s_barrier\n\t"
      "buffer_load_dwordx4 %[vx0], %[sx], %[t2] offen lds\n\t"
      "s_add_u32 m0, %[ldx], 95744\n\t"
      "s_nop 0\n\t"
      "buffer_load_dwordx4 %[vx1], %[sx], %[t2] offen lds\n\t"
      "s_add_u32 m0, %[ldx], 91392\n\t"
      "s_bfe_u32 %[t0], %[o2], 0x80000\n\t"
      "buffer_load_dwordx4 %[vx2], %[sx], %[t2] offen lds\n\t"
      "s_add_u32 m0, %[ldx], 100096\n\t"
      "s_lshl_b32 %[t0], %[t0], 19\n\t"
      "buffer_load_dwordx4 %[vx3], %[sx], %[t2] offen lds\n\t"
      "s_add_u32 m0, %[ldw], 104448\n\t"
      "s_nop 0\n\t"
      "s_waitcnt vmcnt(24)\n\t"
      "s_barrier\n\t"
      "buffer_load_dwordx4 %[vw0], %[sw], %[t0] offen lds\n\t"
      "s_add_u32 m0, %[ldw], 108608\n\t"
      "s_nop 0\n\t"
      "buffer_load_dwordx4 %[vw1], %[sw], %[t0] offen lds\n\t"
      "s_add_u32 m0, %[ldw], 112768\n\t"
      "s_bfe_u32 %[t1], %[o2], 0x80000\n\t"
      "buffer_load_dwordx4 %[vw2], %[sw], %[t0] offen lds\n\t"
      "s_add_u32 m0, %[ldw], 116928\n\t"
      "s_lshl_b32 %[t1], %[t1], 7\n\t"
      "buffer_load_dwordx4 %[vw3], %[sw], %[t0] offen lds\n\t"
      "s_add_u32 m0, %[ldx], 121856\n\t"
      "s_nop 0\n\t"
      "buffer_load_dwordx4 %[vg0], %[sx], %[t1] offen lds\n\t"
      "s_add_u32 m0, %[ldx], 130560\n\t"
      "s_nop 0\n\t"
      "buffer_load_dwordx4 %[vg1], %[sx], %[t1] offen lds\n\t"
      "s_add_u32 m0, %[ldx], 126208\n\t"
      "s_nop 0\n\t"
      "buffer_load_dwordx4 %[vg2], %[sx], %[t1] offen lds\n\t"
      "s_add_u32 m0, %[ldx], 134912\n\t"
      "s_nop 0\n\t"
      "buffer_load_dwordx4 %[vg3], %[sx], %[t1] offen lds\n\t"
      "s_add_u32 m0, %[ldx], 139264\n\t"
      "s_nop 0\n\t"
      "s_waitcnt vmcnt(24)\n\t"
      "s_barrier\n\t"
      "buffer_load_dwordx4 %[vx0], %[sx], %[t1] offen lds\n\t"
      "s_add_u32 m0, %[ldx], 147968\n\t"
      "s_nop 0\n\t"
      "buffer_load_dwordx4 %[vx1], %[sx], %[t1] offen lds\n\t"
      "s_add_u32 m0, %[ldx], 143616\n\t"
      "s_bfe_u32 %[t2], %[o2], 0x80008\n\t"
      "buffer_load_dwordx4 %[vx2], %[sx], %[t1] offen lds\n\t"
      "s_add_u32 m0, %[ldx], 152320\n\t"
      "s_lshl_b32 %[t2], %[t2], 7\n\t"
      "buffer_load_dwordx4 %[vx3], %[sx], %[t1] offen lds\n\t"
      "s_add_u32 m0, %[ldx], 17408\n\t"
      "s_nop 0\n\t"
      "s_waitcnt vmcnt(24)\n\t"
      "s_barrier\n\t"
      "buffer_load_dwordx4 %[vg0], %[sx], %[t2] offen lds\n\t"
      "s_add_u32 m0, %[ldx], 26112\n\t"
      "s_nop 0\n\t"
      "buffer_load_dwordx4 %[vg1], %[sx], %[t2] offen lds\n\t"
      "s_add_u32 m0, %[ldx], 21760\n\t"
      "s_bfe_u32 %[t3], %[o2], 0x80008\n\t"
      "buffer_load_dwordx4 %[vg2], %[sx], %[t2] offen lds\n\t"
      "s_add_u32 m0, %[ldx], 30464\n\t"
      "s_lshl_b32 %[t3], %[t3], 19\n\t"
      "buffer_load_dwordx4 %[vg3], %[sx], %[t2] offen lds\n\t"
      "s_add_u32 m0, %[ldw], 0\n\t"
      "s_nop 0\n\t"
      "buffer_load_dwordx4 %[vw0], %[sw], %[t3] offen lds\n\t"
      "s_add_u32 m0, %[ldw], 4160\n\t"
      "s_nop 0\n\t"
      "buffer_load_dwordx4 %[vw1], %[sw], %[t3] offen lds\n\t"
      "s_add_u32 m0, %[ldw], 8320\n\t"
      "s_nop 0\n\t"
      "buffer_load_dwordx4 %[vw2], %[sw], %[t3] offen lds\n\t"
      "s_add_u32 m0, %[ldw], 12480\n\t"
      "s_nop 0\n\t"
      "buffer_load_dwordx4 %[vw3], %[sw], %[t3] offen lds\n\t"
      "s_add_u32 m0, %[ldx], 34816\n\t"
      "s_nop 0\n\t"
      "s_waitcnt vmcnt(24)\n\t"
      "s_barrier\n\t"
      "buffer_load_dwordx4 %[vx0], %[sx], %[t2] offen lds\n\t"
      "s_add_u32 m0, %[ldx], 43520\n\t"
      "s_nop 0\n\t"
      "buffer_load_dwordx4 %[vx1], %[sx], %[t2] offen lds\n\t"
      "s_add_u32 m0, %[ldx], 39168\n\t"
      "s_bfe_u32 %[t0], %[o2], 0x80010\n\t"
      "buffer_load_dwordx4 %[vx2], %[sx], %[t2] offen lds\n\t"
      "s_add_u32 m0, %[ldx], 47872\n\t"
      "s_lshl_b32 %[t0], %[t0], 19\n\t"
      "buffer_load_dwordx4 %[vx3], %[sx], %[t2] offen lds\n\t"
      "s_add_u32 m0, %[ldw], 52224\n\t"
      "s_nop 0\n\t"
      "s_waitcnt vmcnt(24)\n\t"
      "s_barrier\n\t"
      "buffer_load_dwordx4 %[vw0], %[sw], %[t0] offen lds\n\t"
      "s_add_u32 m0, %[ldw], 56384\n\t"
      "s_nop 0\n\t"
      "buffer_load_dwordx4 %[vw1], %[sw], %[t0] offen lds\n\t"
      "s_add_u32 m0, %[ldw], 60544\n\t"
      "s_bfe_u32 %[t1], %[o2], 0x80010\n\t"
      "buffer_load_dwordx4 %[vw2], %[sw], %[t0] offen lds\n\t"
      "s_add_u32 m0, %[ldw], 64704\n\t"
      "s_lshl_b32 %[t1], %[t1], 7\n\t"
      "buffer_load_dwordx4 %[vw3], %[sw], %[t0] offen lds\n\t"
      "s_add_u32 m0, %[ldx], 69632\n\t"
      "s_nop 0\n\t"
      "buffer_load_dwordx4 %[vg0], %[sx], %[t1] offen lds\n\t"
      "s_add_u32 m0, %[ldx], 78336\n\t"
      "s_nop 0\n\t"
      "buffer_load_dwordx4 %[vg1], %[sx], %[t1] offen lds\n\t"
      "s_add_u32 m0, %[ldx], 73984\n\t"
      "s_nop 0\n\t"
      "buffer_load_dwordx4 %[vg2], %[sx], %[t1] offen lds\n\t"
      "s_add_u32 m0, %[ldx], 82688\n\t"
      "s_nop 0\n\t"
      "buffer_load_dwordx4 %[vg3], %[sx], %[t1] offen lds\n\t"
      "s_add_u32 m0, %[ldx], 87040\n\t"
      "s_nop 0\n\t"
      "s_waitcnt vmcnt(24)\n\t"
      "s_barrier\n\t"
      "buffer_load_dwordx4 %[vx0], %[sx], %[t1] offen lds\n\t"
      "s_add_u32 m0, %[ldx], 95744\n\t"
      "s_nop 0\n\t"
      "buffer_load_dwordx4 %[vx1], %[sx], %[t1] offen lds\n\t"
      "s_add_u32 m0, %[ldx], 91392\n\t"
      "s_bfe_u32 %[t2], %[o2], 0x80018\n\t"
      "buffer_load_dwordx4 %[vx2], %[sx], %[t1] offen lds\n\t"
      "s_add_u32 m0, %[ldx], 100096\n\t"
      "s_lshl_b32 %[t2], %[t2], 19\n\t"
      "buffer_load_dwordx4 %[vx3], %[sx], %[t1] offen lds\n\t"
      "s_add_u32 m0, %[ldw], 104448\n\t"
      "s_nop 0\n\t"
      "s_waitcnt vmcnt(24)\n\t"
      "s_barrier\n\t"
      "buffer_load_dwordx4 %[vw0], %[sw], %[t2] offen lds\n\t"
      "s_add_u32 m0, %[ldw], 108608\n\t"
      "s_nop 0\n\t"
      "buffer_load_dwordx4 %[vw1], %[sw], %[t2] offen lds\n\t"
      "s_add_u32 m0, %[ldw], 112768\n\t"
      "s_bfe_u32 %[t3], %[o2], 0x80018\n\t"
      "buffer_load_dwordx4 %[vw2], %[sw], %[t2] offen lds\n\t"
      "s_add_u32 m0, %[ldw], 116928\n\t"
      "s_lshl_b32 %[t3], %[t3], 7\n\t"
      "buffer_load_dwordx4 %[vw3], %[sw], %[t2] offen lds\n\t"
      "s_add_u32 m0, %[ldx], 121856\n\t"
      "s_nop 0\n\t"
      "buffer_load_dwordx4 %[vg2], %[sx], %[t3] offen lds\n\t"
      "s_add_u32 m0, %[ldx], 130560\n\t"
      "s_nop 0\n\t"
      "buffer_load_dwordx4 %[vg3], %[sx], %[t3] offen lds\n\t"
      "s_add_u32 m0, %[ldx], 126208\n\t"
      "s_nop 0\n\t"
      "buffer_load_dwordx4 %[vg0], %[sx], %[t3] offen lds\n\t"
      "s_add_u32 m0, %[ldx], 134912\n\t"
      "s_nop 0\n\t"
      "buffer_load_dwordx4 %[vg1], %[sx], %[t3] offen lds\n\t"
      "s_add_u32 m0, %[ldx], 139264\n\t"
      "s_bfe_u32 %[t0], %[o3], 0x80000\n\t"
      "s_waitcnt vmcnt(24)\n\t"
      "s_barrier\n\t"
      "buffer_load_dwordx4 %[vx2], %[sx], %[t3] offen lds\n\t"
      "s_add_u32 m0, %[ldx], 147968\n\t"
      "s_lshl_b32 %[t0], %[t0], 19\n\t"
      "buffer_load_dwordx4 %[vx3], %[sx], %[t3] offen lds\n\t"
      "s_add_u32 m0, %[ldw], 0\n\t"
      "s_nop 0\n\t"
      "s_waitcnt vmcnt(22)\n\t"
      "s_barrier\n\t"
      "buffer_load_dwordx4 %[vw0], %[sw], %[t0] offen lds\n\t"
      "s_add_u32 m0, %[ldw], 4160\n\t"
      "s_nop 0\n\t"
      "buffer_load_dwordx4 %[vw1], %[sw], %[t0] offen lds\n\t"
      "s_add_u32 m0, %[ldw], 8320\n\t"
      "s_bfe_u32 %[t1], %[o3], 0x80000\n\t"
      "buffer_load_dwordx4 %[vw2], %[sw], %[t0] offen lds\n\t"
      "s_add_u32 m0, %[ldw], 12480\n\t"
      "s_lshl_b32 %[t1], %[t1], 7\n\t"
      "buffer_load_dwordx4 %[vw3], %[sw], %[t0] offen lds\n\t"
      "s_add_u32 m0, %[ldx], 143616\n\t"
      "s_nop 0\n\t"
      "buffer_load_dwordx4 %[vg2], %[sx], %[t1] offen lds\n\t"
      "s_add_u32 m0, %[ldx], 152320\n\t"
      "s_nop 0\n\t"
      "buffer_load_dwordx4 %[vg3], %[sx], %[t1] offen lds\n\t"
      "s_add_u32 m0, %[ldx], 17408\n\t"
      "s_nop 0\n\t"
      "buffer_load_dwordx4 %[vg0], %[sx], %[t1] offen lds\n\t"
      "s_add_u32 m0, %[ldx], 26112\n\t"
      "s_nop 0\n\t"
      "buffer_load_dwordx4 %[vg1], %[sx], %[t1] offen lds\n\t"
      "s_add_u32 m0, %[ldx], 21760\n\t"
      "s_bfe_u32 %[t2], %[o3], 0x80008\n\t"
      "buffer_load_dwordx4 %[vx2], %[sx], %[t1] offen lds\n\t"
      "s_add_u32 m0, %[ldx], 30464\n\t"
      "s_lshl_b32 %[t2], %[t2], 19\n\t"
      "buffer_load_dwordx4 %[vx3], %[sx], %[t1] offen lds\n\t"
      "s_add_u32 m0, %[ldw], 34816\n\t"
      "s_nop 0\n\t"
      "s_waitcnt vmcnt(24)\n\t"
      "s_barrier\n\t"
      "buffer_load_dwordx4 %[vw0], %[sw], %[t2] offen lds\n\t"
      "s_add_u32 m0, %[ldw], 38976\n\t"
      "s_nop 0\n\t"
      "buffer_load_dwordx4 %[vw1], %[sw], %[t2] offen lds\n\t"
      "s_add_u32 m0, %[ldw], 43136\n\t"
      "s_bfe_u32 %[t3], %[o3], 0x80008\n\t"
      "buffer_load_dwordx4 %[vw2], %[sw], %[t2] offen lds\n\t"
      "s_add_u32 m0, %[ldw], 47296\n\t"
      "s_lshl_b32 %[t3], %[t3], 7\n\t"
      "buffer_load_dwordx4 %[vw3], %[sw], %[t2] offen lds\n\t"
      "s_waitcnt vmcnt(24)\n\t"
      "s_barrier\n\t"
      "s_add_u32 m0, %[ldx], 52224\n\t"
      "s_bfe_u32 %[t0], %[o3], 0x80010\n\t"
      "s_waitcnt vmcnt(18)\n\t"
      "s_barrier\n\t"
      "buffer_load_dwordx4 %[vg2], %[sx], %[t3] offen lds\n\t"
      "s_add_u32 m0, %[ldx], 60928\n\t"
      "s_lshl_b32 %[t0], %[t0], 19\n\t"
      "buffer_load_dwordx4 %[vg3], %[sx], %[t3] offen lds\n\t"
      "s_add_u32 m0, %[ldw], 69632\n\t"
      "s_nop 0\n\t"
      "buffer_load_dwordx4 %[vw0], %[sw], %[t0] offen lds\n\t"
      "s_add_u32 m0, %[ldw], 73792\n\t"
      "s_nop 0\n\t"
      "buffer_load_dwordx4 %[vw1], %[sw], %[t0] offen lds\n\t"
      "s_add_u32 m0, %[ldw], 77952\n\t"
      "s_bfe_u32 %[t1], %[o3], 0x80010\n\t"
      "buffer_load_dwordx4 %[vw2], %[sw], %[t0] offen lds\n\t"
      "s_add_u32 m0, %[ldw], 82112\n\t"
      "s_lshl_b32 %[t1], %[t1], 7\n\t"
      "buffer_load_dwordx4 %[vw3], %[sw], %[t0] offen lds\n\t"
      "s_add_u32 m0, %[ldx], 56576\n\t"
      "s_bfe_u32 %[t2], %[o3], 0x80018\n\t"
      "buffer_load_dwordx4 %[vg2], %[sx], %[t1] offen lds\n\t"
      "s_add_u32 m0, %[ldx], 65280\n\t"
      "s_lshl_b32 %[t2], %[t2], 19\n\t"
      "buffer_load_dwordx4 %[vg3], %[sx], %[t1] offen lds\n\t"
      "s_add_u32 m0, %[ldw], 87040\n\t"
      "s_nop 0\n\t"
      "s_waitcnt vmcnt(24)\n\t"
      "s_barrier\n\t"
      "buffer_load_dwordx4 %[vw0], %[sw], %[t2] offen lds\n\t"
      "s_add_u32 m0, %[ldw], 91200\n\t"
      "s_nop 0\n\t"
      "buffer_load_dwordx4 %[vw1], %[sw], %[t2] offen lds\n\t"
      "s_add_u32 m0, %[ldw], 95360\n\t"
      "s_bfe_u32 %[t3], %[o3], 0x80018\n\t"
      "buffer_load_dwordx4 %[vw2], %[sw], %[t2] offen lds\n\t"
      "s_add_u32 m0, %[ldw], 99520\n\t"
      "s_lshl_b32 %[t3], %[t3], 7\n\t"
      "buffer_load_dwordx4 %[vw3], %[sw], %[t2] offen lds\n\t"
      "s_add_u32 m0, %[ldx], 121856\n\t"
      "s_nop 0\n\t"
      "s_waitcnt vmcnt(26)\n\t"
      "s_barrier\n\t"
      "buffer_load_dwordx4 %[vg2], %[sx], %[t3] offen lds\n\t"
      "s_add_u32 m0, %[ldx], 130560\n\t"
      "s_nop 0\n\t"
      "buffer_load_dwordx4 %[vg3], %[sx], %[t3] offen lds\n\t"
      "s_add_u32 m0, %[ldx], 126208\n\t"
      "s_nop 0\n\t"
      "s_waitcnt vmcnt(22)\n\t"
      "s_barrier\n\t"
      "buffer_load_dwordx4 %[vg0], %[sx], %[t3] offen lds\n\t"
      "s_add_u32 m0, %[ldx], 134912\n\t"
      "s_nop 0\n\t"
      "buffer_load_dwordx4 %[vg1], %[sx], %[t3] offen lds\n\t"
      "s_add_u32 m0, %[ldx], 139264\n\t"
      "s_nop 0\n\t"
      "s_waitcnt vmcnt(22)\n\t"
      "s_barrier\n\t"
      "buffer_load_dwordx4 %[vx2], %[sx], %[t3] offen lds\n\t"
      "s_add_u32 m0, %[ldx], 147968\n\t"
      "s_nop 0\n\t"
      "buffer_load_dwordx4 %[vx3], %[sx], %[t3] offen lds\n\t"
      "s_add_u32 m0, %[ldx], 143616\n\t"
      "s_bfe_u32 %[t0], %[o3], 0x80008\n\t"
      "s_waitcnt vmcnt(22)\n\t"
      "s_barrier\n\t"
      "buffer_load_dwordx4 %[vx2], %[sx], %[t1] offen lds\n\t"
      "s_add_u32 m0, %[ldx], 152320\n\t"
      "s_lshl_b32 %[t0], %[t0], 7\n\t"
      "buffer_load_dwordx4 %[vx3], %[sx], %[t1] offen lds\n\t"
      "s_add_u32 m0, %[ldx], 17408\n\t"
      "s_nop 0\n\t"
      "s_waitcnt vmcnt(18)\n\t"
      "s_barrier\n\t"
      "buffer_load_dwordx4 %[vx2], %[sx], %[t0] offen lds\n\t"
      "s_add_u32 m0, %[ldx], 26112\n\t"
      "s_nop 0\n\t"
      "buffer_load_dwordx4 %[vx3], %[sx], %[t0] offen lds\n\t"
      "s_add_u32 m0, %[ldx], 21760\n\t"
      "s_nop 0\n\t"
      "s_waitcnt vmcnt(14)\n\t"
      "s_barrier\n\t"
      "buffer_load_dwordx4 %[vg0], %[sx], %[t0] offen lds\n\t"
      "s_add_u32 m0, %[ldx], 30464\n\t"
      "s_nop 0\n\t"
      "buffer_load_dwordx4 %[vg1], %[sx], %[t0] offen lds\n\t"
      "s_add_u32 m0, %[ldx], 52224\n\t"
      "s_nop 0\n\t"
      "s_waitcnt vmcnt(10)\n\t"
      "s_barrier\n\t"
      "buffer_load_dwordx4 %[vg0], %[sx], %[t1] offen lds\n\t"
      "s_add_u32 m0, %[ldx], 60928\n\t"
      "s_nop 0\n\t"
      "buffer_load_dwordx4 %[vg1], %[sx], %[t1] offen lds\n\t"
      "s_add_u32 m0, %[ldx], 56576\n\t"
      "s_nop 0\n\t"
      "s_waitcnt vmcnt(10)\n\t"
      "s_barrier\n\t"
      "buffer_load_dwordx4 %[vx0], %[sx], %[t1] offen lds\n\t"
      "s_add_u32 m0, %[ldx], 65280\n\t"
      "s_nop 0\n\t"
      "buffer_load_dwordx4 %[vx1], %[sx], %[t1] offen lds\n\t"
      "s_add_u32 m0, %[ldx], 121856\n\t"
      "s_nop 0\n\t"
      "s_waitcnt vmcnt(10)\n\t"
      "s_barrier\n\t"
      "buffer_load_dwordx4 %[vx0], %[sx], %[t3] offen lds\n\t"
      "s_add_u32 m0, %[ldx], 130560\n\t"
      "s_nop 0\n\t"
      "buffer_load_dwordx4 %[vx1], %[sx], %[t3] offen lds\n\t"
      "s_add_u32 m0, %[ldx], 126208\n\t"
      "s_bfe_u32 %[t1], %[o3], 0x80000\n\t"
      "s_waitcnt vmcnt(10)\n\t"
      "s_barrier\n\t"
      "buffer_load_dwordx4 %[vx0], %[sx], %[t0] offen lds\n\t"
      "s_add_u32 m0, %[ldx], 134912\n\t"
      "s_lshl_b32 %[t1], %[t1], 7\n\t"
      "buffer_load_dwordx4 %[vx1], %[sx], %[t0] offen lds\n\t"
      "s_add_u32 m0, %[ldx], 139264\n\t"
      "s_bfe_u32 %[t2], %[o2], 0x80018\n\t"
      "s_waitcnt vmcnt(10)\n\t"
      "s_barrier\n\t"
      "buffer_load_dwordx4 %[vx0], %[sx], %[t1] offen lds\n\t"
      "s_add_u32 m0, %[ldx], 147968\n\t"
      "s_lshl_b32 %[t2], %[t2], 7\n\t"
      "buffer_load_dwordx4 %[vx1], %[sx], %[t1] offen lds\n\t"
      "s_add_u32 m0, %[ldx], 143616\n\t"
      "s_nop 0\n\t"
      "s_waitcnt vmcnt(10)\n\t"
      "s_barrier\n\t"
      "buffer_load_dwordx4 %[vx0], %[sx], %[t2] offen lds\n\t"
      "s_add_u32 m0, %[ldx], 152320\n\t"
      "s_nop 0\n\t"
      "buffer_load_dwordx4 %[vx1], %[sx], %[t2] offen lds\n\t"
      "s_waitcnt vmcnt(10)\n\t"
      "s_barrier\n\t"
      "s_waitcnt vmcnt(8)\n\t"
      "s_barrier\n\t"
      "s_waitcnt vmcnt(6)\n\t"
      "s_barrier\n\t"
      "s_waitcnt vmcnt(4)\n\t"
      "s_barrier\n\t"
      "s_waitcnt vmcnt(2)\n\t"
      "s_barrier\n\t"
      "s_waitcnt vmcnt(0) lgkmcnt(0)\n\t"
      "s_barrier"
      : [t0] "+s"(t0_), [t1] "+s"(t1_), [t2] "+s"(t2_), [t3] "+s"(t3_)
      : [vx0] "v"(vx[0]), [vx1] "v"(vx[1]), [vx2] "v"(vx[2]), [vx3] "v"(vx[3]), [vg0] "v"(vg[0]), [vg1] "v"(vg[1]), [vg2] "v"(vg[2]), [vg3] "v"(vg[3]), [vw0] "v"(vw[0]), [vw1] "v"(vw[1]), [vw2] "v"(vw[2]), [vw3] "v"(vw[3]), [sx] "s"(sx), [sw] "s"(sw), [ldx] "s"(ldx), [ldw] "s"(ldw), [o0] "s"(ow[0]), [o1] "s"(ow[1]), [o2] "s"(ow[2]), [o3] "s"(ow[3])
      : "memory", "scc");
    asm volatile("" : "+s"(cv));
    const unsigned seq = cv + 1u;
    STAMP(1);
    if (b >= 8 ? lw > 0 : b > 0 && lw > 1) {
      STAMPS_OUT(ws);
      return;
    }
    unsigned* const tmo = (unsigned*)(ws + WS_TMO);
    const __amdgpu_buffer_rsrc_t rw = make_rsrc(ws, (unsigned)WS_XBUF);
    if (lw == 1) {
      unsigned char* tgt = P.ws[lane >> 3] + WS_AMAX + (seq & 1u) * 8192u + (lane & 7) * 1024 + skip3((rank * 8 + b) * 8);
      asm volatile("" : "+v"(tgt));
      wg_barrier();
      {
      u32x4 g;
      const bool ok = poll(g, [&]() { asm volatile("" ::: "memory"); return __builtin_amdgcn_raw_buffer_load_b128(rw, skip3(lane * 16), (unsigned)WS_AGG + b * 2048, AUX_AGENT); },
                           [&](const u32x4& v) { return __all(v[1] == seq && v[3] == seq); }, tmo, seq, (unsigned*)(ws + WS_AGG + b * 2048 + skip3(lane * 16)));
      const unsigned mx = ok ? wave_max_u32(max(g[0], g[2])) : NAN_BITS;
      __hip_atomic_store((GAS unsigned long long*)tgt, ((unsigned long long)seq << 32) | mx, __ATOMIC_RELAXED, __HIP_MEMORY_SCOPE_SYSTEM);
      }
      STAMP(2);
      wg_barrier();
    } else {
      if (tx == 256) Lm[0] = 0;
      __builtin_amdgcn_s_waitcnt(0xC07F);
      wg_barrier();
      if (b < 8) wg_barrier();
    }
    if (lw == 0) {
      const i32x4 rq = {wr[0], wr[1], (int)WS_XBUF, 0x00020000};
      const unsigned va = skip3(lane * 8), sa = (unsigned)WS_AMAX + (seq & 1u) * 8192u + xcd * 1024;
      const unsigned long long t0 = __builtin_amdgcn_s_memrealtime();
      unsigned have = 0;
      u32x2 g;
      asm volatile("buffer_load_dwordx2 v[40:41], %1, %2, %3 offen sc0 sc1" : "={v[40:41]}"(g) : "v"(va), "s"(rq), "s"(sa) : "memory");
      for (unsigned it = 1;; ++it) {
        unsigned long long ok, fin;
        unsigned h;
        asm volatile("s_waitcnt vmcnt(0)\n\tv_cmp_eq_u32 %1, %7, v41\n\tv_cmp_ge_u32 %2, %8, v40\n\ts_and_b64 %2, %2, %1\n\ts_cmp_eq_u64 %2, -1\n\ts_cbranch_scc0 1f\n\t"
                     "s_barrier\n1:\n\tv_cndmask_b32 %3, 0, v40, %1\n\ts_cmp_eq_u64 %1, -1\n\ts_cbranch_scc1 2f\n\t"
                     "buffer_load_dwordx2 v[40:41], %4, %5, %6 offen sc0 sc1\n2:"
                     : "+{v[40:41]}"(g), "=&s"(ok), "=&s"(fin), "=&v"(h) : "v"(va), "s"(rq), "s"(sa), "s"(seq), "s"(have) : "scc", "memory");
        if (fin == ~0ull) break;
        if (__any(h > have)) {
          have = wave_max_u32(h);
          if (lane == 0) Lm[0] = inv_of(have);
          __builtin_amdgcn_s_waitcnt(0xC07F);
        }
        if ((it & 63) == 0) {
          asm volatile("buffer_wbl2 sc1" ::: "memory");
          if (expired(tmo, seq, t0)) asm volatile("s_waitcnt vmcnt(0)\n\tv_mov_b32 v40, %1\n\tv_mov_b32 v41, %2" : "+{v[40:41]}"(g) : "s"(NAN_BITS), "s"(seq));
        }
      }
    }
    if (lw) wg_barrier();
    if (lw == 0 && b >= 8 && b < 16) {
      __builtin_amdgcn_s_sleep(WBL);
      asm volatile("buffer_wbl2 sc1" ::: "memory");
    }
    if (tx == 256) __hip_atomic_store(cnt, seq, __ATOMIC_RELAXED, __HIP_MEMORY_SCOPE_AGENT);
    if (b == 0) {
      const unsigned vo = (unsigned)((lw * 2 + (lane >> 5)) * 1024) + skip3((lane & 31) * 16);
      const bool mine = (lw * 64 + lane) / 32 == rank || (lane & 31) >= NPUSH;
      u32x4 g;
      if (!poll(g, [&]() { asm volatile("" ::: "memory"); return __builtin_amdgcn_raw_buffer_load_b128(rw, vo, (unsigned)WS_DONE, AUX_SYS); },
                [&](const u32x4& v) { return __all(mine || (v[0] >= seq && v[1] >= seq && v[2] >= seq && v[3] >= seq)); }, tmo, seq)) {
        if (lane == 0) P.out[rank][(size_t)(rank * M) * OUT_LD] = 0x7f;
      }
      STAMP(6);
    }
    STAMPS_OUT(ws);
    return;
  }

  const int wm = (wv & 1) * 64, wn = (wv >> 1) * 64;
  f16v A[4];
  const unsigned xb = l0 + (wm / 8 + (fr >> 3)) * PX + (fr & 7) * 128 + ((fq ^ ((fr >> 1) & 3)) * 16);
  const int q4 = (lane & 15) >> 2;
  const unsigned wb0 = l0 + 2 * fq * PW + q4 * 256 + (((wn / 16) ^ (q4 << 1)) * 32) + (lane & 3) * 8;
  const unsigned wb1 = l0 + 2 * fq * PW + q4 * 256 + ((((wn / 16) | 2) ^ (q4 << 1)) * 32) + (lane & 3) * 8;
  const unsigned one = fq == (fr >> 2) ? 0x3f80u << ((fr & 1) * 16) : 0u;
  u32x4 sel0 = {(fr & 2) ? 0u : one, (fr & 2) ? one : 0u, 0u, 0u}, sel1 = {0u, 0u, sel0[0], sel0[1]};
  unsigned vxb = skip3(lane * 16);
  unsigned xa = xb, wa0 = wb0, wa1 = wb1, roff = 0, cnt = LOOP_T1 - 1;
  i32x4 wr;
  unsigned cv, oc = (unsigned)WS_CNT + b * 1024;
  asm volatile("s_load_dwordx4 %0, %4, %6\n\ts_waitcnt lgkmcnt(0)"
               : "=&s"(wr), "+v"(sel0), "+v"(sel1), "+v"(vxb)
               : "s"(__builtin_amdgcn_kernarg_segment_ptr()), "s"(sxw), "i"(KA_SELF));
  unsigned char* const ws = (unsigned char*)(((unsigned long long)(unsigned)wr[1] << 32) | (unsigned)wr[0]);
  const int rank = wr[2];
  i32x4 rxs = {wr[0], wr[1], (int)WS_STAMP, 0x00020000};
  unsigned s2o[2], s2i[2], s1o[2], s1i[2];
#pragma unroll
  for (int h = 0; h < 2; ++h) {
    s2o[h] = (unsigned)WS_XBUF + ((unsigned)(b * 4 + wv) << XSH) + h * 4096; s2i[h] = (unsigned)WS_XBUF + ((unsigned)((b ^ X2) * 4 + wv) << XSH) + h * 4096;
    s1o[h] = (unsigned)WS_XB1 + ((unsigned)(b * 4 + wv) << XSH) + h * 4096; s1i[h] = (unsigned)WS_XB1 + ((unsigned)((b ^ X1) * 4 + wv) << XSH) + h * 4096;
  }
  unsigned sag = (unsigned)WS_AGG + grp * 2048 + skip3(gpos * 32 + wv * 8), vag = lane == 63 ? 0u : 0x40000000u;
  asm volatile("" : "+s"(rxs), "+s"(s2o[0]), "+s"(s2o[1]), "+s"(s2i[0]), "+s"(s2i[1]), "+s"(s1o[0]), "+s"(s1o[1]), "+s"(s1i[0]), "+s"(s1i[1]), "+s"(sag), "+v"(vag));
  unsigned c0, c1, c2;
  const unsigned vpf = (lane >> 4) * 2048 + skip3((lane & 15) * 64);
  asm volatile(
      "v_mov_b64 v[128:129], 0\n\t"
      "v_mov_b64 v[130:131], 0\n\t"
      "v_mov_b64 v[132:133], 0\n\t"
      "v_mov_b64 v[134:135], 0\n\t"
      "v_mov_b64 v[136:137], 0\n\t"
      "v_mov_b64 v[138:139], 0\n\t"
      "v_mov_b64 v[140:141], 0\n\t"
      "v_mov_b64 v[142:143], 0\n\t"
      "v_mov_b64 v[144:145], 0\n\t"
      "v_mov_b64 v[146:147], 0\n\t"
      "v_mov_b64 v[148:149], 0\n\t"
      "v_mov_b64 v[150:151], 0\n\t"
      "v_mov_b64 v[152:153], 0\n\t"
      "v_mov_b64 v[154:155], 0\n\t"
      "v_mov_b64 v[156:157], 0\n\t"
      "v_mov_b64 v[158:159], 0\n\t"
      "v_mov_b64 v[160:161], 0\n\t"
      "v_mov_b64 v[162:163], 0\n\t"
      "v_mov_b64 v[164:165], 0\n\t"
      "v_mov_b64 v[166:167], 0\n\t"
      "v_mov_b64 v[168:169], 0\n\t"
      "v_mov_b64 v[170:171], 0\n\t"
      "v_mov_b64 v[172:173], 0\n\t"
      "v_mov_b64 v[174:175], 0\n\t"
      "v_mov_b64 v[176:177], 0\n\t"
      "v_mov_b64 v[178:179], 0\n\t"
      "v_mov_b64 v[180:181], 0\n\t"
      "v_mov_b64 v[182:183], 0\n\t"
      "v_mov_b64 v[184:185], 0\n\t"
      "v_mov_b64 v[186:187], 0\n\t"
      "v_mov_b64 v[188:189], 0\n\t"
      "v_mov_b64 v[190:191], 0\n\t"
      "v_mov_b64 v[32:33], 0\n\t"
      "v_mov_b64 v[34:35], 0\n\t"
      "v_mov_b64 v[36:37], 0\n\t"
      "v_mov_b64 v[38:39], 0\n\t"
      "v_mov_b64 v[40:41], 0\n\t"
      "v_mov_b64 v[42:43], 0\n\t"
      "v_mov_b64 v[44:45], 0\n\t"
      "v_mov_b64 v[46:47], 0\n\t"
      "v_mov_b64 v[48:49], 0\n\t"
      "v_mov_b64 v[50:51], 0\n\t"
      "v_mov_b64 v[52:53], 0\n\t"
      "v_mov_b64 v[54:55], 0\n\t"
      "v_mov_b64 v[56:57], 0\n\t"
      "v_mov_b64 v[58:59], 0\n\t"
      "v_mov_b64 v[60:61], 0\n\t"
      "v_mov_b64 v[62:63], 0\n\t"
      "v_mov_b64 v[64:65], 0\n\t"
      "v_mov_b64 v[66:67], 0\n\t"
      "v_mov_b64 v[68:69], 0\n\t"
      "v_mov_b64 v[70:71], 0\n\t"
      "v_mov_b64 v[72:73], 0\n\t"
      "v_mov_b64 v[74:75], 0\n\t"
      "v_mov_b64 v[76:77], 0\n\t"
      "v_mov_b64 v[78:79], 0\n\t"
      "v_mov_b64 v[80:81], 0\n\t"
      "v_mov_b64 v[82:83], 0\n\t"
      "v_mov_b64 v[84:85], 0\n\t"
      "v_mov_b64 v[86:87], 0\n\t"
      "v_mov_b64 v[88:89], 0\n\t"
      "v_mov_b64 v[90:91], 0\n\t"
      "v_mov_b64 v[92:93], 0\n\t"
      "v_mov_b64 v[94:95], 0\n\t"
      "s_barrier\n\t"
      "ds_read_b128 a[16:19], %[xa] offset:17408\n\t"
      "ds_read_b128 a[20:23], %[xa] offset:17472\n\t"
      "ds_read_b128 a[28:31], %[xa] offset:19648\n\t"
      "ds_read_b128 a[24:27], %[xa] offset:19584\n\t"
      "ds_read_b64_tr_b16 a[0:1], %[wa0] offset:0\n\t"
      "ds_read_b64_tr_b16 a[2:3], %[wa0] offset:1040\n\t"
      "ds_read_b64_tr_b16 a[32:33], %[wa0] offset:8320\n\t"
      "ds_read_b64_tr_b16 a[34:35], %[wa0] offset:9360\n\t"
      "ds_read_b64_tr_b16 a[4:5], %[wa0] offset:32\n\t"
      "ds_read_b64_tr_b16 a[6:7], %[wa0] offset:1072\n\t"
      "ds_read_b64_tr_b16 a[36:37], %[wa0] offset:8352\n\t"
      "ds_read_b64_tr_b16 a[38:39], %[wa0] offset:9392\n\t"
      "ds_read_b64_tr_b16 a[8:9], %[wa1] offset:0\n\t"
      "ds_read_b64_tr_b16 a[10:11], %[wa1] offset:1040\n\t"
      "ds_read_b64_tr_b16 a[40:41], %[wa1] offset:8320\n\t"
      "ds_read_b64_tr_b16 a[42:43], %[wa1] offset:9360"
      : "=&{v[32:47]}"(A[0]), "=&{v[48:63]}"(A[1]), "=&{v[64:79]}"(A[2]), "=&{v[80:95]}"(A[3]), [xa] "+v"(xa), [wa0] "+v"(wa0), [wa1] "+v"(wa1)
      :
      : "memory", "scc", ACLOB, "v191");
  STAMP(8);
  asm volatile(
      ".p2align 5\n\t"
      "s_waitcnt lgkmcnt(8)\n\t"
      "1:\n\t"
      "v_mfma_f32_16x16x32_bf16 v[128:131], a[0:3], a[16:19], v[128:131]\n\t"
      "ds_read_b64_tr_b16 a[12:13], %[wa1] offset:32\n\t"
      "v_mfma_f32_16x16x32_bf16 v[128:131], a[32:35], a[20:23], v[128:131]\n\t"
      "ds_read_b64_tr_b16 a[14:15], %[wa1] offset:1072\n\t"
      "v_mfma_f32_16x16x32_bf16 v[144:147], a[32:35], a[28:31], v[144:147]\n\t"
      "ds_read_b64_tr_b16 a[44:45], %[wa1] offset:8352\n\t"
      "v_mfma_f32_16x16x32_bf16 v[144:147], a[0:3], a[24:27], v[144:147]\n\t"
      "ds_read_b64_tr_b16 a[46:47], %[wa1] offset:9392\n\t"
      "s_waitcnt lgkmcnt(8)\n\t"
      "v_mfma_f32_16x16x32_bf16 v[148:151], a[4:7], a[24:27], v[148:151]\n\t"
      "ds_read_b128 a[48:51], %[xa] offset:21760\n\t"
      "v_mfma_f32_16x16x32_bf16 v[148:151], a[36:39], a[28:31], v[148:151]\n\t"
      "ds_read_b128 a[52:55], %[xa] offset:21824\n\t"
      "v_mfma_f32_16x16x32_bf16 v[132:135], a[36:39], a[20:23], v[132:135]\n\t"
      "ds_read_b128 a[60:63], %[xa] offset:24000\n\t"
      "v_mfma_f32_16x16x32_bf16 v[132:135], a[4:7], a[16:19], v[132:135]\n\t"
      "ds_read_b128 a[56:59], %[xa] offset:23936\n\t"
      "s_waitcnt lgkmcnt(8)\n\t"
      "v_mfma_f32_16x16x32_bf16 v[136:139], a[8:11], a[16:19], v[136:139]\n\t"
      "v_mfma_f32_16x16x32_bf16 v[136:139], a[40:43], a[20:23], v[136:139]\n\t"
      "v_mfma_f32_16x16x32_bf16 v[152:155], a[40:43], a[28:31], v[152:155]\n\t"
      "v_mfma_f32_16x16x32_bf16 v[152:155], a[8:11], a[24:27], v[152:155]\n\t"
      "s_waitcnt lgkmcnt(4)\n\t"
      "v_mfma_f32_16x16x32_bf16 v[156:159], a[12:15], a[24:27], v[156:159]\n\t"
      "v_mfma_f32_16x16x32_bf16 v[156:159], a[44:47], a[28:31], v[156:159]\n\t"
      "v_mfma_f32_16x16x32_bf16 v[140:143], a[44:47], a[20:23], v[140:143]\n\t"
      "v_mfma_f32_16x16x32_bf16 v[140:143], a[12:15], a[16:19], v[140:143]\n\t"
      "s_waitcnt lgkmcnt(0)\n\t"
      "s_barrier\n\t"
      "v_mfma_f32_16x16x32_bf16 v[172:175], a[12:15], a[48:51], v[172:175]\n\t"
      "ds_read_b128 a[16:19], %[xa] offset:34816\n\t"
      "v_mfma_f32_16x16x32_bf16 v[172:175], a[44:47], a[52:55], v[172:175]\n\t"
      "ds_read_b128 a[20:23], %[xa] offset:34880\n\t"
      "v_mfma_f32_16x16x32_bf16 v[188:191], a[44:47], a[60:63], v[188:191]\n\t"
      "ds_read_b128 a[28:31], %[xa] offset:37056\n\t"
      "v_mfma_f32_16x16x32_bf16 v[188:191], a[12:15], a[56:59], v[188:191]\n\t"
      "ds_read_b128 a[24:27], %[xa] offset:36992\n\t"
      "v_mfma_f32_16x16x32_bf16 v[184:187], a[8:11], a[56:59], v[184:187]\n\t"
      "s_add_u32 %[roff], %[roff], 52224\n\t"
      "v_mfma_f32_16x16x32_bf16 v[184:187], a[40:43], a[60:63], v[184:187]\n\t"
      "s_cmp_eq_u32 %[roff], 156672\n\t"
      "v_mfma_f32_16x16x32_bf16 v[168:171], a[40:43], a[52:55], v[168:171]\n\t"
      "s_cselect_b32 %[roff], 0, %[roff]\n\t"
      "v_mfma_f32_16x16x32_bf16 v[168:171], a[8:11], a[48:51], v[168:171]\n\t"
      "v_mfma_f32_16x16x32_bf16 v[164:167], a[4:7], a[48:51], v[164:167]\n\t"
      "v_mfma_f32_16x16x32_bf16 v[164:167], a[36:39], a[52:55], v[164:167]\n\t"
      "v_mfma_f32_16x16x32_bf16 v[180:183], a[36:39], a[60:63], v[180:183]\n\t"
      "v_mfma_f32_16x16x32_bf16 v[180:183], a[4:7], a[56:59], v[180:183]\n\t"
      "v_mfma_f32_16x16x32_bf16 v[176:179], a[0:3], a[56:59], v[176:179]\n\t"
      "v_mfma_f32_16x16x32_bf16 v[176:179], a[32:35], a[60:63], v[176:179]\n\t"
      "v_mfma_f32_16x16x32_bf16 v[160:163], a[32:35], a[52:55], v[160:163]\n\t"
      "v_mfma_f32_16x16x32_bf16 v[160:163], a[0:3], a[48:51], v[160:163]\n\t"
      "s_waitcnt lgkmcnt(0)\n\t"
      "v_mfma_f32_16x16x32_bf16 v[32:35], a[0:3], a[16:19], v[32:35]\n\t"
      "ds_read_b128 a[48:51], %[xa] offset:39168\n\t"
      "v_mfma_f32_16x16x32_bf16 v[32:35], a[32:35], a[20:23], v[32:35]\n\t"
      "ds_read_b128 a[52:55], %[xa] offset:39232\n\t"
      "v_mfma_f32_16x16x32_bf16 v[36:39], a[36:39], a[20:23], v[36:39]\n\t"
      "ds_read_b128 a[60:63], %[xa] offset:41408\n\t"
      "v_mfma_f32_16x16x32_bf16 v[36:39], a[4:7], a[16:19], v[36:39]\n\t"
      "ds_read_b128 a[56:59], %[xa] offset:41344\n\t"
      "v_mfma_f32_16x16x32_bf16 v[40:43], a[8:11], a[16:19], v[40:43]\n\t"
      "s_cmp_lg_u32 %[cnt], 1\n\t"
      "v_mfma_f32_16x16x32_bf16 v[40:43], a[40:43], a[20:23], v[40:43]\n\t"
      "s_cbranch_scc1 9f\n\ts_load_dword %[cv], %[ws], %[oc]\n9:\n\t"
      "v_mfma_f32_16x16x32_bf16 v[44:47], a[44:47], a[20:23], v[44:47]\n\t"
      "v_add_u32 %[xa], %[roff], %[xb]\n\t"
      "v_mfma_f32_16x16x32_bf16 v[44:47], a[12:15], a[16:19], v[44:47]\n\t"
      "v_add_u32 %[wa0], %[roff], %[wb0]\n\t"
      "v_mfma_f32_16x16x32_bf16 v[60:63], a[12:15], a[24:27], v[60:63]\n\t"
      "v_add_u32 %[wa1], %[roff], %[wb1]\n\t"
      "v_mfma_f32_16x16x32_bf16 v[60:63], a[44:47], a[28:31], v[60:63]\n\t"
      "s_sub_u32 %[cnt], %[cnt], 1\n\t"
      "v_mfma_f32_16x16x32_bf16 v[56:59], a[40:43], a[28:31], v[56:59]\n\t"
      "s_cmp_lg_u32 %[cnt], 0\n\t"
      "v_mfma_f32_16x16x32_bf16 v[56:59], a[8:11], a[24:27], v[56:59]\n\t"
      "v_mfma_f32_16x16x32_bf16 v[52:55], a[4:7], a[24:27], v[52:55]\n\t"
      "v_mfma_f32_16x16x32_bf16 v[52:55], a[36:39], a[28:31], v[52:55]\n\t"
      "v_mfma_f32_16x16x32_bf16 v[48:51], a[32:35], a[28:31], v[48:51]\n\t"
      "v_mfma_f32_16x16x32_bf16 v[48:51], a[0:3], a[24:27], v[48:51]\n\t"
      "s_waitcnt lgkmcnt(0)\n\t"
      "s_barrier\n\t"
      "v_mfma_f32_16x16x32_bf16 v[64:67], a[0:3], a[48:51], v[64:67]\n\t"
      "ds_read_b128 a[16:19], %[xa] offset:17408\n\t"
      "v_mfma_f32_16x16x32_bf16 v[64:67], a[32:35], a[52:55], v[64:67]\n\t"
      "ds_read_b128 a[20:23], %[xa] offset:17472\n\t"
      "v_mfma_f32_16x16x32_bf16 v[80:83], a[32:35], a[60:63], v[80:83]\n\t"
      "ds_read_b128 a[28:31], %[xa] offset:19648\n\t"
      "v_mfma_f32_16x16x32_bf16 v[80:83], a[0:3], a[56:59], v[80:83]\n\t"
      "ds_read_b128 a[24:27], %[xa] offset:19584\n\t"
      "v_mfma_f32_16x16x32_bf16 v[84:87], a[4:7], a[56:59], v[84:87]\n\t"
      "ds_read_b64_tr_b16 a[0:1], %[wa0] offset:0\n\t"
      "v_mfma_f32_16x16x32_bf16 v[84:87], a[36:39], a[60:63], v[84:87]\n\t"
      "ds_read_b64_tr_b16 a[2:3], %[wa0] offset:1040\n\t"
      "v_mfma_f32_16x16x32_bf16 v[68:71], a[36:39], a[52:55], v[68:71]\n\t"
      "ds_read_b64_tr_b16 a[32:33], %[wa0] offset:8320\n\t"
      "v_mfma_f32_16x16x32_bf16 v[68:71], a[4:7], a[48:51], v[68:71]\n\t"
      "ds_read_b64_tr_b16 a[34:35], %[wa0] offset:9360\n\t"
      "v_mfma_f32_16x16x32_bf16 v[72:75], a[8:11], a[48:51], v[72:75]\n\t"
      "ds_read_b64_tr_b16 a[4:5], %[wa0] offset:32\n\t"
      "v_mfma_f32_16x16x32_bf16 v[72:75], a[40:43], a[52:55], v[72:75]\n\t"
      "ds_read_b64_tr_b16 a[6:7], %[wa0] offset:1072\n\t"
      "v_mfma_f32_16x16x32_bf16 v[88:91], a[40:43], a[60:63], v[88:91]\n\t"
      "ds_read_b64_tr_b16 a[36:37], %[wa0] offset:8352\n\t"
      "v_mfma_f32_16x16x32_bf16 v[88:91], a[8:11], a[56:59], v[88:91]\n\t"
      "ds_read_b64_tr_b16 a[38:39], %[wa0] offset:9392\n\t"
      "v_mfma_f32_16x16x32_bf16 v[92:95], a[12:15], a[56:59], v[92:95]\n\t"
      "ds_read_b64_tr_b16 a[8:9], %[wa1] offset:0\n\t"
      "v_mfma_f32_16x16x32_bf16 v[92:95], a[44:47], a[60:63], v[92:95]\n\t"
      "ds_read_b64_tr_b16 a[10:11], %[wa1] offset:1040\n\t"
      "v_mfma_f32_16x16x32_bf16 v[76:79], a[44:47], a[52:55], v[76:79]\n\t"
      "ds_read_b64_tr_b16 a[40:41], %[wa1] offset:8320\n\t"
      "v_mfma_f32_16x16x32_bf16 v[76:79], a[12:15], a[48:51], v[76:79]\n\t"
      "ds_read_b64_tr_b16 a[42:43], %[wa1] offset:9360\n\t"
      "s_waitcnt lgkmcnt(8)\n\t"
      "s_cbranch_scc1 1b"
      : "+{v[32:47]}"(A[0]), "+{v[48:63]}"(A[1]), "+{v[64:79]}"(A[2]), "+{v[80:95]}"(A[3]), [xa] "+v"(xa), [wa0] "+v"(wa0), [wa1] "+v"(wa1), [roff] "+s"(roff), [cnt] "+s"(cnt), [cv] "=&s"(cv)
      : [xb] "v"(xb), [wb0] "v"(wb0), [wb1] "v"(wb1), [ws] "s"(ws), [oc] "s"(oc)
      : "memory", "scc", ACLOB);
  STAMP(9);
  const unsigned seq = cv + 1u, tag = seq & 1u;
  unsigned mx = 0;
  unsigned long long bad;
  auto await = [&](auto reload) INL {
    if (__builtin_expect(bad != 0, 0)) {
      const unsigned long long t0 = __builtin_amdgcn_s_memrealtime();
      bool ex = false;
      unsigned it = 0;
      do {
        reload();
        if ((++it & 63) == 0) {
          asm volatile("buffer_wbl2 sc1\n\tbuffer_atomic_add %0, %1, %2, %3 offen sc1\n\tbuffer_atomic_add %0, %1, %2, %3 offen offset:2048 sc1\n\t"
                       "buffer_atomic_add %0, %1, %2, %4 offen sc1\n\tbuffer_atomic_add %0, %1, %2, %4 offen offset:2048 sc1\n\ts_waitcnt vmcnt(0)"
                       ::"v"(0), "v"(vxb), "s"(rxs), "s"(s1o[0]), "s"(s1o[1]) : "memory");
          ex = expired((unsigned*)(ws + WS_TMO), seq, t0);
        }
      } while (bad != 0 && !ex);
      if (ex) mx = NAN_BITS;
    }
  };
  auto AWAIT_A = [&]() INL {
    STAMP(10);
    await([&]() INL {
  asm volatile(
      "buffer_load_dwordx4 v[164:167], %[vxb], %[rx], %[s1i0] offen offset:0 sc1\n\t"
      "buffer_load_dwordx4 v[172:175], %[vxb], %[rx], %[s1i0] offen offset:2048 sc1\n\t"
      "buffer_load_dwordx4 v[180:183], %[vxb], %[rx], %[s1i1] offen offset:0 sc1\n\t"
      "buffer_load_dwordx4 v[188:191], %[vxb], %[rx], %[s1i1] offen offset:2048 sc1\n\t"
      "s_waitcnt vmcnt(0)\n\t"
      "v_bitop3_b32 %[c0], v164, v166, v172 bitop3:0x80\n\t"
      "v_bitop3_b32 %[c0], %[c0], v174, v180 bitop3:0x80\n\t"
      "v_bitop3_b32 %[c0], %[c0], v182, v188 bitop3:0x80\n\t"
      "v_bitop3_b32 %[c0], %[c0], v190, 1 bitop3:0x2a\n\t"
      "v_cmp_ne_u32_e64 %[bad], 0, %[c0]"
      : [bad] "=&s"(bad), [c0] "=&v"(c0)
      : [vxb] "v"(vxb), [rx] "s"(rxs), [s1i0] "s"(s1i[0]), [s1i1] "s"(s1i[1])
      : "memory", "scc");
    });
    STAMP(1);
  };
  auto AWAIT_2 = [&]() INL {
    STAMP(11);
    await([&]() INL {
  asm volatile(
      "buffer_load_dwordx4 v[164:167], %[vxb], %[rx], %[s2i0] offen offset:0 sc1\n\t"
      "buffer_load_dwordx4 v[172:175], %[vxb], %[rx], %[s2i0] offen offset:2048 sc1\n\t"
      "buffer_load_dwordx4 v[180:183], %[vxb], %[rx], %[s2i1] offen offset:0 sc1\n\t"
      "buffer_load_dwordx4 v[188:191], %[vxb], %[rx], %[s2i1] offen offset:2048 sc1\n\t"
      "s_waitcnt vmcnt(0)\n\t"
      "v_bitop3_b32 %[c0], v164, v166, %[tag] bitop3:0x7e\n\t"
      "v_bitop3_b32 %[c1], v172, v174, %[tag] bitop3:0x7e\n\t"
      "v_bitop3_b32 %[c2], v180, v182, %[tag] bitop3:0x7e\n\t"
      "v_or3_b32 %[c0], %[c0], %[c1], %[c2]\n\t"
      "v_bitop3_b32 %[c1], v188, v190, %[tag] bitop3:0x7e\n\t"
      "v_bitop3_b32 %[c0], %[c0], %[c1], 1 bitop3:0xa8\n\t"
      "v_cmp_ne_u32_e64 %[bad], 0, %[c0]"
      : [bad] "=&s"(bad), [c0] "=&v"(c0), [c1] "=&v"(c1), [c2] "=&v"(c2)
      : [tag] "s"(tag), [vxb] "v"(vxb), [rx] "s"(rxs), [s2i0] "s"(s2i[0]), [s2i1] "s"(s2i[1])
      : "memory", "scc");
    });
  };
  auto AWAIT_B = [&]() INL {
    STAMP(3);
    await([&]() INL {
  asm volatile(
      "buffer_load_dwordx4 v[160:163], %[vxb], %[rx], %[s1o0] offen offset:0 sc1\n\t"
      "buffer_load_dwordx4 v[168:171], %[vxb], %[rx], %[s1o0] offen offset:2048 sc1\n\t"
      "buffer_load_dwordx4 v[176:179], %[vxb], %[rx], %[s1o1] offen offset:0 sc1\n\t"
      "buffer_load_dwordx4 v[184:187], %[vxb], %[rx], %[s1o1] offen offset:2048 sc1\n\t"
      "s_waitcnt vmcnt(0)\n\t"
      "v_bitop3_b32 %[c0], v160, v162, v168 bitop3:0xfe\n\t"
      "v_bitop3_b32 %[c0], %[c0], v170, v176 bitop3:0xfe\n\t"
      "v_bitop3_b32 %[c0], %[c0], v178, v184 bitop3:0xfe\n\t"
      "v_bitop3_b32 %[c0], %[c0], v186, 1 bitop3:0xa8\n\t"
      "v_cmp_ne_u32_e64 %[bad], 0, %[c0]"
      : [bad] "=&s"(bad), [c0] "=&v"(c0)
      : [vxb] "v"(vxb), [rx] "s"(rxs), [s1o0] "s"(s1o[0]), [s1o1] "s"(s1o[1])
      : "memory", "scc");
    });
    STAMP(12);
  };
  asm volatile(
      "v_mfma_f32_16x16x32_bf16 v[128:131], a[0:3], a[16:19], v[128:131]\n\t"
      "ds_read_b64_tr_b16 a[12:13], %[wa1] offset:32\n\t"
      "ds_read_b64_tr_b16 a[14:15], %[wa1] offset:1072\n\t"
      "v_mfma_f32_16x16x32_bf16 v[128:131], a[32:35], a[20:23], v[128:131]\n\t"
      "ds_read_b64_tr_b16 a[44:45], %[wa1] offset:8352\n\t"
      "ds_read_b64_tr_b16 a[46:47], %[wa1] offset:9392\n\t"
      "v_mfma_f32_16x16x32_bf16 v[144:147], a[32:35], a[28:31], v[144:147]\n\t"
      "ds_read_b128 a[48:51], %[xa] offset:21760\n\t"
      "ds_read_b128 a[52:55], %[xa] offset:21824\n\t"
      "v_mfma_f32_16x16x32_bf16 v[144:147], a[0:3], a[24:27], v[144:147]\n\t"
      "ds_read_b128 a[56:59], %[xa] offset:24000\n\t"
      "s_waitcnt lgkmcnt(11)\n\t"
      "v_mfma_f32_16x16x32_bf16 v[148:151], a[4:7], a[24:27], v[148:151]\n\t"
      "ds_read_b128 a[60:63], %[xa] offset:23936\n\t"
      "v_mfma_f32_16x16x32_bf16 v[148:151], a[36:39], a[28:31], v[148:151]\n\t"
      "v_mfma_f32_16x16x32_bf16 v[132:135], a[36:39], a[20:23], v[132:135]\n\t"
      "v_mfma_f32_16x16x32_bf16 v[132:135], a[4:7], a[16:19], v[132:135]\n\t"
      "s_waitcnt lgkmcnt(8)\n\t"
      "v_mfma_f32_16x16x32_bf16 v[136:139], a[8:11], a[16:19], v[136:139]\n\t"
      "v_mfma_f32_16x16x32_bf16 v[136:139], a[40:43], a[20:23], v[136:139]\n\t"
      "v_mfma_f32_16x16x32_bf16 v[152:155], a[40:43], a[28:31], v[152:155]\n\t"
      "v_mfma_f32_16x16x32_bf16 v[152:155], a[8:11], a[24:27], v[152:155]\n\t"
      "s_waitcnt lgkmcnt(4)\n\t"
      "v_mfma_f32_16x16x32_bf16 v[156:159], a[12:15], a[24:27], v[156:159]\n\t"
      "v_mfma_f32_16x16x32_bf16 v[156:159], a[44:47], a[28:31], v[156:159]\n\t"
      "v_mfma_f32_16x16x32_bf16 v[140:143], a[44:47], a[20:23], v[140:143]\n\t"
      "v_mfma_f32_16x16x32_bf16 v[140:143], a[12:15], a[16:19], v[140:143]\n\t"
      "s_waitcnt lgkmcnt(0)\n\t"
      "v_mfma_f32_16x16x32_bf16 v[160:163], a[0:3], a[48:51], v[160:163]\n\t"
      "v_mfma_f32_16x16x32_bf16 v[160:163], a[32:35], a[52:55], v[160:163]\n\t"
      "v_mfma_f32_16x16x32_bf16 v[176:179], a[32:35], a[56:59], v[176:179]\n\t"
      "s_barrier\n\t"
      "ds_read_b128 a[24:27], %[xa] offset:34816\n\t"
      "v_mfma_f32_16x16x32_bf16 v[176:179], a[0:3], a[60:63], v[176:179]\n\t"
      "ds_read_b128 a[28:31], %[xa] offset:34880\n\t"
      "ds_read_b128 a[20:23], %[xa] offset:37056\n\t"
      "v_mfma_f32_16x16x32_bf16 v[180:183], a[4:7], a[60:63], v[180:183]\n\t"
      "ds_read_b128 a[16:19], %[xa] offset:36992\n\t"
      "v_mfma_f32_16x16x32_bf16 v[180:183], a[36:39], a[56:59], v[180:183]\n\t"
      "v_mfma_f32_16x16x32_bf16 v[164:167], a[36:39], a[52:55], v[164:167]\n\t"
      "v_mfma_f32_16x16x32_bf16 v[164:167], a[4:7], a[48:51], v[164:167]\n\t"
      "v_mfma_f32_16x16x32_bf16 v[168:171], a[8:11], a[48:51], v[168:171]\n\t"
      "v_mfma_f32_16x16x32_bf16 v[168:171], a[40:43], a[52:55], v[168:171]\n\t"
      "v_mfma_f32_16x16x32_bf16 v[184:187], a[40:43], a[56:59], v[184:187]\n\t"
      "v_mfma_f32_16x16x32_bf16 v[184:187], a[8:11], a[60:63], v[184:187]\n\t"
      "v_mfma_f32_16x16x32_bf16 v[188:191], a[12:15], a[60:63], v[188:191]\n\t"
      "v_mfma_f32_16x16x32_bf16 v[188:191], a[44:47], a[56:59], v[188:191]\n\t"
      "v_mfma_f32_16x16x32_bf16 v[172:175], a[44:47], a[52:55], v[172:175]\n\t"
      "v_mfma_f32_16x16x32_bf16 v[172:175], a[12:15], a[48:51], v[172:175]\n\t"
      "s_waitcnt lgkmcnt(0)\n\t"
      "v_mfma_f32_16x16x32_bf16 v[32:35], a[0:3], a[24:27], v[32:35]\n\t"
      "v_mfma_f32_16x16x32_bf16 v[32:35], a[32:35], a[28:31], v[32:35]\n\t"
      "v_mfma_f32_16x16x32_bf16 v[48:51], a[32:35], a[20:23], v[48:51]\n\t"
      "ds_read_b128 a[60:63], %[xa] offset:39168\n\t"
      "ds_read_b128 a[56:59], %[xa] offset:39232\n\t"
      "v_mfma_f32_16x16x32_bf16 v[48:51], a[0:3], a[16:19], v[48:51]\n\t"
      "ds_read_b128 a[52:55], %[xa] offset:41408\n\t"
      "ds_read_b128 a[48:51], %[xa] offset:41344\n\t"
      "v_mfma_f32_16x16x32_bf16 v[52:55], a[4:7], a[16:19], v[52:55]\n\t"
      "v_mfma_f32_16x16x32_bf16 v[52:55], a[36:39], a[20:23], v[52:55]\n\t"
      "v_mfma_f32_16x16x32_bf16 v[36:39], a[36:39], a[28:31], v[36:39]\n\t"
      "v_mfma_f32_16x16x32_bf16 v[36:39], a[4:7], a[24:27], v[36:39]\n\t"
      "v_mfma_f32_16x16x32_bf16 v[40:43], a[8:11], a[24:27], v[40:43]\n\t"
      "v_mfma_f32_16x16x32_bf16 v[40:43], a[40:43], a[28:31], v[40:43]\n\t"
      "v_mfma_f32_16x16x32_bf16 v[56:59], a[40:43], a[20:23], v[56:59]\n\t"
      "v_mfma_f32_16x16x32_bf16 v[56:59], a[8:11], a[16:19], v[56:59]\n\t"
      "v_mfma_f32_16x16x32_bf16 v[60:63], a[12:15], a[16:19], v[60:63]\n\t"
      "v_mfma_f32_16x16x32_bf16 v[60:63], a[44:47], a[20:23], v[60:63]\n\t"
      "v_mfma_f32_16x16x32_bf16 v[44:47], a[44:47], a[28:31], v[44:47]\n\t"
      "v_mfma_f32_16x16x32_bf16 v[44:47], a[12:15], a[24:27], v[44:47]\n\t"
      "s_waitcnt lgkmcnt(0)\n\t"
      "v_mfma_f32_16x16x32_bf16 v[64:67], a[0:3], a[60:63], v[64:67]\n\t"
      "v_mfma_f32_16x16x32_bf16 v[64:67], a[32:35], a[56:59], v[64:67]\n\t"
      "v_mfma_f32_16x16x32_bf16 v[80:83], a[32:35], a[52:55], v[80:83]\n\t"
      "s_barrier\n\t"
      "v_add_u32 %[xa], 121856, %[xb]\n\t"
      "v_mfma_f32_16x16x32_bf16 v[80:83], a[0:3], a[48:51], v[80:83]\n\t"
      "ds_read_b128 a[16:19], %[xa] offset:0\n\t"
      "ds_read_b64_tr_b16 a[20:21], %[wa0] offset:52224\n\t"
      "v_mfma_f32_16x16x32_bf16 v[84:87], a[4:7], a[48:51], v[84:87]\n\t"
      "ds_read_b64_tr_b16 a[22:23], %[wa0] offset:53264\n\t"
      "ds_read_b128 a[28:31], %[xa] offset:64\n\t"
      "v_mfma_f32_16x16x32_bf16 v[84:87], a[36:39], a[52:55], v[84:87]\n\t"
      "ds_read_b64_tr_b16 a[24:25], %[wa0] offset:60544\n\t"
      "ds_read_b64_tr_b16 a[26:27], %[wa0] offset:61584\n\t"
      "v_mfma_f32_16x16x32_bf16 v[68:71], a[36:39], a[56:59], v[68:71]\n\t"
      "ds_read_b128 a[32:35], %[xa] offset:2240\n\t"
      "ds_read_b128 a[0:3], %[xa] offset:2176\n\t"
      "v_mfma_f32_16x16x32_bf16 v[68:71], a[4:7], a[60:63], v[68:71]\n\t"
      "ds_read_b64_tr_b16 a[36:37], %[wa0] offset:52256\n\t"
      "ds_read_b64_tr_b16 a[38:39], %[wa0] offset:53296\n\t"
      "v_mfma_f32_16x16x32_bf16 v[72:75], a[8:11], a[60:63], v[72:75]\n\t"
      "ds_read_b64_tr_b16 a[4:5], %[wa0] offset:60576\n\t"
      "ds_read_b64_tr_b16 a[6:7], %[wa0] offset:61616\n\t"
      "v_mfma_f32_16x16x32_bf16 v[72:75], a[40:43], a[56:59], v[72:75]\n\t"
      "v_mfma_f32_16x16x32_bf16 v[88:91], a[40:43], a[52:55], v[88:91]\n\t"
      "ds_read_b64_tr_b16 a[40:41], %[wa1] offset:52224\n\t"
      "ds_read_b64_tr_b16 a[42:43], %[wa1] offset:53264\n\t"
      "v_mfma_f32_16x16x32_bf16 v[88:91], a[8:11], a[48:51], v[88:91]\n\t"
      "ds_read_b64_tr_b16 a[8:9], %[wa1] offset:60544\n\t"
      "ds_read_b64_tr_b16 a[10:11], %[wa1] offset:61584\n\t"
      "v_mfma_f32_16x16x32_bf16 v[92:95], a[12:15], a[48:51], v[92:95]\n\t"
      "v_mfma_f32_16x16x32_bf16 v[92:95], a[44:47], a[52:55], v[92:95]\n\t"
      "v_mfma_f32_16x16x32_bf16 v[76:79], a[44:47], a[56:59], v[76:79]\n\t"
      "ds_read_b64_tr_b16 a[48:49], %[wa1] offset:52256\n\t"
      "ds_read_b64_tr_b16 a[50:51], %[wa1] offset:53296\n\t"
      "v_mfma_f32_16x16x32_bf16 v[76:79], a[12:15], a[60:63], v[76:79]\n\t"
      "ds_read_b64_tr_b16 a[52:53], %[wa1] offset:60576\n\t"
      "s_waitcnt lgkmcnt(11)\n\t"
      "v_mfma_f32_16x16x32_bf16 v[160:163], a[20:23], a[16:19], v[160:163]\n\t"
      "ds_read_b64_tr_b16 a[54:55], %[wa1] offset:61616\n\t"
      "v_mfma_f32_16x16x32_bf16 v[160:163], a[24:27], a[28:31], v[160:163]\n\t"
      "v_mfma_f32_16x16x32_bf16 v[176:179], a[24:27], a[32:35], v[176:179]\n\t"
      "s_barrier\n\t"
      "ds_read_b128 a[44:47], %[xa] offset:4352\n\t"
      "v_mfma_f32_16x16x32_bf16 v[176:179], a[20:23], a[0:3], v[176:179]\n\t"
      "ds_read_b128 a[56:59], %[xa] offset:4416\n\t"
      "s_waitcnt lgkmcnt(10)\n\t"
      "v_mfma_f32_16x16x32_bf16 v[180:183], a[36:39], a[0:3], v[180:183]\n\t"
      "ds_read_b128 a[12:15], %[xa] offset:6592\n\t"
      "ds_read_b128 a[60:63], %[xa] offset:6528\n\t"
      "v_mfma_f32_16x16x32_bf16 v[180:183], a[4:7], a[32:35], v[180:183]\n\t"
      "v_mfma_f32_16x16x32_bf16 v[164:167], a[4:7], a[28:31], v[164:167]\n\t"
      "v_mfma_f32_16x16x32_bf16 v[164:167], a[36:39], a[16:19], v[164:167]\n\t"
      "s_waitcnt lgkmcnt(8)\n\t"
      "v_mfma_f32_16x16x32_bf16 v[168:171], a[40:43], a[16:19], v[168:171]\n\t"
      "v_mfma_f32_16x16x32_bf16 v[168:171], a[8:11], a[28:31], v[168:171]\n\t"
      "v_mfma_f32_16x16x32_bf16 v[184:187], a[8:11], a[32:35], v[184:187]\n\t"
      "v_mfma_f32_16x16x32_bf16 v[184:187], a[40:43], a[0:3], v[184:187]\n\t"
      "s_waitcnt lgkmcnt(4)\n\t"
      "v_mfma_f32_16x16x32_bf16 v[188:191], a[48:51], a[0:3], v[188:191]\n\t"
      "v_mfma_f32_16x16x32_bf16 v[188:191], a[52:55], a[32:35], v[188:191]\n\t"
      "v_mfma_f32_16x16x32_bf16 v[172:175], a[52:55], a[28:31], v[172:175]\n\t"
      "v_mfma_f32_16x16x32_bf16 v[172:175], a[48:51], a[16:19], v[172:175]\n\t"
      "s_waitcnt lgkmcnt(0)\n\t"
      "v_mfma_f32_16x16x32_bf16 v[128:131], a[20:23], a[44:47], v[128:131]\n\t"
      "v_mfma_f32_16x16x32_bf16 v[128:131], a[24:27], a[56:59], v[128:131]\n\t"
      "v_mfma_f32_16x16x32_bf16 v[144:147], a[24:27], a[12:15], v[144:147]\n\t"
      "s_barrier\n\t"
      "ds_read_b128 a[0:3], %[xa] offset:17408\n\t"
      "v_mfma_f32_16x16x32_bf16 v[144:147], a[20:23], a[60:63], v[144:147]\n\t"
      "ds_read_b128 a[32:35], %[xa] offset:17472\n\t"
      "ds_read_b128 a[28:31], %[xa] offset:19648\n\t"
      "v_mfma_f32_16x16x32_bf16 v[148:151], a[36:39], a[60:63], v[148:151]\n\t"
      "ds_read_b128 a[16:19], %[xa] offset:19584\n\t"
      "v_mfma_f32_16x16x32_bf16 v[148:151], a[4:7], a[12:15], v[148:151]\n\t"
      "v_mfma_f32_16x16x32_bf16 v[132:135], a[4:7], a[56:59], v[132:135]\n\t"
      "v_mfma_f32_16x16x32_bf16 v[132:135], a[36:39], a[44:47], v[132:135]\n\t"
      "v_mfma_f32_16x16x32_bf16 v[136:139], a[40:43], a[44:47], v[136:139]\n\t"
      "v_mfma_f32_16x16x32_bf16 v[136:139], a[8:11], a[56:59], v[136:139]\n\t"
      "v_mfma_f32_16x16x32_bf16 v[152:155], a[8:11], a[12:15], v[152:155]\n\t"
      "v_mfma_f32_16x16x32_bf16 v[152:155], a[40:43], a[60:63], v[152:155]\n\t"
      "v_mfma_f32_16x16x32_bf16 v[156:159], a[48:51], a[60:63], v[156:159]\n\t"
      "v_mfma_f32_16x16x32_bf16 v[156:159], a[52:55], a[12:15], v[156:159]\n\t"
      "v_mfma_f32_16x16x32_bf16 v[140:143], a[52:55], a[56:59], v[140:143]\n\t"
      "v_mfma_f32_16x16x32_bf16 v[140:143], a[48:51], a[44:47], v[140:143]\n\t"
      "s_waitcnt lgkmcnt(0)\n\t"
      "v_mfma_f32_16x16x32_bf16 v[64:67], a[20:23], a[0:3], v[64:67]\n\t"
      "v_mfma_f32_16x16x32_bf16 v[64:67], a[24:27], a[32:35], v[64:67]\n\t"
      "v_mfma_f32_16x16x32_bf16 v[80:83], a[24:27], a[28:31], v[80:83]\n\t"
      "s_barrier\n\t"
      "ds_read_b128 a[60:63], %[xa] offset:21760\n\t"
      "v_mfma_f32_16x16x32_bf16 v[80:83], a[20:23], a[16:19], v[80:83]\n\t"
      "v_add_u32_e64 %[wa0], 0, %[wb0]\n\t"
      "v_add_u32_e64 %[wa1], 0, %[wb1]\n\t"
      "v_mfma_f32_16x16x32_bf16 v[84:87], a[36:39], a[16:19], v[84:87]\n\t"
      "ds_read_b64_tr_b16 a[12:13], %[wa0] offset:0\n\t"
      "ds_read_b64_tr_b16 a[14:15], %[wa0] offset:1040\n\t"
      "v_mfma_f32_16x16x32_bf16 v[84:87], a[4:7], a[28:31], v[84:87]\n\t"
      "ds_read_b128 a[56:59], %[xa] offset:21824\n\t"
      "ds_read_b64_tr_b16 a[44:45], %[wa0] offset:8320\n\t"
      "v_mfma_f32_16x16x32_bf16 v[68:71], a[4:7], a[32:35], v[68:71]\n\t"
      "ds_read_b64_tr_b16 a[46:47], %[wa0] offset:9360\n\t"
      "ds_read_b128 a[24:27], %[xa] offset:24000\n\t"
      "v_mfma_f32_16x16x32_bf16 v[68:71], a[36:39], a[0:3], v[68:71]\n\t"
      "ds_read_b128 a[20:23], %[xa] offset:23936\n\t"
      "ds_read_b64_tr_b16 a[4:5], %[wa0] offset:32\n\t"
      "v_mfma_f32_16x16x32_bf16 v[72:75], a[40:43], a[0:3], v[72:75]\n\t"
      "ds_read_b64_tr_b16 a[6:7], %[wa0] offset:1072\n\t"
      "ds_read_b64_tr_b16 a[36:37], %[wa0] offset:8352\n\t"
      "v_mfma_f32_16x16x32_bf16 v[72:75], a[8:11], a[32:35], v[72:75]\n\t"
      "ds_read_b64_tr_b16 a[38:39], %[wa0] offset:9392\n\t"
      "v_mfma_f32_16x16x32_bf16 v[88:91], a[8:11], a[28:31], v[88:91]\n\t"
      "ds_read_b64_tr_b16 a[8:9], %[wa1] offset:0\n\t"
      "ds_read_b64_tr_b16 a[10:11], %[wa1] offset:1040\n\t"
      "v_mfma_f32_16x16x32_bf16 v[88:91], a[40:43], a[16:19], v[88:91]\n\t"
      "ds_read_b64_tr_b16 a[40:41], %[wa1] offset:8320\n\t"
      "ds_read_b64_tr_b16 a[42:43], %[wa1] offset:9360\n\t"
      "v_mfma_f32_16x16x32_bf16 v[92:95], a[48:51], a[16:19], v[92:95]\n\t"
      "v_mfma_f32_16x16x32_bf16 v[92:95], a[52:55], a[28:31], v[92:95]\n\t"
      "buffer_load_dword %[c1], %[vpf], %[rx], %[s1o0] offen\n\t"
      "v_mfma_f32_16x16x32_bf16 v[76:79], a[52:55], a[32:35], v[76:79]\n\t"
      "ds_read_b64_tr_b16 a[16:17], %[wa1] offset:32\n\t"
      "ds_read_b64_tr_b16 a[18:19], %[wa1] offset:1072\n\t"
      "v_mfma_f32_16x16x32_bf16 v[76:79], a[48:51], a[0:3], v[76:79]\n\t"
      "ds_read_b64_tr_b16 a[28:29], %[wa1] offset:8352\n\t"
      "s_waitcnt lgkmcnt(11)\n\t"
      "v_mfma_f32_16x16x32_bf16 v[160:163], a[12:15], a[60:63], v[160:163]\n\t"
      "ds_read_b64_tr_b16 a[30:31], %[wa1] offset:9392\n\t"
      "v_mfma_f32_16x16x32_bf16 v[160:163], a[44:47], a[56:59], v[160:163]\n\t"
      "v_mfma_f32_16x16x32_bf16 v[176:179], a[44:47], a[24:27], v[176:179]\n\t"
      "s_barrier\n\t"
      "v_add_u32 %[xa], 17408, %[xb]\n\t"
      "v_mfma_f32_16x16x32_bf16 v[176:179], a[12:15], a[20:23], v[176:179]\n\t"
      "ds_read_b128 a[52:55], %[xa] offset:0\n\t"
      "s_waitcnt lgkmcnt(9)\n\t"
      "v_mfma_f32_16x16x32_bf16 v[180:183], a[4:7], a[20:23], v[180:183]\n\t"
      "ds_read_b128 a[32:35], %[xa] offset:64\n\t"
      "ds_read_b128 a[48:51], %[xa] offset:2240\n\t"
      "v_mfma_f32_16x16x32_bf16 v[180:183], a[36:39], a[24:27], v[180:183]\n\t"
      "ds_read_b128 a[0:3], %[xa] offset:2176\n\t"
      "v_mfma_f32_16x16x32_bf16 v[164:167], a[36:39], a[56:59], v[164:167]\n\t"
      "v_mfma_f32_16x16x32_bf16 v[164:167], a[4:7], a[60:63], v[164:167]\n\t"
      "s_waitcnt lgkmcnt(8)\n\t"
      "v_mfma_f32_16x16x32_bf16 v[168:171], a[8:11], a[60:63], v[168:171]\n\t"
      "v_mfma_f32_16x16x32_bf16 v[168:171], a[40:43], a[56:59], v[168:171]\n\t"
      "v_mfma_f32_16x16x32_bf16 v[184:187], a[40:43], a[24:27], v[184:187]\n\t"
      "v_mfma_f32_16x16x32_bf16 v[184:187], a[8:11], a[20:23], v[184:187]\n\t"
      "s_waitcnt lgkmcnt(4)\n\t"
      "v_mfma_f32_16x16x32_bf16 v[188:191], a[16:19], a[20:23], v[188:191]\n\t"
      "v_mfma_f32_16x16x32_bf16 v[188:191], a[28:31], a[24:27], v[188:191]\n\t"
      "v_mfma_f32_16x16x32_bf16 v[172:175], a[28:31], a[56:59], v[172:175]\n\t"
      "v_mfma_f32_16x16x32_bf16 v[172:175], a[16:19], a[60:63], v[172:175]\n\t"
      "s_waitcnt lgkmcnt(0)\n\t"
      "v_mfma_f32_16x16x32_bf16 v[128:131], a[12:15], a[52:55], v[128:131]\n\t"
      "v_mfma_f32_16x16x32_bf16 v[128:131], a[44:47], a[32:35], v[128:131]\n\t"
      "v_mfma_f32_16x16x32_bf16 v[144:147], a[44:47], a[48:51], v[144:147]\n\t"
      "s_barrier\n\t"
      "ds_read_b128 a[20:23], %[xa] offset:4352\n\t"
      "v_mfma_f32_16x16x32_bf16 v[144:147], a[12:15], a[0:3], v[144:147]\n\t"
      "ds_read_b128 a[24:27], %[xa] offset:4416\n\t"
      "ds_read_b128 a[56:59], %[xa] offset:6592\n\t"
      "v_mfma_f32_16x16x32_bf16 v[148:151], a[4:7], a[0:3], v[148:151]\n\t"
      "ds_read_b128 a[60:63], %[xa] offset:6528\n\t"
      "v_mfma_f32_16x16x32_bf16 v[148:151], a[36:39], a[48:51], v[148:151]\n\t"
      "v_mfma_f32_16x16x32_bf16 v[132:135], a[36:39], a[32:35], v[132:135]\n\t"
      "v_mfma_f32_16x16x32_bf16 v[132:135], a[4:7], a[52:55], v[132:135]\n\t"
      "v_mfma_f32_16x16x32_bf16 v[136:139], a[8:11], a[52:55], v[136:139]\n\t"
      "v_mfma_f32_16x16x32_bf16 v[136:139], a[40:43], a[32:35], v[136:139]\n\t"
      "v_mfma_f32_16x16x32_bf16 v[152:155], a[40:43], a[48:51], v[152:155]\n\t"
      "v_mfma_f32_16x16x32_bf16 v[152:155], a[8:11], a[0:3], v[152:155]\n\t"
      "v_mfma_f32_16x16x32_bf16 v[156:159], a[16:19], a[0:3], v[156:159]\n\t"
      "v_mfma_f32_16x16x32_bf16 v[156:159], a[28:31], a[48:51], v[156:159]\n\t"
      "v_mfma_f32_16x16x32_bf16 v[140:143], a[28:31], a[32:35], v[140:143]\n\t"
      "v_mfma_f32_16x16x32_bf16 v[140:143], a[16:19], a[52:55], v[140:143]\n\t"
      "s_waitcnt lgkmcnt(0)\n\t"
      "v_mfma_f32_16x16x32_bf16 v[64:67], a[12:15], a[20:23], v[64:67]\n\t"
      "v_mfma_f32_16x16x32_bf16 v[64:67], a[44:47], a[24:27], v[64:67]\n\t"
      "v_mfma_f32_16x16x32_bf16 v[80:83], a[44:47], a[56:59], v[80:83]\n\t"
      "s_barrier\n\t"
      "ds_read_b128 a[0:3], %[xa] offset:34816\n\t"
      "v_mfma_f32_16x16x32_bf16 v[80:83], a[12:15], a[60:63], v[80:83]\n\t"
      "ds_read_b64_tr_b16 a[48:49], %[wa0] offset:34816\n\t"
      "ds_read_b64_tr_b16 a[50:51], %[wa0] offset:35856\n\t"
      "v_mfma_f32_16x16x32_bf16 v[84:87], a[4:7], a[60:63], v[84:87]\n\t"
      "ds_read_b128 a[32:35], %[xa] offset:34880\n\t"
      "ds_read_b64_tr_b16 a[52:53], %[wa0] offset:43136\n\t"
      "v_mfma_f32_16x16x32_bf16 v[84:87], a[36:39], a[56:59], v[84:87]\n\t"
      "ds_read_b64_tr_b16 a[54:55], %[wa0] offset:44176\n\t"
      "ds_read_b128 a[44:47], %[xa] offset:37056\n\t"
      "v_mfma_f32_16x16x32_bf16 v[68:71], a[36:39], a[24:27], v[68:71]\n\t"
      "ds_read_b128 a[12:15], %[xa] offset:36992\n\t"
      "ds_read_b64_tr_b16 a[36:37], %[wa0] offset:34848\n\t"
      "v_mfma_f32_16x16x32_bf16 v[68:71], a[4:7], a[20:23], v[68:71]\n\t"
      "ds_read_b64_tr_b16 a[38:39], %[wa0] offset:35888\n\t"
      "ds_read_b64_tr_b16 a[4:5], %[wa0] offset:43168\n\t"
      "v_mfma_f32_16x16x32_bf16 v[72:75], a[8:11], a[20:23], v[72:75]\n\t"
      "ds_read_b64_tr_b16 a[6:7], %[wa0] offset:44208\n\t"
      "v_mfma_f32_16x16x32_bf16 v[72:75], a[40:43], a[24:27], v[72:75]\n\t"
      "v_mfma_f32_16x16x32_bf16 v[88:91], a[40:43], a[56:59], v[88:91]\n\t"
      "ds_read_b64_tr_b16 a[40:41], %[wa1] offset:34816\n\t"
      "ds_read_b64_tr_b16 a[42:43], %[wa1] offset:35856\n\t"
      "v_mfma_f32_16x16x32_bf16 v[88:91], a[8:11], a[60:63], v[88:91]\n\t"
      "ds_read_b64_tr_b16 a[8:9], %[wa1] offset:43136\n\t"
      "ds_read_b64_tr_b16 a[10:11], %[wa1] offset:44176\n\t"
      "v_mfma_f32_16x16x32_bf16 v[92:95], a[16:19], a[60:63], v[92:95]\n\t"
      "v_mfma_f32_16x16x32_bf16 v[92:95], a[28:31], a[56:59], v[92:95]\n\t"
      "v_mfma_f32_16x16x32_bf16 v[76:79], a[28:31], a[24:27], v[76:79]\n\t"
      "ds_read_b64_tr_b16 a[60:61], %[wa1] offset:34848\n\t"
      "ds_read_b64_tr_b16 a[62:63], %[wa1] offset:35888\n\t"
      "v_mfma_f32_16x16x32_bf16 v[76:79], a[16:19], a[20:23], v[76:79]\n\t"
      "ds_read_b64_tr_b16 a[56:57], %[wa1] offset:43168\n\t"
      "s_waitcnt lgkmcnt(11)\n\t"
      "v_mfma_f32_16x16x32_bf16 v[160:163], a[48:51], a[0:3], v[160:163]\n\t"
      "ds_read_b64_tr_b16 a[58:59], %[wa1] offset:44208\n\t"
      "v_mfma_f32_16x16x32_bf16 v[160:163], a[52:55], a[32:35], v[160:163]\n\t"
      "v_mfma_f32_16x16x32_bf16 v[176:179], a[52:55], a[44:47], v[176:179]\n\t"
      "s_barrier\n\t"
      "ds_read_b128 a[28:31], %[xa] offset:39168\n\t"
      "v_mfma_f32_16x16x32_bf16 v[176:179], a[48:51], a[12:15], v[176:179]\n\t"
      "v_add_u32 %[wa0], 69632, %[wb0]\n\t"
      "s_waitcnt lgkmcnt(9)\n\t"
      "v_mfma_f32_16x16x32_bf16 v[180:183], a[36:39], a[12:15], v[180:183]\n\t"
      "v_add_u32 %[wa1], 69632, %[wb1]\n\t"
      "ds_read_b64_tr_b16 a[24:25], %[wa0] offset:0\n\t"
      "v_mfma_f32_16x16x32_bf16 v[180:183], a[4:7], a[44:47], v[180:183]\n\t"
      "ds_read_b64_tr_b16 a[26:27], %[wa0] offset:1040\n\t"
      "ds_read_b128 a[16:19], %[xa] offset:39232\n\t"
      "v_mfma_f32_16x16x32_bf16 v[164:167], a[4:7], a[32:35], v[164:167]\n\t"
      "ds_read_b64_tr_b16 a[20:21], %[wa0] offset:8320\n\t"
      "ds_read_b64_tr_b16 a[22:23], %[wa0] offset:9360\n\t"
      "v_mfma_f32_16x16x32_bf16 v[164:167], a[36:39], a[0:3], v[164:167]\n\t"
      "ds_read_b128 a[52:55], %[xa] offset:41408\n\t"
      "s_waitcnt lgkmcnt(11)\n\t"
      "v_mfma_f32_16x16x32_bf16 v[168:171], a[40:43], a[0:3], v[168:171]\n\t"
      "ds_read_b128 a[48:51], %[xa] offset:41344\n\t"
      "ds_read_b64_tr_b16 a[4:5], %[wa0] offset:32\n\t"
      "v_mfma_f32_16x16x32_bf16 v[168:171], a[8:11], a[32:35], v[168:171]\n\t"
      "ds_read_b64_tr_b16 a[6:7], %[wa0] offset:1072\n\t"
      "ds_read_b64_tr_b16 a[36:37], %[wa0] offset:8352\n\t"
      "v_mfma_f32_16x16x32_bf16 v[184:187], a[8:11], a[44:47], v[184:187]\n\t"
      "ds_read_b64_tr_b16 a[38:39], %[wa0] offset:9392\n\t"
      "ds_read_b64_tr_b16 a[8:9], %[wa1] offset:0\n\t"
      "v_mfma_f32_16x16x32_bf16 v[184:187], a[40:43], a[12:15], v[184:187]\n\t"
      "ds_read_b64_tr_b16 a[10:11], %[wa1] offset:1040\n\t"
      "s_waitcnt lgkmcnt(14)\n\t"
      "v_mfma_f32_16x16x32_bf16 v[188:191], a[60:63], a[12:15], v[188:191]\n\t"
      "ds_read_b64_tr_b16 a[40:41], %[wa1] offset:8320\n\t"
      "ds_read_b64_tr_b16 a[42:43], %[wa1] offset:9360\n\t"
      "v_mfma_f32_16x16x32_bf16 v[188:191], a[56:59], a[44:47], v[188:191]\n\t"
      "v_mfma_f32_16x16x32_bf16 v[172:175], a[56:59], a[32:35], v[172:175]\n\t"
      "ds_read_b64_tr_b16 a[12:13], %[wa1] offset:32\n\t"
      "ds_read_b64_tr_b16 a[14:15], %[wa1] offset:1072\n\t"
      "v_mfma_f32_16x16x32_bf16 v[172:175], a[60:63], a[0:3], v[172:175]\n\t"
      "ds_read_b64_tr_b16 a[44:45], %[wa1] offset:8352\n\t"
      "s_waitcnt lgkmcnt(11)\n\t"
      "v_mfma_f32_16x16x32_bf16 v[160:163], a[24:27], a[28:31], v[160:163]\n\t"
      "ds_read_b64_tr_b16 a[46:47], %[wa1] offset:9392\n\t"
      "v_mfma_f32_16x16x32_bf16 v[160:163], a[20:23], a[16:19], v[160:163]\n\t"
      "v_mfma_f32_16x16x32_bf16 v[176:179], a[20:23], a[52:55], v[176:179]\n\t"
      "s_barrier\n\t"
      "v_add_u32 %[xa], 121856, %[xb]\n\t"
      "v_mfma_f32_16x16x32_bf16 v[176:179], a[24:27], a[48:51], v[176:179]\n\t"
      "ds_read_b128 a[56:59], %[xa] offset:0\n\t"
      "s_waitcnt lgkmcnt(9)\n\t"
      "v_mfma_f32_16x16x32_bf16 v[180:183], a[4:7], a[48:51], v[180:183]\n\t"
      "ds_read_b64_tr_b16 a[32:33], %[wa0] offset:17408\n\t"
      "ds_read_b64_tr_b16 a[34:35], %[wa0] offset:18448\n\t"
      "v_mfma_f32_16x16x32_bf16 v[180:183], a[36:39], a[52:55], v[180:183]\n\t"
      "ds_read_b128 a[60:63], %[xa] offset:64\n\t"
      "ds_read_b64_tr_b16 a[0:1], %[wa0] offset:25728\n\t"
      "v_mfma_f32_16x16x32_bf16 v[164:167], a[36:39], a[16:19], v[164:167]\n\t"
      "ds_read_b64_tr_b16 a[2:3], %[wa0] offset:26768\n\t"
      "ds_read_b64_tr_b16 a[20:21], %[wa0] offset:25760\n\t"
      "v_mfma_f32_16x16x32_bf16 v[164:167], a[4:7], a[28:31], v[164:167]\n\t"
      "ds_read_b64_tr_b16 a[22:23], %[wa0] offset:26800\n\t"
      "s_waitcnt lgkmcnt(12)\n\t"
      "v_mfma_f32_16x16x32_bf16 v[168:171], a[8:11], a[28:31], v[168:171]\n\t"
      "ds_read_b64_tr_b16 a[24:25], %[wa0] offset:17440\n\t"
      "ds_read_b64_tr_b16 a[26:27], %[wa0] offset:18480\n\t"
      "v_mfma_f32_16x16x32_bf16 v[168:171], a[40:43], a[16:19], v[168:171]\n\t"
      "ds_read_b64_tr_b16 a[36:37], %[wa1] offset:17408\n\t"
      "ds_read_b64_tr_b16 a[38:39], %[wa1] offset:18448\n\t"
      "v_mfma_f32_16x16x32_bf16 v[184:187], a[40:43], a[52:55], v[184:187]\n\t"
      "ds_read_b64_tr_b16 a[4:5], %[wa1] offset:25728\n\t"
      "ds_read_b64_tr_b16 a[6:7], %[wa1] offset:26768\n\t"
      "v_mfma_f32_16x16x32_bf16 v[184:187], a[8:11], a[48:51], v[184:187]\n\t"
      "ds_read_b64_tr_b16 a[40:41], %[wa1] offset:25760\n\t"
      "s_waitcnt lgkmcnt(14)\n\t"
      "v_mfma_f32_16x16x32_bf16 v[188:191], a[12:15], a[48:51], v[188:191]\n\t"
      "ds_read_b64_tr_b16 a[42:43], %[wa1] offset:26800\n\t"
      "ds_read_b64_tr_b16 a[8:9], %[wa1] offset:17440\n\t"
      "v_mfma_f32_16x16x32_bf16 v[188:191], a[44:47], a[52:55], v[188:191]\n\t"
      "ds_read_b64_tr_b16 a[10:11], %[wa1] offset:18480\n\t"
      "ds_read_b128 a[48:51], %[xa] offset:2176\n\t"
      "v_mfma_f32_16x16x32_bf16 v[172:175], a[44:47], a[16:19], v[172:175]\n\t"
      "ds_read_b128 a[52:55], %[xa] offset:2240\n\t"
      "v_mfma_f32_16x16x32_bf16 v[172:175], a[12:15], a[28:31], v[172:175]\n\t"
      "s_waitcnt lgkmcnt(10)\n\t"
      "v_mfma_f32_16x16x32_bf16 v[160:163], a[32:35], a[56:59], v[160:163]\n\t"
      "v_mfma_f32_16x16x32_bf16 v[160:163], a[0:3], a[60:63], v[160:163]\n\t"
      "v_mfma_f32_16x16x32_bf16 v[164:167], a[20:23], a[60:63], v[164:167]\n\t"
      "s_barrier\n\t"
      "ds_read_b128 a[44:47], %[xa] offset:4352\n\t"
      "v_mfma_f32_16x16x32_bf16 v[164:167], a[24:27], a[56:59], v[164:167]\n\t"
      "ds_read_b128 a[16:19], %[xa] offset:4416\n\t"
      "s_waitcnt lgkmcnt(4)\n\t"
      "v_mfma_f32_16x16x32_bf16 v[168:171], a[36:39], a[56:59], v[168:171]\n\t"
      "ds_read_b128 a[12:15], %[xa] offset:6592\n\t"
      "ds_read_b128 a[28:31], %[xa] offset:6528\n\t"
      "v_mfma_f32_16x16x32_bf16 v[168:171], a[4:7], a[60:63], v[168:171]\n\t"
      "v_mfma_f32_16x16x32_bf16 v[172:175], a[40:43], a[60:63], v[172:175]\n\t"
      "v_mfma_f32_16x16x32_bf16 v[172:175], a[8:11], a[56:59], v[172:175]\n\t"
      "v_cvt_pk_bf16_f32 v160, v160, v161\n\t"
      "s_waitcnt lgkmcnt(4)\n\t"
      "v_mfma_f32_16x16x32_bf16 v[188:191], a[8:11], a[48:51], v[188:191]\n\t"
      "v_cvt_pk_bf16_f32 v161, v162, v163\n\t"
      "v_cvt_pk_bf16_f32 v162, v164, v165\n\t"
      "v_mfma_f32_16x16x32_bf16 v[188:191], a[40:43], a[52:55], v[188:191]\n\t"
      "v_or_b32 v160, 1, v160\n\t"
      "v_cvt_pk_bf16_f32 v163, v166, v167\n\t"
      "v_mfma_f32_16x16x32_bf16 v[184:187], a[4:7], a[52:55], v[184:187]\n\t"
      "v_or_b32 v162, 1, v162\n\t"
      "buffer_store_dwordx4 v[160:163], %[vxb], %[rx], %[s1o0] offen offset:0\n\t"
      "v_mfma_f32_16x16x32_bf16 v[184:187], a[36:39], a[48:51], v[184:187]\n\t"
      "v_cvt_pk_bf16_f32 v168, v168, v169\n\t"
      "v_cvt_pk_bf16_f32 v169, v170, v171\n\t"
      "v_mfma_f32_16x16x32_bf16 v[180:183], a[24:27], a[48:51], v[180:183]\n\t"
      "v_cvt_pk_bf16_f32 v170, v172, v173\n\t"
      "v_or_b32 v168, 1, v168\n\t"
      "v_mfma_f32_16x16x32_bf16 v[180:183], a[20:23], a[52:55], v[180:183]\n\t"
      "v_cvt_pk_bf16_f32 v171, v174, v175\n\t"
      "v_or_b32 v170, 1, v170\n\t"
      "v_mfma_f32_16x16x32_bf16 v[176:179], a[0:3], a[52:55], v[176:179]\n\t"
      "buffer_store_dwordx4 v[168:171], %[vxb], %[rx], %[s1o0] offen offset:2048\n\t"
      "v_mfma_f32_16x16x32_bf16 v[176:179], a[32:35], a[48:51], v[176:179]\n\t"
      "s_waitcnt lgkmcnt(0)\n\t"
      "v_mfma_f32_16x16x32_bf16 v[128:131], a[32:35], a[44:47], v[128:131]\n\t"
      "v_mfma_f32_16x16x32_bf16 v[128:131], a[0:3], a[16:19], v[128:131]\n\t"
      "v_mfma_f32_16x16x32_bf16 v[144:147], a[0:3], a[12:15], v[144:147]\n\t"
      "s_barrier\n\t"
      "ds_read_b128 a[60:63], %[xa] offset:17408\n\t"
      "v_mfma_f32_16x16x32_bf16 v[144:147], a[32:35], a[28:31], v[144:147]\n\t"
      "v_cvt_pk_bf16_f32 v176, v176, v177\n\t"
      "v_cvt_pk_bf16_f32 v177, v178, v179\n\t"
      "v_mfma_f32_16x16x32_bf16 v[148:151], a[24:27], a[28:31], v[148:151]\n\t"
      "v_cvt_pk_bf16_f32 v178, v180, v181\n\t"
      "v_or_b32 v176, 1, v176\n\t"
      "v_mfma_f32_16x16x32_bf16 v[148:151], a[20:23], a[12:15], v[148:151]\n\t"
      "v_cvt_pk_bf16_f32 v179, v182, v183\n\t"
      "v_or_b32 v178, 1, v178\n\t"
      "v_mfma_f32_16x16x32_bf16 v[132:135], a[20:23], a[16:19], v[132:135]\n\t"
      "buffer_store_dwordx4 v[176:179], %[vxb], %[rx], %[s1o1] offen offset:0\n\t"
      "v_cvt_pk_bf16_f32 v184, v184, v185\n\t"
      "v_mfma_f32_16x16x32_bf16 v[132:135], a[24:27], a[44:47], v[132:135]\n\t"
      "v_cvt_pk_bf16_f32 v185, v186, v187\n\t"
      "v_cvt_pk_bf16_f32 v186, v188, v189\n\t"
      "v_mfma_f32_16x16x32_bf16 v[136:139], a[36:39], a[44:47], v[136:139]\n\t"
      "ds_read_b128 a[56:59], %[xa] offset:17472\n\t"
      "v_or_b32 v184, 1, v184\n\t"
      "v_mfma_f32_16x16x32_bf16 v[136:139], a[4:7], a[16:19], v[136:139]\n\t"
      "ds_read_b128 a[52:55], %[xa] offset:19648\n\t"
      "ds_read_b128 a[48:51], %[xa] offset:19584\n\t"
      "v_cvt_pk_bf16_f32 v187, v190, v191\n\t"
      "v_mfma_f32_16x16x32_bf16 v[152:155], a[4:7], a[12:15], v[152:155]\n\t"
      "v_or_b32 v186, 1, v186\n\t"
      "buffer_store_dwordx4 v[184:187], %[vxb], %[rx], %[s1o1] offen offset:2048\n\t"
      "v_mfma_f32_16x16x32_bf16 v[152:155], a[36:39], a[28:31], v[152:155]\n\t"
      "v_mfma_f32_16x16x32_bf16 v[156:159], a[8:11], a[28:31], v[156:159]\n\t"
      "v_mfma_f32_16x16x32_bf16 v[156:159], a[40:43], a[12:15], v[156:159]\n\t"
      "buffer_load_dwordx4 v[164:167], %[vxb], %[rx], %[s1i0] offen offset:0 sc1\n\t"
      "v_mfma_f32_16x16x32_bf16 v[140:143], a[40:43], a[16:19], v[140:143]\n\t"
      "v_mfma_f32_16x16x32_bf16 v[140:143], a[8:11], a[44:47], v[140:143]\n\t"
      "s_waitcnt lgkmcnt(0)\n\t"
      "v_mfma_f32_16x16x32_bf16 v[64:67], a[32:35], a[60:63], v[64:67]\n\t"
      "buffer_load_dwordx4 v[172:175], %[vxb], %[rx], %[s1i0] offen offset:2048 sc1\n\t"
      "v_mfma_f32_16x16x32_bf16 v[64:67], a[0:3], a[56:59], v[64:67]\n\t"
      "v_mfma_f32_16x16x32_bf16 v[80:83], a[0:3], a[52:55], v[80:83]\n\t"
      "s_barrier\n\t"
      "ds_read_b128 a[28:31], %[xa] offset:21760\n\t"
      "v_mfma_f32_16x16x32_bf16 v[80:83], a[32:35], a[48:51], v[80:83]\n\t"
      "buffer_load_dwordx4 v[180:183], %[vxb], %[rx], %[s1i1] offen offset:0 sc1\n\t"
      "ds_read_b64_tr_b16 a[12:13], %[wa0] offset:0\n\t"
      "v_mfma_f32_16x16x32_bf16 v[84:87], a[24:27], a[48:51], v[84:87]\n\t"
      "ds_read_b64_tr_b16 a[14:15], %[wa0] offset:1040\n\t"
      "ds_read_b128 a[16:19], %[xa] offset:21824\n\t"
      "v_mfma_f32_16x16x32_bf16 v[84:87], a[20:23], a[52:55], v[84:87]\n\t"
      "ds_read_b64_tr_b16 a[44:45], %[wa0] offset:8320\n\t"
      "ds_read_b64_tr_b16 a[46:47], %[wa0] offset:9360\n\t"
      "v_mfma_f32_16x16x32_bf16 v[68:71], a[20:23], a[56:59], v[68:71]\n\t"
      "buffer_load_dwordx4 v[188:191], %[vxb], %[rx], %[s1i1] offen offset:2048 sc1\n\t"
      "ds_read_b128 a[0:3], %[xa] offset:24000\n\t"
      "v_mfma_f32_16x16x32_bf16 v[68:71], a[24:27], a[60:63], v[68:71]\n\t"
      "ds_read_b128 a[32:35], %[xa] offset:23936\n\t"
      "ds_read_b64_tr_b16 a[20:21], %[wa0] offset:32\n\t"
      "v_mfma_f32_16x16x32_bf16 v[72:75], a[36:39], a[60:63], v[72:75]\n\t"
      "ds_read_b64_tr_b16 a[22:23], %[wa0] offset:1072\n\t"
      "ds_read_b64_tr_b16 a[24:25], %[wa0] offset:8352\n\t"
      "v_mfma_f32_16x16x32_bf16 v[72:75], a[4:7], a[56:59], v[72:75]\n\t"
      "ds_read_b64_tr_b16 a[26:27], %[wa0] offset:9392\n\t"
      "v_mfma_f32_16x16x32_bf16 v[88:91], a[4:7], a[52:55], v[88:91]\n\t"
      "ds_read_b64_tr_b16 a[4:5], %[wa1] offset:0\n\t"
      "ds_read_b64_tr_b16 a[6:7], %[wa1] offset:1040\n\t"
      "v_mfma_f32_16x16x32_bf16 v[88:91], a[36:39], a[48:51], v[88:91]\n\t"
      "ds_read_b64_tr_b16 a[36:37], %[wa1] offset:8320\n\t"
      "ds_read_b64_tr_b16 a[38:39], %[wa1] offset:9360\n\t"
      "v_mfma_f32_16x16x32_bf16 v[92:95], a[8:11], a[48:51], v[92:95]\n\t"
      "v_mfma_f32_16x16x32_bf16 v[92:95], a[40:43], a[52:55], v[92:95]\n\t"
      "v_mfma_f32_16x16x32_bf16 v[76:79], a[40:43], a[56:59], v[76:79]\n\t"
      "ds_read_b64_tr_b16 a[48:49], %[wa1] offset:32\n\t"
      "ds_read_b64_tr_b16 a[50:51], %[wa1] offset:1072\n\t"
      "v_mfma_f32_16x16x32_bf16 v[76:79], a[8:11], a[60:63], v[76:79]\n\t"
      "ds_read_b64_tr_b16 a[52:53], %[wa1] offset:8352\n\t"
      "s_waitcnt lgkmcnt(11)\n\t"
      "v_mfma_f32_16x16x32_bf16 v[64:67], a[12:15], a[28:31], v[64:67]\n\t"
      "ds_read_b64_tr_b16 a[54:55], %[wa1] offset:9392\n\t"
      "v_mfma_f32_16x16x32_bf16 v[64:67], a[44:47], a[16:19], v[64:67]\n\t"
      "v_mfma_f32_16x16x32_bf16 v[80:83], a[44:47], a[0:3], v[80:83]\n\t"
      "s_barrier\n\t"
      "v_add_u32 %[xa], 17408, %[xb]\n\t"
      "v_mfma_f32_16x16x32_bf16 v[80:83], a[12:15], a[32:35], v[80:83]\n\t"
      "ds_read_b128 a[40:43], %[xa] offset:0\n\t"
      "s_waitcnt lgkmcnt(9)\n\t"
      "v_mfma_f32_16x16x32_bf16 v[84:87], a[20:23], a[32:35], v[84:87]\n\t"
      "v_add_u32 %[wa0], 34816, %[wb0]\n\t"
      "v_add_u32 %[wa1], 34816, %[wb1]\n\t"
      "v_mfma_f32_16x16x32_bf16 v[84:87], a[24:27], a[0:3], v[84:87]\n\t"
      "ds_read_b64_tr_b16 a[56:57], %[wa0] offset:0\n\t"
      "ds_read_b64_tr_b16 a[58:59], %[wa0] offset:1040\n\t"
      "v_mfma_f32_16x16x32_bf16 v[68:71], a[24:27], a[16:19], v[68:71]\n\t"
      "ds_read_b128 a[8:11], %[xa] offset:64\n\t"
      "ds_read_b64_tr_b16 a[60:61], %[wa0] offset:8320\n\t"
      "v_mfma_f32_16x16x32_bf16 v[68:71], a[20:23], a[28:31], v[68:71]\n\t"
      "ds_read_b64_tr_b16 a[62:63], %[wa0] offset:9360\n\t"
      "s_waitcnt lgkmcnt(10)\n\t"
      "v_mfma_f32_16x16x32_bf16 v[72:75], a[4:7], a[28:31], v[72:75]\n\t"
      "ds_read_b128 a[44:47], %[xa] offset:2240\n\t"
      "ds_read_b128 a[12:15], %[xa] offset:2176\n\t"
      "v_mfma_f32_16x16x32_bf16 v[72:75], a[36:39], a[16:19], v[72:75]\n\t"
      "ds_read_b64_tr_b16 a[24:25], %[wa0] offset:32\n\t"
      "ds_read_b64_tr_b16 a[26:27], %[wa0] offset:1072\n\t"
      "v_mfma_f32_16x16x32_bf16 v[88:91], a[36:39], a[0:3], v[88:91]\n\t"
      "ds_read_b64_tr_b16 a[20:21], %[wa0] offset:8352\n\t"
      "ds_read_b64_tr_b16 a[22:23], %[wa0] offset:9392\n\t"
      "v_mfma_f32_16x16x32_bf16 v[88:91], a[4:7], a[32:35], v[88:91]\n\t"
      "ds_read_b64_tr_b16 a[36:37], %[wa1] offset:0\n\t"
      "s_waitcnt lgkmcnt(13)\n\t"
      "v_mfma_f32_16x16x32_bf16 v[92:95], a[48:51], a[32:35], v[92:95]\n\t"
      "ds_read_b64_tr_b16 a[38:39], %[wa1] offset:1040\n\t"
      "ds_read_b64_tr_b16 a[4:5], %[wa1] offset:8320\n\t"
      "v_mfma_f32_16x16x32_bf16 v[92:95], a[52:55], a[0:3], v[92:95]\n\t"
      "ds_read_b64_tr_b16 a[6:7], %[wa1] offset:9360\n\t"
      "v_mfma_f32_16x16x32_bf16 v[76:79], a[52:55], a[16:19], v[76:79]\n\t"
      "ds_read_b64_tr_b16 a[32:33], %[wa1] offset:32\n\t"
      "ds_read_b64_tr_b16 a[34:35], %[wa1] offset:1072\n\t"
      "v_mfma_f32_16x16x32_bf16 v[76:79], a[48:51], a[28:31], v[76:79]\n\t"
      "ds_read_b64_tr_b16 a[0:1], %[wa1] offset:8352\n\t"
      "s_waitcnt lgkmcnt(11)\n\t"
      "v_mfma_f32_16x16x32_bf16 v[64:67], a[56:59], a[40:43], v[64:67]\n\t"
      "ds_read_b64_tr_b16 a[2:3], %[wa1] offset:9392\n\t"
      "v_mfma_f32_16x16x32_bf16 v[64:67], a[60:63], a[8:11], v[64:67]\n\t"
      "v_mfma_f32_16x16x32_bf16 v[80:83], a[60:63], a[44:47], v[80:83]\n\t"
      "v_mfma_f32_16x16x32_bf16 v[80:83], a[56:59], a[12:15], v[80:83]\n\t"
      "s_waitcnt lgkmcnt(8)\n\t"
      "v_mfma_f32_16x16x32_bf16 v[84:87], a[24:27], a[12:15], v[84:87]\n\t"
      "v_mfma_f32_16x16x32_bf16 v[84:87], a[20:23], a[44:47], v[84:87]\n\t"
      "v_mfma_f32_16x16x32_bf16 v[68:71], a[20:23], a[8:11], v[68:71]\n\t"
      "v_mfma_f32_16x16x32_bf16 v[68:71], a[24:27], a[40:43], v[68:71]\n\t"
      "s_waitcnt lgkmcnt(4)\n\t"
      "v_mfma_f32_16x16x32_bf16 v[72:75], a[36:39], a[40:43], v[72:75]\n\t"
      "v_mfma_f32_16x16x32_bf16 v[72:75], a[4:7], a[8:11], v[72:75]\n\t"
      "v_mfma_f32_16x16x32_bf16 v[88:91], a[4:7], a[44:47], v[88:91]\n\t"
      "s_barrier\n\t"
      "ds_read_b128 a[52:55], %[xa] offset:4352\n\t"
      "v_mfma_f32_16x16x32_bf16 v[88:91], a[36:39], a[12:15], v[88:91]\n\t"
      "ds_read_b128 a[16:19], %[xa] offset:4416\n\t"
      "s_waitcnt lgkmcnt(2)\n\t"
      "v_mfma_f32_16x16x32_bf16 v[92:95], a[32:35], a[12:15], v[92:95]\n\t"
      "s_waitcnt vmcnt(0)\n\t"
      "v_bitop3_b32 %[c0], v164, v166, v172 bitop3:0x80\n\t"
      "v_mfma_f32_16x16x32_bf16 v[92:95], a[0:3], a[44:47], v[92:95]\n\t"
      "v_bitop3_b32 %[c0], %[c0], v174, v180 bitop3:0x80\n\t"
      "v_bitop3_b32 %[c0], %[c0], v182, v188 bitop3:0x80\n\t"
      "v_mfma_f32_16x16x32_bf16 v[76:79], a[0:3], a[8:11], v[76:79]\n\t"
      "v_bitop3_b32 %[c0], %[c0], v190, 1 bitop3:0x2a\n\t"
      "v_cmp_ne_u32_e64 %[bad], 0, %[c0]\n\t"
      "v_mfma_f32_16x16x32_bf16 v[76:79], a[32:35], a[40:43], v[76:79]\n\t"
      "ds_read_b128 a[48:51], %[xa] offset:6592\n\t"
      "ds_read_b128 a[28:31], %[xa] offset:6528"
      : "+{v[32:47]}"(A[0]), "+{v[48:63]}"(A[1]), "+{v[64:79]}"(A[2]), "+{v[80:95]}"(A[3]), [xa] "+v"(xa), [wa0] "+v"(wa0), [wa1] "+v"(wa1), [bad] "=&s"(bad), [c0] "=&v"(c0), [c1] "=&v"(c1)
      : [xb] "v"(xb), [wb0] "v"(wb0), [wb1] "v"(wb1), [vxb] "v"(vxb), [rx] "s"(rxs), [s1o0] "s"(s1o[0]), [s1o1] "s"(s1o[1]), [s1i0] "s"(s1i[0]), [s1i1] "s"(s1i[1]), [vpf] "v"(vpf)
      : "memory", "scc", ACLOB);
  AWAIT_A();
  asm volatile(
      "v_mfma_f32_16x16x32_bf16 v[64:67], %[sel0], v[164:167], v[64:67]\n\t"
      "v_mfma_f32_16x16x32_bf16 v[68:71], %[sel1], v[164:167], v[68:71]\n\t"
      "v_mfma_f32_16x16x32_bf16 v[72:75], %[sel0], v[172:175], v[72:75]\n\t"
      "v_mfma_f32_16x16x32_bf16 v[76:79], %[sel1], v[172:175], v[76:79]\n\t"
      "v_mfma_f32_16x16x32_bf16 v[80:83], %[sel0], v[180:183], v[80:83]\n\t"
      "v_mfma_f32_16x16x32_bf16 v[84:87], %[sel1], v[180:183], v[84:87]\n\t"
      "v_cvt_pk_bf16_f32 v64, v64, v65\n\t"
      "v_cvt_pk_bf16_f32 v65, v66, v67\n\t"
      "v_mfma_f32_16x16x32_bf16 v[88:91], %[sel0], v[188:191], v[88:91]\n\t"
      "v_cvt_pk_bf16_f32 v66, v68, v69\n\t"
      "v_and_or_b32 v64, v64, -2, %[tag]\n\t"
      "v_mfma_f32_16x16x32_bf16 v[92:95], %[sel1], v[188:191], v[92:95]\n\t"
      "v_cvt_pk_bf16_f32 v67, v70, v71\n\t"
      "s_waitcnt lgkmcnt(0)\n\t"
      "v_mfma_f32_16x16x32_bf16 v[128:131], a[56:59], a[52:55], v[128:131]\n\t"
      "v_and_or_b32 v66, v66, -2, %[tag]\n\t"
      "buffer_store_dwordx4 v[64:67], %[vxb], %[rx], %[s2o0] offen offset:0 sc1\n\t"
      "v_mfma_f32_16x16x32_bf16 v[128:131], a[60:63], a[16:19], v[128:131]\n\t"
      "v_cvt_pk_bf16_f32 v72, v72, v73\n\t"
      "v_cvt_pk_bf16_f32 v73, v74, v75\n\t"
      "v_mfma_f32_16x16x32_bf16 v[144:147], a[60:63], a[48:51], v[144:147]\n\t"
      "s_barrier\n\t"
      "ds_read_b128 a[12:15], %[xa] offset:34816\n\t"
      "v_cvt_pk_bf16_f32 v74, v76, v77\n\t"
      "v_mfma_f32_16x16x32_bf16 v[144:147], a[56:59], a[28:31], v[144:147]\n\t"
      "ds_read_b64_tr_b16 a[44:45], %[wa0] offset:34816\n\t"
      "ds_read_b64_tr_b16 a[46:47], %[wa0] offset:35856\n\t"
      "v_and_or_b32 v72, v72, -2, %[tag]\n\t"
      "v_mfma_f32_16x16x32_bf16 v[148:151], a[24:27], a[28:31], v[148:151]\n\t"
      "ds_read_b128 a[8:11], %[xa] offset:34880\n\t"
      "v_cvt_pk_bf16_f32 v75, v78, v79\n\t"
      "v_mfma_f32_16x16x32_bf16 v[148:151], a[20:23], a[48:51], v[148:151]\n\t"
      "ds_read_b64_tr_b16 a[40:41], %[wa0] offset:43136\n\t"
      "ds_read_b64_tr_b16 a[42:43], %[wa0] offset:44176\n\t"
      "v_and_or_b32 v74, v74, -2, %[tag]\n\t"
      "v_mfma_f32_16x16x32_bf16 v[132:135], a[20:23], a[16:19], v[132:135]\n\t"
      "ds_read_b64_tr_b16 a[60:61], %[wa0] offset:43168\n\t"
      "buffer_store_dwordx4 v[72:75], %[vxb], %[rx], %[s2o0] offen offset:2048 sc1\n\t"
      "v_mfma_f32_16x16x32_bf16 v[132:135], a[24:27], a[52:55], v[132:135]\n\t"
      "ds_read_b64_tr_b16 a[62:63], %[wa0] offset:44208\n\t"
      "ds_read_b64_tr_b16 a[56:57], %[wa0] offset:34848\n\t"
      "v_cvt_pk_bf16_f32 v80, v80, v81\n\t"
      "v_mfma_f32_16x16x32_bf16 v[136:139], a[36:39], a[52:55], v[136:139]\n\t"
      "ds_read_b64_tr_b16 a[58:59], %[wa0] offset:35888\n\t"
      "v_cvt_pk_bf16_f32 v81, v82, v83\n\t"
      "v_mfma_f32_16x16x32_bf16 v[136:139], a[4:7], a[16:19], v[136:139]\n\t"
      "ds_read_b64_tr_b16 a[20:21], %[wa1] offset:34816\n\t"
      "ds_read_b64_tr_b16 a[22:23], %[wa1] offset:35856\n\t"
      "v_cvt_pk_bf16_f32 v82, v84, v85\n\t"
      "v_mfma_f32_16x16x32_bf16 v[152:155], a[4:7], a[48:51], v[152:155]\n\t"
      "ds_read_b64_tr_b16 a[24:25], %[wa1] offset:43136\n\t"
      "v_and_or_b32 v80, v80, -2, %[tag]\n\t"
      "v_mfma_f32_16x16x32_bf16 v[152:155], a[36:39], a[28:31], v[152:155]\n\t"
      "ds_read_b64_tr_b16 a[26:27], %[wa1] offset:44176\n\t"
      "ds_read_b64_tr_b16 a[4:5], %[wa1] offset:43168\n\t"
      "v_cvt_pk_bf16_f32 v83, v86, v87\n\t"
      "v_mfma_f32_16x16x32_bf16 v[156:159], a[32:35], a[28:31], v[156:159]\n\t"
      "ds_read_b64_tr_b16 a[6:7], %[wa1] offset:44208\n\t"
      "v_and_or_b32 v82, v82, -2, %[tag]\n\t"
      "v_mfma_f32_16x16x32_bf16 v[156:159], a[0:3], a[48:51], v[156:159]\n\t"
      "ds_read_b64_tr_b16 a[36:37], %[wa1] offset:34848\n\t"
      "ds_read_b64_tr_b16 a[38:39], %[wa1] offset:35888\n\t"
      "buffer_store_dwordx4 v[80:83], %[vxb], %[rx], %[s2o1] offen offset:0 sc1\n\t"
      "v_mfma_f32_16x16x32_bf16 v[140:143], a[0:3], a[16:19], v[140:143]\n\t"
      "v_cvt_pk_bf16_f32 v88, v88, v89\n\t"
      "v_cvt_pk_bf16_f32 v89, v90, v91\n\t"
      "v_mfma_f32_16x16x32_bf16 v[140:143], a[32:35], a[52:55], v[140:143]\n\t"
      "v_cvt_pk_bf16_f32 v90, v92, v93\n\t"
      "s_waitcnt lgkmcnt(8)\n\t"
      "v_mfma_f32_16x16x32_bf16 v[128:131], a[44:47], a[12:15], v[128:131]\n\t"
      "v_and_or_b32 v88, v88, -2, %[tag]\n\t"
      "v_cvt_pk_bf16_f32 v91, v94, v95\n\t"
      "v_mfma_f32_16x16x32_bf16 v[128:131], a[40:43], a[8:11], v[128:131]\n\t"
      "ds_read_b128 a[28:31], %[xa] offset:36992\n\t"
      "ds_read_b128 a[48:51], %[xa] offset:37056\n\t"
      "v_and_or_b32 v90, v90, -2, %[tag]\n\t"
      "v_mfma_f32_16x16x32_bf16 v[132:135], a[60:63], a[8:11], v[132:135]\n\t"
      "buffer_store_dwordx4 v[88:91], %[vxb], %[rx], %[s2o1] offen offset:2048 sc1\n\t"
      "s_barrier\n\t"
      "v_mfma_f32_16x16x32_bf16 v[132:135], a[56:59], a[12:15], v[132:135]\n\t"
      "ds_read_b128 a[0:3], %[xa] offset:39168\n\t"
      "s_waitcnt lgkmcnt(3)\n\t"
      "v_mfma_f32_16x16x32_bf16 v[136:139], a[20:23], a[12:15], v[136:139]\n\t"
      "ds_read_b128 a[16:19], %[xa] offset:39232\n\t"
      "ds_read_b128 a[32:35], %[xa] offset:41408\n\t"
      "v_mfma_f32_16x16x32_bf16 v[136:139], a[24:27], a[8:11], v[136:139]\n\t"
      "ds_read_b128 a[52:55], %[xa] offset:41344\n\t"
      "v_mfma_f32_16x16x32_bf16 v[140:143], a[4:7], a[8:11], v[140:143]\n\t"
      "v_mfma_f32_16x16x32_bf16 v[140:143], a[36:39], a[12:15], v[140:143]\n\t"
      "v_cvt_pk_bf16_f32 v128, v128, v129\n\t"
      "s_waitcnt lgkmcnt(4)\n\t"
      "v_mfma_f32_16x16x32_bf16 v[156:159], a[36:39], a[28:31], v[156:159]\n\t"
      "v_cvt_pk_bf16_f32 v129, v130, v131\n\t"
      "v_cvt_pk_bf16_f32 v130, v132, v133\n\t"
      "v_mfma_f32_16x16x32_bf16 v[156:159], a[4:7], a[48:51], v[156:159]\n\t"
      "v_and_b32 v128, -2, v128\n\t"
      "v_cvt_pk_bf16_f32 v131, v134, v135\n\t"
      "v_mfma_f32_16x16x32_bf16 v[152:155], a[24:27], a[48:51], v[152:155]\n\t"
      "v_and_b32 v130, -2, v130\n\t"
      "buffer_store_dwordx4 v[128:131], %[vxb], %[rx], %[s1i0] offen offset:0\n\t"
      "v_mfma_f32_16x16x32_bf16 v[152:155], a[20:23], a[28:31], v[152:155]\n\t"
      "v_cvt_pk_bf16_f32 v136, v136, v137\n\t"
      "v_cvt_pk_bf16_f32 v137, v138, v139\n\t"
      "v_mfma_f32_16x16x32_bf16 v[148:151], a[56:59], a[28:31], v[148:151]\n\t"
      "v_cvt_pk_bf16_f32 v138, v140, v141\n\t"
      "v_and_b32 v136, -2, v136\n\t"
      "v_mfma_f32_16x16x32_bf16 v[148:151], a[60:63], a[48:51], v[148:151]\n\t"
      "v_cvt_pk_bf16_f32 v139, v142, v143\n\t"
      "v_and_b32 v138, -2, v138\n\t"
      "v_mfma_f32_16x16x32_bf16 v[144:147], a[40:43], a[48:51], v[144:147]\n\t"
      "buffer_store_dwordx4 v[136:139], %[vxb], %[rx], %[s1i0] offen offset:2048\n\t"
      "v_mfma_f32_16x16x32_bf16 v[144:147], a[44:47], a[28:31], v[144:147]\n\t"
      "s_waitcnt lgkmcnt(0)\n\t"
      "v_mfma_f32_16x16x32_bf16 v[32:35], a[44:47], a[0:3], v[32:35]\n\t"
      "v_mfma_f32_16x16x32_bf16 v[32:35], a[40:43], a[16:19], v[32:35]\n\t"
      "v_mfma_f32_16x16x32_bf16 v[48:51], a[40:43], a[32:35], v[48:51]\n\t"
      "s_barrier\n\t"
      "v_add_u32 %[xa], 121856, %[xb]\n\t"
      "v_mfma_f32_16x16x32_bf16 v[48:51], a[44:47], a[52:55], v[48:51]\n\t"
      "v_cvt_pk_bf16_f32 v144, v144, v145\n\t"
      "v_cvt_pk_bf16_f32 v145, v146, v147\n\t"
      "v_mfma_f32_16x16x32_bf16 v[52:55], a[56:59], a[52:55], v[52:55]\n\t"
      "v_cvt_pk_bf16_f32 v146, v148, v149\n\t"
      "v_and_b32 v144, -2, v144\n\t"
      "v_mfma_f32_16x16x32_bf16 v[52:55], a[60:63], a[32:35], v[52:55]\n\t"
      "ds_read_b128 a[8:11], %[xa] offset:0\n\t"
      "ds_read_b64_tr_b16 a[12:13], %[wa0] offset:52224\n\t"
      "v_cvt_pk_bf16_f32 v147, v150, v151\n\t"
      "v_mfma_f32_16x16x32_bf16 v[36:39], a[60:63], a[16:19], v[36:39]\n\t"
      "ds_read_b64_tr_b16 a[14:15], %[wa0] offset:53264\n\t"
      "v_and_b32 v146, -2, v146\n\t"
      "v_mfma_f32_16x16x32_bf16 v[36:39], a[56:59], a[0:3], v[36:39]\n\t"
      "ds_read_b128 a[48:51], %[xa] offset:64\n\t"
      "ds_read_b64_tr_b16 a[28:29], %[wa0] offset:60544\n\t"
      "buffer_store_dwordx4 v[144:147], %[vxb], %[rx], %[s1i1] offen offset:0\n\t"
      "v_mfma_f32_16x16x32_bf16 v[40:43], a[20:23], a[0:3], v[40:43]\n\t"
      "ds_read_b64_tr_b16 a[30:31], %[wa0] offset:61584\n\t"
      "v_cvt_pk_bf16_f32 v152, v152, v153\n\t"
      "v_mfma_f32_16x16x32_bf16 v[40:43], a[24:27], a[16:19], v[40:43]\n\t"
      "ds_read_b128 a[40:43], %[xa] offset:2240\n\t"
      "ds_read_b128 a[44:47], %[xa] offset:2176\n\t"
      "v_cvt_pk_bf16_f32 v153, v154, v155\n\t"
      "v_mfma_f32_16x16x32_bf16 v[56:59], a[24:27], a[32:35], v[56:59]\n\t"
      "v_cvt_pk_bf16_f32 v154, v156, v157\n\t"
      "v_and_b32 v152, -2, v152\n\t"
      "v_mfma_f32_16x16x32_bf16 v[56:59], a[20:23], a[52:55], v[56:59]\n\t"
      "ds_read_b64_tr_b16 a[60:61], %[wa0] offset:52256\n\t"
      "v_cvt_pk_bf16_f32 v155, v158, v159\n\t"
      "v_mfma_f32_16x16x32_bf16 v[60:63], a[36:39], a[52:55], v[60:63]\n\t"
      "ds_read_b64_tr_b16 a[62:63], %[wa0] offset:53296\n\t"
      "v_and_b32 v154, -2, v154\n\t"
      "v_mfma_f32_16x16x32_bf16 v[60:63], a[4:7], a[32:35], v[60:63]\n\t"
      "ds_read_b64_tr_b16 a[56:57], %[wa0] offset:60576\n\t"
      "ds_read_b64_tr_b16 a[58:59], %[wa0] offset:61616\n\t"
      "buffer_store_dwordx4 v[152:155], %[vxb], %[rx], %[s1i1] offen offset:2048\n\t"
      "v_mfma_f32_16x16x32_bf16 v[44:47], a[4:7], a[16:19], v[44:47]\n\t"
      "ds_read_b64_tr_b16 a[24:25], %[wa1] offset:52224\n\t"
      "ds_read_b64_tr_b16 a[26:27], %[wa1] offset:53264\n\t"
      "v_mfma_f32_16x16x32_bf16 v[44:47], a[36:39], a[0:3], v[44:47]\n\t"
      "ds_read_b64_tr_b16 a[20:21], %[wa1] offset:60544\n\t"
      "s_waitcnt lgkmcnt(7)\n\t"
      "v_mfma_f32_16x16x32_bf16 v[32:35], a[12:15], a[8:11], v[32:35]\n\t"
      "ds_read_b64_tr_b16 a[22:23], %[wa1] offset:61584\n\t"
      "ds_read_b64_tr_b16 a[52:53], %[wa1] offset:52256\n\t"
      "v_mfma_f32_16x16x32_bf16 v[32:35], a[28:31], a[48:51], v[32:35]\n\t"
      "ds_read_b64_tr_b16 a[54:55], %[wa1] offset:53296\n\t"
      "ds_read_b64_tr_b16 a[32:33], %[wa1] offset:60576\n\t"
      "v_mfma_f32_16x16x32_bf16 v[48:51], a[28:31], a[40:43], v[48:51]\n\t"
      "ds_read_b64_tr_b16 a[34:35], %[wa1] offset:61616\n\t"
      "s_barrier\n\t"
      "v_mfma_f32_16x16x32_bf16 v[48:51], a[12:15], a[44:47], v[48:51]\n\t"
      "ds_read_b128 a[4:7], %[xa] offset:4352\n\t"
      "s_waitcnt lgkmcnt(9)\n\t"
      "v_mfma_f32_16x16x32_bf16 v[52:55], a[60:63], a[44:47], v[52:55]\n\t"
      "ds_read_b64_tr_b16 a[16:17], %[wa0] offset:0\n\t"
      "ds_read_b64_tr_b16 a[18:19], %[wa0] offset:1040\n\t"
      "v_mfma_f32_16x16x32_bf16 v[52:55], a[56:59], a[40:43], v[52:55]\n\t"
      "ds_read_b128 a[36:39], %[xa] offset:4416\n\t"
      "ds_read_b64_tr_b16 a[0:1], %[wa0] offset:8320\n\t"
      "v_mfma_f32_16x16x32_bf16 v[36:39], a[56:59], a[48:51], v[36:39]\n\t"
      "ds_read_b64_tr_b16 a[2:3], %[wa0] offset:9360\n\t"
      "ds_read_b128 a[28:31], %[xa] offset:6592\n\t"
      "v_mfma_f32_16x16x32_bf16 v[36:39], a[60:63], a[8:11], v[36:39]\n\t"
      "ds_read_b128 a[12:15], %[xa] offset:6528\n\t"
      "s_waitcnt lgkmcnt(12)\n\t"
      "v_mfma_f32_16x16x32_bf16 v[40:43], a[24:27], a[8:11], v[40:43]\n\t"
      "ds_read_b64_tr_b16 a[56:57], %[wa0] offset:32\n\t"
      "ds_read_b64_tr_b16 a[58:59], %[wa0] offset:1072\n\t"
      "v_mfma_f32_16x16x32_bf16 v[40:43], a[20:23], a[48:51], v[40:43]\n\t"
      "ds_read_b64_tr_b16 a[60:61], %[wa0] offset:8352\n\t"
      "ds_read_b64_tr_b16 a[62:63], %[wa0] offset:9392\n\t"
      "v_mfma_f32_16x16x32_bf16 v[56:59], a[20:23], a[40:43], v[56:59]\n\t"
      "ds_read_b64_tr_b16 a[20:21], %[wa1] offset:0\n\t"
      "ds_read_b64_tr_b16 a[22:23], %[wa1] offset:1040\n\t"
      "v_mfma_f32_16x16x32_bf16 v[56:59], a[24:27], a[44:47], v[56:59]\n\t"
      "ds_read_b64_tr_b16 a[24:25], %[wa1] offset:8320\n\t"
      "s_waitcnt lgkmcnt(14)\n\t"
      "v_mfma_f32_16x16x32_bf16 v[60:63], a[52:55], a[44:47], v[60:63]\n\t"
      "ds_read_b64_tr_b16 a[26:27], %[wa1] offset:9360\n\t"
      "v_mfma_f32_16x16x32_bf16 v[60:63], a[32:35], a[40:43], v[60:63]\n\t"
      "v_mfma_f32_16x16x32_bf16 v[44:47], a[32:35], a[48:51], v[44:47]\n\t"
      "ds_read_b64_tr_b16 a[44:45], %[wa1] offset:32\n\t"
      "ds_read_b64_tr_b16 a[46:47], %[wa1] offset:1072\n\t"
      "v_mfma_f32_16x16x32_bf16 v[44:47], a[52:55], a[8:11], v[44:47]\n\t"
      "ds_read_b64_tr_b16 a[40:41], %[wa1] offset:8352\n\t"
      "s_waitcnt lgkmcnt(11)\n\t"
      "v_mfma_f32_16x16x32_bf16 v[32:35], a[16:19], a[4:7], v[32:35]\n\t"
      "ds_read_b64_tr_b16 a[42:43], %[wa1] offset:9392\n\t"
      "v_mfma_f32_16x16x32_bf16 v[32:35], a[0:3], a[36:39], v[32:35]\n\t"
      "v_mfma_f32_16x16x32_bf16 v[48:51], a[0:3], a[28:31], v[48:51]\n\t"
      "s_barrier\n\t"
      "ds_read_b128 a[32:35], %[xa] offset:17408\n\t"
      "v_mfma_f32_16x16x32_bf16 v[48:51], a[16:19], a[12:15], v[48:51]\n\t"
      "buffer_load_dwordx4 v[164:167], %[vxb], %[rx], %[s2i0] offen offset:0 sc1\n\t"
      "s_waitcnt lgkmcnt(9)\n\t"
      "v_mfma_f32_16x16x32_bf16 v[52:55], a[56:59], a[12:15], v[52:55]\n\t"
      "v_add_u32_e64 %[wa0], 0, %[wb0]\n\t"
      "v_add_u32_e64 %[wa1], 0, %[wb1]\n\t"
      "v_mfma_f32_16x16x32_bf16 v[52:55], a[60:63], a[28:31], v[52:55]\n\t"
      "ds_read_b64_tr_b16 a[48:49], %[wa0] offset:0\n\t"
      "ds_read_b64_tr_b16 a[50:51], %[wa0] offset:1040\n\t"
      "v_mfma_f32_16x16x32_bf16 v[36:39], a[60:63], a[36:39], v[36:39]\n\t"
      "buffer_load_dwordx4 v[172:175], %[vxb], %[rx], %[s2i0] offen offset:2048 sc1\n\t"
      "ds_read_b128 a[52:55], %[xa] offset:17472\n\t"
      "v_mfma_f32_16x16x32_bf16 v[36:39], a[56:59], a[4:7], v[36:39]\n\t"
      "ds_read_b64_tr_b16 a[8:9], %[wa0] offset:8320\n\t"
      "s_waitcnt lgkmcnt(9)\n\t"
      "v_mfma_f32_16x16x32_bf16 v[40:43], a[20:23], a[4:7], v[40:43]\n\t"
      "ds_read_b64_tr_b16 a[10:11], %[wa0] offset:9360\n\t"
      "ds_read_b128 a[0:3], %[xa] offset:19648\n\t"
      "v_mfma_f32_16x16x32_bf16 v[40:43], a[24:27], a[36:39], v[40:43]\n\t"
      "buffer_load_dwordx4 v[180:183], %[vxb], %[rx], %[s2i1] offen offset:0 sc1\n\t"
      "ds_read_b128 a[16:19], %[xa] offset:19584\n\t"
      "v_mfma_f32_16x16x32_bf16 v[56:59], a[24:27], a[28:31], v[56:59]\n\t"
      "ds_read_b64_tr_b16 a[60:61], %[wa0] offset:32\n\t"
      "ds_read_b64_tr_b16 a[62:63], %[wa0] offset:1072\n\t"
      "v_mfma_f32_16x16x32_bf16 v[56:59], a[20:23], a[12:15], v[56:59]\n\t"
      "ds_read_b64_tr_b16 a[56:57], %[wa0] offset:8352\n\t"
      "s_waitcnt lgkmcnt(11)\n\t"
      "v_mfma_f32_16x16x32_bf16 v[60:63], a[44:47], a[12:15], v[60:63]\n\t"
      "buffer_load_dwordx4 v[188:191], %[vxb], %[rx], %[s2i1] offen offset:2048 sc1\n\t"
      "ds_read_b64_tr_b16 a[58:59], %[wa0] offset:9392\n\t"
      "v_mfma_f32_16x16x32_bf16 v[60:63], a[40:43], a[28:31], v[60:63]\n\t"
      "ds_read_b64_tr_b16 a[24:25], %[wa1] offset:0\n\t"
      "ds_read_b64_tr_b16 a[26:27], %[wa1] offset:1040\n\t"
      "v_mfma_f32_16x16x32_bf16 v[44:47], a[40:43], a[36:39], v[44:47]\n\t"
      "ds_read_b64_tr_b16 a[20:21], %[wa1] offset:8320\n\t"
      "ds_read_b64_tr_b16 a[22:23], %[wa1] offset:9360\n\t"
      "v_mfma_f32_16x16x32_bf16 v[44:47], a[44:47], a[4:7], v[44:47]\n\t"
      "buffer_load_dwordx4 v[160:163], %[vxb], %[rx], %[s1o0] offen offset:0 sc1\n\t"
      "s_waitcnt lgkmcnt(8)\n\t"
      "v_mfma_f32_16x16x32_bf16 v[32:35], a[48:51], a[32:35], v[32:35]\n\t"
      "ds_read_b64_tr_b16 a[12:13], %[wa1] offset:32\n\t"
      "ds_read_b64_tr_b16 a[14:15], %[wa1] offset:1072\n\t"
      "v_mfma_f32_16x16x32_bf16 v[32:35], a[8:11], a[52:55], v[32:35]\n\t"
      "ds_read_b64_tr_b16 a[28:29], %[wa1] offset:8352\n\t"
      "ds_read_b64_tr_b16 a[30:31], %[wa1] offset:9392\n\t"
      "v_mfma_f32_16x16x32_bf16 v[48:51], a[8:11], a[0:3], v[48:51]\n\t"
      "buffer_load_dwordx4 v[168:171], %[vxb], %[rx], %[s1o0] offen offset:2048 sc1\n\t"
      "s_barrier\n\t"
      "v_mfma_f32_16x16x32_bf16 v[48:51], a[48:51], a[16:19], v[48:51]\n\t"
      "ds_read_b128 a[40:43], %[xa] offset:21760\n\t"
      "s_waitcnt lgkmcnt(9)\n\t"
      "v_mfma_f32_16x16x32_bf16 v[52:55], a[60:63], a[16:19], v[52:55]\n\t"
      "v_add_u32 %[wa0], 104448, %[wb0]\n\t"
      "v_add_u32 %[wa1], 104448, %[wb1]\n\t"
      "v_mfma_f32_16x16x32_bf16 v[52:55], a[56:59], a[0:3], v[52:55]\n\t"
      "buffer_load_dwordx4 v[176:179], %[vxb], %[rx], %[s1o1] offen offset:0 sc1\n\t"
      "ds_read_b64_tr_b16 a[36:37], %[wa0] offset:0\n\t"
      "v_mfma_f32_16x16x32_bf16 v[36:39], a[56:59], a[52:55], v[36:39]\n\t"
      "ds_read_b64_tr_b16 a[38:39], %[wa0] offset:1040\n\t"
      "ds_read_b128 a[44:47], %[xa] offset:21824\n\t"
      "v_mfma_f32_16x16x32_bf16 v[36:39], a[60:63], a[32:35], v[36:39]\n\t"
      "ds_read_b64_tr_b16 a[4:5], %[wa0] offset:8320\n\t"
      "s_waitcnt lgkmcnt(9)\n\t"
      "v_mfma_f32_16x16x32_bf16 v[40:43], a[24:27], a[32:35], v[40:43]\n\t"
      "buffer_load_dwordx4 v[184:187], %[vxb], %[rx], %[s1o1] offen offset:2048 sc1\n\t"
      "ds_read_b64_tr_b16 a[6:7], %[wa0] offset:9360\n\t"
      "v_mfma_f32_16x16x32_bf16 v[40:43], a[20:23], a[52:55], v[40:43]\n\t"
      "ds_read_b128 a[8:11], %[xa] offset:24000\n\t"
      "ds_read_b128 a[48:51], %[xa] offset:23936\n\t"
      "v_mfma_f32_16x16x32_bf16 v[56:59], a[20:23], a[0:3], v[56:59]\n\t"
      "ds_read_b64_tr_b16 a[56:57], %[wa0] offset:32\n\t"
      "ds_read_b64_tr_b16 a[58:59], %[wa0] offset:1072\n\t"
      "v_mfma_f32_16x16x32_bf16 v[56:59], a[24:27], a[16:19], v[56:59]\n\t"
      "ds_read_b64_tr_b16 a[60:61], %[wa0] offset:8352\n\t"
      "s_waitcnt lgkmcnt(11)\n\t"
      "v_mfma_f32_16x16x32_bf16 v[60:63], a[12:15], a[16:19], v[60:63]\n\t"
      "ds_read_b64_tr_b16 a[62:63], %[wa0] offset:9392\n\t"
      "ds_read_b64_tr_b16 a[20:21], %[wa1] offset:0\n\t"
      "v_mfma_f32_16x16x32_bf16 v[60:63], a[28:31], a[0:3], v[60:63]\n\t"
      "ds_read_b64_tr_b16 a[22:23], %[wa1] offset:1040\n\t"
      "ds_read_b64_tr_b16 a[24:25], %[wa1] offset:8320\n\t"
      "v_mfma_f32_16x16x32_bf16 v[44:47], a[28:31], a[52:55], v[44:47]\n\t"
      "ds_read_b64_tr_b16 a[26:27], %[wa1] offset:9360\n\t"
      "ds_read_b64_tr_b16 a[16:17], %[wa1] offset:32\n\t"
      "v_mfma_f32_16x16x32_bf16 v[44:47], a[12:15], a[32:35], v[44:47]\n\t"
      "ds_read_b64_tr_b16 a[18:19], %[wa1] offset:1072\n\t"
      "s_waitcnt lgkmcnt(10)\n\t"
      "v_mfma_f32_16x16x32_bf16 v[32:35], a[36:39], a[40:43], v[32:35]\n\t"
      "ds_read_b64_tr_b16 a[0:1], %[wa1] offset:8352\n\t"
      "ds_read_b64_tr_b16 a[2:3], %[wa1] offset:9392\n\t"
      "v_mfma_f32_16x16x32_bf16 v[32:35], a[4:7], a[44:47], v[32:35]\n\t"
      "v_mfma_f32_16x16x32_bf16 v[48:51], a[4:7], a[8:11], v[48:51]\n\t"
      "v_mfma_f32_16x16x32_bf16 v[48:51], a[36:39], a[48:51], v[48:51]\n\t"
      "s_waitcnt lgkmcnt(8)\n\t"
      "v_mfma_f32_16x16x32_bf16 v[52:55], a[56:59], a[48:51], v[52:55]\n\t"
      "v_mfma_f32_16x16x32_bf16 v[52:55], a[60:63], a[8:11], v[52:55]\n\t"
      "v_mfma_f32_16x16x32_bf16 v[36:39], a[60:63], a[44:47], v[36:39]\n\t"
      "v_mfma_f32_16x16x32_bf16 v[36:39], a[56:59], a[40:43], v[36:39]\n\t"
      "s_waitcnt lgkmcnt(4)\n\t"
      "v_mfma_f32_16x16x32_bf16 v[40:43], a[20:23], a[40:43], v[40:43]\n\t"
      "v_mfma_f32_16x16x32_bf16 v[40:43], a[24:27], a[44:47], v[40:43]\n\t"
      "v_mfma_f32_16x16x32_bf16 v[56:59], a[24:27], a[8:11], v[56:59]\n\t"
      "v_mfma_f32_16x16x32_bf16 v[56:59], a[20:23], a[48:51], v[56:59]\n\t"
      "s_waitcnt lgkmcnt(0)\n\t"
      "v_mfma_f32_16x16x32_bf16 v[60:63], a[16:19], a[48:51], v[60:63]\n\t"
      "s_waitcnt vmcnt(4)\n\t"
      "v_bitop3_b32 %[c0], v164, v166, %[tag] bitop3:0x7e\n\t"
      "v_mfma_f32_16x16x32_bf16 v[60:63], a[0:3], a[8:11], v[60:63]\n\t"
      "v_bitop3_b32 %[c1], v172, v174, %[tag] bitop3:0x7e\n\t"
      "v_bitop3_b32 %[c2], v180, v182, %[tag] bitop3:0x7e\n\t"
      "v_mfma_f32_16x16x32_bf16 v[44:47], a[0:3], a[44:47], v[44:47]\n\t"
      "v_or3_b32 %[c0], %[c0], %[c1], %[c2]\n\t"
      "v_bitop3_b32 %[c1], v188, v190, %[tag] bitop3:0x7e\n\t"
      "v_mfma_f32_16x16x32_bf16 v[44:47], a[16:19], a[40:43], v[44:47]\n\t"
      "v_bitop3_b32 %[c0], %[c0], %[c1], 1 bitop3:0xa8\n\t"
      "v_cmp_ne_u32_e64 %[bad], 0, %[c0]"
      : "+{v[32:47]}"(A[0]), "+{v[48:63]}"(A[1]), "+{v[64:79]}"(A[2]), "+{v[80:95]}"(A[3]), [xa] "+v"(xa), [wa0] "+v"(wa0), [wa1] "+v"(wa1), [bad] "=&s"(bad), [c0] "=&v"(c0), [c1] "=&v"(c1), [c2] "=&v"(c2)
      : [xb] "v"(xb), [wb0] "v"(wb0), [wb1] "v"(wb1), [tag] "s"(tag), [vxb] "v"(vxb), [rx] "s"(rxs), [sel0] "v"(sel0), [sel1] "v"(sel1), [s1o0] "s"(s1o[0]), [s1o1] "s"(s1o[1]), [s1i0] "s"(s1i[0]), [s1i1] "s"(s1i[1]), [s2o0] "s"(s2o[0]), [s2o1] "s"(s2o[1]), [s2i0] "s"(s2i[0]), [s2i1] "s"(s2i[1])
      : "memory", "scc", ACLOB);
  AWAIT_2();
  asm volatile(
      "v_mfma_f32_16x16x32_bf16 v[32:35], %[sel0], v[164:167], v[32:35]\n\t"
      "v_mfma_f32_16x16x32_bf16 v[36:39], %[sel1], v[164:167], v[36:39]\n\t"
      "v_mfma_f32_16x16x32_bf16 v[40:43], %[sel0], v[172:175], v[40:43]\n\t"
      "v_mfma_f32_16x16x32_bf16 v[44:47], %[sel1], v[172:175], v[44:47]\n\t"
      "v_mfma_f32_16x16x32_bf16 v[48:51], %[sel0], v[180:183], v[48:51]\n\t"
      "s_waitcnt vmcnt(0)\n\t"
      "v_bitop3_b32 %[c0], v160, v162, v168 bitop3:0xfe\n\t"
      "v_mfma_f32_16x16x32_bf16 v[52:55], %[sel1], v[180:183], v[52:55]\n\t"
      "v_bitop3_b32 %[c0], %[c0], v170, v176 bitop3:0xfe\n\t"
      "v_bitop3_b32 %[c0], %[c0], v178, v184 bitop3:0xfe\n\t"
      "v_mfma_f32_16x16x32_bf16 v[56:59], %[sel0], v[188:191], v[56:59]\n\t"
      "v_bitop3_b32 %[c0], %[c0], v186, 1 bitop3:0xa8\n\t"
      "v_cmp_ne_u32_e64 %[bad], 0, %[c0]\n\t"
      "v_mfma_f32_16x16x32_bf16 v[60:63], %[sel1], v[188:191], v[60:63]"
      : "+{v[32:47]}"(A[0]), "+{v[48:63]}"(A[1]), [bad] "=&s"(bad), [c0] "=&v"(c0)
      : [sel0] "v"(sel0), [sel1] "v"(sel1)
      : "memory", "scc", ACLOB);
  AWAIT_B();
  asm volatile(
      "v_mfma_f32_16x16x32_bf16 v[32:35], %[sel0], v[160:163], v[32:35]\n\t"
      "v_mfma_f32_16x16x32_bf16 v[36:39], %[sel1], v[160:163], v[36:39]\n\t"
      "v_mov_b32 v164, %[mx]\n\t"
      "v_mov_b32 v165, %[seq]\n\t"
      "v_mfma_f32_16x16x32_bf16 v[40:43], %[sel0], v[168:171], v[40:43]\n\t"
      "v_mfma_f32_16x16x32_bf16 v[44:47], %[sel1], v[168:171], v[44:47]\n\t"
      "v_maximum3_f32 v164, |v32|, |v33|, v164\n\t"
      "v_maximum3_f32 v164, |v34|, |v35|, v164\n\t"
      "v_mfma_f32_16x16x32_bf16 v[48:51], %[sel0], v[176:179], v[48:51]\n\t"
      "v_maximum3_f32 v164, |v36|, |v37|, v164\n\t"
      "v_maximum3_f32 v164, |v38|, |v39|, v164\n\t"
      "v_mfma_f32_16x16x32_bf16 v[52:55], %[sel1], v[176:179], v[52:55]\n\t"
      "v_maximum3_f32 v164, |v40|, |v41|, v164\n\t"
      "v_maximum3_f32 v164, |v42|, |v43|, v164\n\t"
      "v_mfma_f32_16x16x32_bf16 v[56:59], %[sel0], v[184:187], v[56:59]\n\t"
      "v_maximum3_f32 v164, |v44|, |v45|, v164\n\t"
      "v_maximum3_f32 v164, |v46|, |v47|, v164\n\t"
      "v_mfma_f32_16x16x32_bf16 v[60:63], %[sel1], v[184:187], v[60:63]\n\t"
      "v_maximum3_f32 v164, |v48|, |v49|, v164\n\t"
      "v_maximum3_f32 v164, |v50|, |v51|, v164\n\t"
      "v_maximum3_f32 v164, |v52|, |v53|, v164\n\t"
      "v_maximum3_f32 v164, |v54|, |v55|, v164\n\t"
      "v_maximum3_f32 v164, |v56|, |v57|, v164\n\t"
      "v_maximum3_f32 v164, |v58|, |v59|, v164\n\t"
      "s_nop 1\n\t"
      "v_maximum3_f32 v164, |v60|, |v61|, v164\n\t"
      "v_maximum3_f32 v164, |v62|, |v63|, v164\n\t"
      "v_and_b32 v164, 0x7fffffff, v164\n\t"
      "s_nop 1\n\t"
      "v_max_u32_dpp v164, v164, v164 quad_perm:[1,0,3,2] row_mask:0xf bank_mask:0xf\n\t"
      "s_nop 1\n\t"
      "v_max_u32_dpp v164, v164, v164 quad_perm:[2,3,0,1] row_mask:0xf bank_mask:0xf\n\t"
      "s_nop 1\n\t"
      "v_max_u32_dpp v164, v164, v164 row_half_mirror row_mask:0xf bank_mask:0xf\n\t"
      "s_nop 1\n\t"
      "v_max_u32_dpp v164, v164, v164 row_mirror row_mask:0xf bank_mask:0xf\n\t"
      "s_nop 1\n\t"
      "v_max_u32_dpp v164, v164, v164 row_bcast:15 row_mask:0xa bank_mask:0xf\n\t"
      "s_nop 1\n\t"
      "v_max_u32_dpp v164, v164, v164 row_bcast:31 row_mask:0xc bank_mask:0xf\n\t"
      "buffer_store_dwordx2 v[164:165], %[vag], %[rx], %[sag] offen"
      : "+{v[32:47]}"(A[0]), "+{v[48:63]}"(A[1])
      : [rx] "s"(rxs), [sel0] "v"(sel0), [sel1] "v"(sel1), [mx] "s"(mx), [seq] "s"(seq), [vag] "v"(vag), [sag] "s"(sag)
      : "memory", "scc", ACLOB);
  STAMP(4);
  wg_barrier();
  u32x2 po, pw;
  asm volatile("s_load_dwordx2 %0, %2, %3 offset:%5\n\ts_load_dwordx2 %1, %2, %4 offset:%6\n\ts_waitcnt lgkmcnt(0)" : "=&s"(po), "=&s"(pw)
               : "s"(__builtin_amdgcn_kernarg_segment_ptr()), "s"((send ? dst : NR) * 8), "s"(dst * 8), "i"(KA_OUT), "i"(KA_WS) : "memory");
  const bool direct = send || dst == rank;
  const i32x4 rout = {(int)po[0], (int)po[1], NR * M * OUT_LD, 0x00020000}, rpeer = {(int)pw[0], (int)pw[1], (int)WS_STAMP, 0x00020000};
  i32x4 ro = direct ? rout : rxs;
  unsigned vo = direct ? (unsigned)((rank * M + m0 + wm + kh * 32 + fr) * OUT_LD + tile * 128 + wn + fq * 16) : ((wv & 1) * 4 + (fr >> 3)) * 2048 + skip3((fr & 7) * 128 + wn + fq * 16);
  unsigned so0 = direct ? 0u : stg(dst, tag, slot), so1 = so0 + (direct ? 16 * OUT_LD : 4096), xm = direct ? 0u : 0x7f7f7f7fu, lm = l0 + LDS_MISC, inv;
  asm volatile("" : "+s"(ro), "+v"(vo), "+s"(so0), "+s"(so1), "+s"(xm), "+v"(lm));
  if (b < 8) wg_barrier();
  wg_barrier();
  STAMP(5);
  asm volatile(
      "ds_read_b32 v128, %[lm]\n\t"
      "s_waitcnt lgkmcnt(0)\n\t"
      "v_pk_mul_f32 v[48:49], v[48:49], v[128:129] op_sel_hi:[1,0]\n\t"
      "v_pk_mul_f32 v[56:57], v[56:57], v[128:129] op_sel_hi:[1,0]\n\t"
      "v_pk_mul_f32 v[52:53], v[52:53], v[128:129] op_sel_hi:[1,0]\n\t"
      "v_pk_mul_f32 v[60:61], v[60:61], v[128:129] op_sel_hi:[1,0]\n\t"
      "v_pk_mul_f32 v[50:51], v[50:51], v[128:129] op_sel_hi:[1,0]\n\t"
      "v_pk_mul_f32 v[58:59], v[58:59], v[128:129] op_sel_hi:[1,0]\n\t"
      "v_pk_mul_f32 v[54:55], v[54:55], v[128:129] op_sel_hi:[1,0]\n\t"
      "v_pk_mul_f32 v[62:63], v[62:63], v[128:129] op_sel_hi:[1,0]\n\t"
      "v_cvt_pk_fp8_f32 v132, v48, v49\n\t"
      "v_cvt_pk_fp8_f32 v134, v56, v57\n\t"
      "v_cvt_pk_fp8_f32 v133, v52, v53\n\t"
      "v_cvt_pk_fp8_f32 v135, v60, v61\n\t"
      "v_cvt_pk_fp8_f32 v132, v50, v51 op_sel:[0,0,1]\n\t"
      "v_cvt_pk_fp8_f32 v134, v58, v59 op_sel:[0,0,1]\n\t"
      "v_cvt_pk_fp8_f32 v133, v54, v55 op_sel:[0,0,1]\n\t"
      "v_cvt_pk_fp8_f32 v135, v62, v63 op_sel:[0,0,1]\n\t"
      "v_permlane32_swap_b32 v132, v134\n\t"
      "v_pk_mul_f32 v[32:33], v[32:33], v[128:129] op_sel_hi:[1,0]\n\t"
      "v_permlane32_swap_b32 v133, v135\n\t"
      "v_pk_mul_f32 v[40:41], v[40:41], v[128:129] op_sel_hi:[1,0]\n\t"
      "v_pk_mul_f32 v[36:37], v[36:37], v[128:129] op_sel_hi:[1,0]\n\t"
      "v_permlane16_swap_b32 v132, v133\n\t"
      "v_permlane16_swap_b32 v134, v135\n\t"
      "v_xor_b32 v132, %[xm], v132\n\t"
      "v_xor_b32 v133, %[xm], v133\n\t"
      "v_xor_b32 v134, %[xm], v134\n\t"
      "v_xor_b32 v135, %[xm], v135\n\t"
      "buffer_store_dwordx4 v[132:135], %[vo], %[ro], %[so1] offen sc0 sc1\n\t"
      "v_pk_mul_f32 v[44:45], v[44:45], v[128:129] op_sel_hi:[1,0]\n\t"
      "v_pk_mul_f32 v[34:35], v[34:35], v[128:129] op_sel_hi:[1,0]\n\t"
      "v_pk_mul_f32 v[42:43], v[42:43], v[128:129] op_sel_hi:[1,0]\n\t"
      "v_pk_mul_f32 v[38:39], v[38:39], v[128:129] op_sel_hi:[1,0]\n\t"
      "v_pk_mul_f32 v[46:47], v[46:47], v[128:129] op_sel_hi:[1,0]\n\t"
      "v_cvt_pk_fp8_f32 v136, v32, v33\n\t"
      "v_cvt_pk_fp8_f32 v138, v40, v41\n\t"
      "v_cvt_pk_fp8_f32 v137, v36, v37\n\t"
      "v_cvt_pk_fp8_f32 v139, v44, v45\n\t"
      "v_cvt_pk_fp8_f32 v136, v34, v35 op_sel:[0,0,1]\n\t"
      "v_cvt_pk_fp8_f32 v138, v42, v43 op_sel:[0,0,1]\n\t"
      "v_cvt_pk_fp8_f32 v137, v38, v39 op_sel:[0,0,1]\n\t"
      "v_cvt_pk_fp8_f32 v139, v46, v47 op_sel:[0,0,1]\n\t"
      "v_permlane32_swap_b32 v136, v138\n\t"
      "v_readfirstlane_b32 %[inv], v128\n\t"
      "v_permlane32_swap_b32 v137, v139\n\t"
      "s_nop 0\n\t"
      "s_nop 0\n\t"
      "v_permlane16_swap_b32 v136, v137\n\t"
      "v_permlane16_swap_b32 v138, v139\n\t"
      "v_xor_b32 v136, %[xm], v136\n\t"
      "v_xor_b32 v137, %[xm], v137\n\t"
      "v_xor_b32 v138, %[xm], v138\n\t"
      "v_xor_b32 v139, %[xm], v139\n\t"
      "buffer_store_dwordx4 v[136:139], %[vo], %[ro], %[so0] offen sc0 sc1"
      : "+{v[32:47]}"(A[0]), "+{v[48:63]}"(A[1]), [inv] "=&s"(inv)
      : [lm] "v"(lm), [vo] "v"(vo), [ro] "s"(ro), [so1] "s"(so1), [so0] "s"(so0), [xm] "s"(xm)
      : "memory", "scc");
  STAMP(6);
  if (direct) {
    if (dst != rank) asm volatile("s_waitcnt vmcnt(0)\n\tbuffer_store_dword %0, %1, %2, %3 offen sc0 sc1"
                                  ::"v"(seq), "v"(vag), "s"(rpeer), "s"((unsigned)WS_DONE + rank * 1024 + skip3(slot * 16 + wv * 4)) : "memory");
  } else {
    const unsigned long long t6 = __builtin_amdgcn_s_memrealtime();
    const unsigned sp = stg(rank, tag, slot) + wv * 4096, sc = stg(dst, tag ^ 1u, slot) + wv * 4096;
    const unsigned vq = (unsigned)((dst * M + m0 + (wv >> 1) * 64 + kh * 32 + (wv & 1) * 16 + (lane >> 3)) * OUT_LD + ((nb + rank) & 3) * 128 + (lane & 7) * 16);
    while (__builtin_amdgcn_s_memrealtime() - t6 < MARGIN) __builtin_amdgcn_s_sleep(1);
    u32x4 v0, v1;
    asm volatile("buffer_load_dwordx4 %0, %2, %3, %4 offen sc0 sc1\n\tbuffer_load_dwordx4 %1, %2, %3, %4 offen offset:2048 sc0 sc1\n\t"
                 "buffer_store_dwordx4 %7, %2, %5, %6 offen sc0 sc1\n\tbuffer_store_dwordx4 %7, %2, %5, %6 offen offset:2048 sc0 sc1\n\ts_waitcnt vmcnt(2)"
                 : "=&v"(v0), "=&v"(v1) : "v"(vxb), "s"(rpeer), "s"(sp), "s"(rxs), "s"(sc), "v"(u32x4{0u, 0u, 0u, 0u}) : "memory");
    STAMP(13);
    for (unsigned it = 1; __builtin_expect(__any(min(min(min(v0[0], v0[1]), min(v0[2], v0[3])), min(min(v1[0], v1[1]), min(v1[2], v1[3]))) == 0u), 0) && inv != NAN_BITS; ++it) {
      if ((it & 63) == 0 && expired((unsigned*)(ws + WS_TMO), seq, t6)) break;
      asm volatile("buffer_load_dwordx4 %0, %2, %3, %4 offen sc0 sc1\n\tbuffer_load_dwordx4 %1, %2, %3, %4 offen offset:2048 sc0 sc1\n\ts_waitcnt vmcnt(0)"
                   : "=&v"(v0), "=&v"(v1) : "v"(vxb), "s"(rpeer), "s"(sp) : "memory");
    }
    asm volatile("buffer_store_dwordx4 %0, %2, %3, 0 offen sc0 sc1\n\tbuffer_store_dwordx4 %1, %2, %3, %4 offen sc0 sc1"
                 ::"v"(v0 ^ 0x7f7f7f7fu), "v"(v1 ^ 0x7f7f7f7fu), "v"(vq), "s"(rout), "s"(8 * OUT_LD) : "memory");
  }
  __builtin_amdgcn_s_waitcnt(0xC07F);
  STAMPS_OUT(ws);
}

}

extern "C" void kernel_launch(int rank, int nranks, ncclComm_t comm,
                              hipStream_t stream,
                              void* const* d_in, const int* in_sizes, int n_in,
                              void* d_out, int out_size,
                              const int* in_dtypes, int out_dtype,
                              void* const* d_out_all,
                              void* const* d_ws_all, size_t ws_bytes) {
  (void)comm; (void)in_dtypes; (void)out_dtype; (void)d_out;
  if (nranks != NR || n_in < 2 || in_sizes[0] != M * K || in_sizes[1] != K * N || out_size != NR * M * OUT_LD || ws_bytes < WS_NEED ||
      (unsigned)rank >= (unsigned)NR) {
    fprintf(stderr, "kernel_launch: unexpected problem shape\n");
    return;
  }
  HIP_CHECK(hipFuncSetAttribute((const void*)gemm_a2a_fp8, hipFuncAttributeMaxDynamicSharedMemorySize, LDS_BYTES));
  Params P;
  const unsigned long long px = (unsigned long long)d_in[0], pw = (unsigned long long)d_in[1];
  P.sx = i32x4{(int)px, (int)(px >> 32), M * K * 2, 0x00020000};
  P.sw = i32x4{(int)pw, (int)(pw >> 32), K * N * 2, 0x00020000};
  for (int p = 0; p < NR; ++p) {
    P.out[p] = (unsigned char*)d_out_all[p];
    P.ws[p] = (unsigned char*)d_ws_all[p];
  }
  P.out[NR] = P.out[rank];
  P.ws_self = (unsigned char*)d_ws_all[rank];
  P.rank = rank;
  P.pad = 0;
  make_order(P.ord);
  gemm_a2a_fp8<<<NWG, NTHR, LDS_BYTES, stream>>>(P);
  HIP_CHECK(hipGetLastError());
}
```
